```python
import numpy as np

kernel_name = "gpu_a2a_gemm_m4096_k4096_n16384_w8a8_dequant"


ABSMAX_RTOL = 0.001

COMPUTE_DTYPE = "int8"
EPILOGUE = "dequant"
IN_QUANT = "int8"
OUT_QUANT = None
IN_DTYPES = {"x": "int8", "w_mat": "int8",
             "scale_x": "float32", "scale_w": "float32"}
WORLD_SIZE = 8

def setup_inputs():

    x = np.random.randint(-127, 128, size=(4096, 4096)).astype(np.int8)
    w_mat = np.random.randint(-127, 128, size=(4096, 16384)).astype(np.int8)
    scale_x = (0.02 * (1.0 + np.random.rand(1))).astype(np.float32)
    scale_w = (0.02 * (1.0 + np.random.rand(1))).astype(np.float32)
    return {"x": x, "w_mat": w_mat, "scale_x": scale_x, "scale_w": scale_w}

def reference(x, w_mat, scale_x, scale_w):
    acc = x.astype(np.float64) @ w_mat.astype(np.float64)
    y = (acc.astype(np.float32) * (scale_x * scale_w)).astype(np.float32)
    return y
```

```cpp
#include <hip/hip_runtime.h>
#include <rccl/rccl.h>
#include <cstdio>
#include <cstdint>

#define HIP_CHECK(expr) do { hipError_t _e = (expr); if (_e != hipSuccess) { \
    fprintf(stderr, "HIP error %s at %s:%d\n", hipGetErrorString(_e), __FILE__, __LINE__); \
    return; } } while (0)

typedef int v4i __attribute__((ext_vector_type(4)));
typedef int v2i __attribute__((ext_vector_type(2)));
typedef unsigned u32x4 __attribute__((ext_vector_type(4)));
typedef float f32x4 __attribute__((ext_vector_type(4)));

constexpr int NR = 8;
constexpr int NN = 16384, KK = 4096, ML = 512, KL = 512;
constexpr unsigned STG = ML * KK;
constexpr unsigned OFF_FLAG = 2 * STG;
constexpr unsigned OFF_CNT = OFF_FLAG + 65536;
constexpr unsigned OFF_TMO = OFF_CNT + 131072;
constexpr unsigned OFF_DBG = OFF_TMO + 4096;
constexpr size_t WS_NEED = OFF_DBG + 262144;
constexpr unsigned long long DEADLINE_TICKS = 50000000ull;
#define SEND_DELAY 50

constexpr unsigned L_RING = 0;
constexpr unsigned L_CTL = 131072;
constexpr unsigned L_RDY = L_CTL;
constexpr unsigned L_SCALE = L_CTL + 4;
constexpr unsigned L_PROG = L_CTL + 16;
constexpr unsigned L_SX = L_CTL + 64;
constexpr unsigned L_SIZE = L_CTL + 512;

#define AS3 __attribute__((address_space(3)))
#define LDSP(p) ((AS3 v2i*)(size_t)(p))
#define SB() __builtin_amdgcn_sched_barrier(0)
__device__ __forceinline__ unsigned rfl(unsigned x) { return __builtin_amdgcn_readfirstlane(x); }
__device__ __forceinline__ u32x4 mk_srd(const void* p, unsigned bytes) {
  unsigned long long a = (unsigned long long)p;
  u32x4 r; r[0] = rfl((unsigned)a); r[1] = rfl((unsigned)(a >> 32) & 0xffff); r[2] = bytes; r[3] = 0x00020000; return r;
}
template<int OFF> __device__ __forceinline__ void xload(v4i& d, unsigned voff, u32x4 srd, unsigned soff) {
  asm volatile("buffer_load_dwordx4 %0, %1, %2, %3 offen offset:%4 sc0 sc1" : "+v"(d) : "v"(voff), "s"(srd), "s"(soff), "i"(OFF) : "memory");
}
__device__ __forceinline__ void setm0(unsigned v) { asm volatile("s_mov_b32 m0, %0\n\ts_nop 0" :: "s"(v) : "memory"); }
template<int OFF> __device__ __forceinline__ void glds(unsigned voff, u32x4 srd, unsigned soff) {
  asm volatile("buffer_load_dwordx4 %0, %1, %2 offen offset:%3 lds" :: "v"(voff), "s"(srd), "s"(soff), "i"(OFF) : "memory");
}
__device__ __forceinline__ void wait_ready(unsigned cur, unsigned need, unsigned addr) {
  unsigned vt, st;
  asm volatile(
    "s_andn2_b32 %1, %3, %2\n\t"
    "s_cbranch_scc0 2f\n"
    "1:\n\t"
    "s_sleep 1\n\t"
    "ds_read_b32 %0, %4\n\t"
    "s_waitcnt lgkmcnt(0)\n\t"
    "v_readfirstlane_b32 %1, %0\n\t"
    "s_nop 3\n\t"
    "s_andn2_b32 %1, %3, %1\n\t"
    "s_cbranch_scc1 1b\n"
    "2:"
    : "=&v"(vt), "=&s"(st) : "s"(cur), "s"(need), "v"(addr) : "scc", "memory");
}
#define STAMP(i)

struct Args { char* ws[NR]; };

__device__ __forceinline__ unsigned step_of(unsigned u, unsigned rank) {
  if (u < 8) return rank * 8 + u;
  unsigned v = u - 8, s = v / 7, i = v - s * 7;
  return ((rank + 1 + i) & 7) * 8 + s;
}

__global__ __launch_bounds__(448, 2) void a2a_gemm(const int8_t* __restrict__ X, const int8_t* __restrict__ W,
    const float* __restrict__ sxp, const float* __restrict__ swp, float* __restrict__ out, Args args, unsigned rank) {
  __shared__ __attribute__((aligned(1024))) char lds[L_SIZE];
  const unsigned b = blockIdx.x, xcd = b & 7, jj = b >> 3, nt = xcd * 16 + (jj >> 1), mh = jj & 1;
  const unsigned wv = rfl(threadIdx.x >> 6), lane = threadIdx.x & 63;
  const unsigned n0 = nt * 128;
  const unsigned lb = (unsigned)(size_t)(AS3 char*)lds;
  char* ws = args.ws[0];

  if (wv == 6) {
    __builtin_amdgcn_s_setprio(3);
    unsigned long long t0 = __builtin_amdgcn_s_memrealtime();
    const unsigned qi = b >> 5, q = (rank + qi) & 7, s = (b & 31) >> 2, c4 = b & 3;
    unsigned call = __hip_atomic_fetch_add((unsigned*)(ws + OFF_CNT + (b * 2) * 256) + lane, 1u, __ATOMIC_RELAXED, __HIP_MEMORY_SCOPE_AGENT);
    char* dws = args.ws[qi & 7];
    const int8_t* src = X + (size_t)(q * ML + c4 * 128 + (lane & 15)) * KL + s * 64 + (lane >> 4) * 16;
    u32x4 f[8];
    #pragma unroll
    for (int i = 0; i < 8; ++i) f[i] = *(const u32x4*)(src + i * 16 * KL);
    __builtin_amdgcn_s_barrier();
    asm volatile("s_waitcnt vmcnt(0)" : "+v"(f[0]), "+v"(f[1]), "+v"(f[2]), "+v"(f[3]), "+v"(f[4]), "+v"(f[5]), "+v"(f[6]), "+v"(f[7]), "+v"(call) :: "memory");
    call = rfl(call) + 1;
    const unsigned t = rank * 8 + s;
    char* dst = dws + (call & 1) * STG + t * 32768 + c4 * 8192 + lane * 16;
    STAMP(8);
    if (qi != 0) { while (__builtin_amdgcn_s_memrealtime() - t0 < (unsigned long long)(s * SEND_DELAY)) __builtin_amdgcn_s_sleep(1); }
    #pragma unroll
    for (int i = 0; i < 4; ++i)
      asm volatile("global_store_dwordx4 %0, %1, off offset:%2 sc0 sc1\n\ts_nop 1" :: "v"(dst), "v"(f[i]), "i"(i * 1024) : "memory");
    dst += 4096;
    #pragma unroll
    for (int i = 0; i < 4; ++i)
      asm volatile("global_store_dwordx4 %0, %1, off offset:%2 sc0 sc1\n\ts_nop 1" :: "v"(dst), "v"(f[4 + i]), "i"(i * 1024) : "memory");
    asm volatile("s_waitcnt vmcnt(0)" ::: "memory");
    if (lane == 0) __hip_atomic_store((unsigned*)(dws + OFF_FLAG + (t * 4 + c4) * 128), call, __ATOMIC_RELAXED, __HIP_MEMORY_SCOPE_SYSTEM);
    STAMP(9);
    return;
  }
  if (wv == 5) {
    unsigned call = __hip_atomic_fetch_add((unsigned*)(ws + OFF_CNT + (b * 2 + 1) * 256) + lane, 1u, __ATOMIC_RELAXED, __HIP_MEMORY_SCOPE_AGENT);
    float scale = sxp[0] * swp[0];
    __builtin_amdgcn_s_barrier();
    SB();
    call = rfl(call) + 1;
    const unsigned t = step_of(lane, rank);
    *(AS3 unsigned*)(size_t)(lb + L_SX + lane * 4) = (call & 1) * STG + t * 32768;
    if (lane < 4) *(AS3 unsigned*)(size_t)(lb + L_SX + 256 + lane * 4) = 2 * STG;
    unsigned* f0 = (unsigned*)(ws + OFF_FLAG + (t * 4 + mh * 2) * 128);
    unsigned* tmo = (unsigned*)(ws + OFF_TMO);
    unsigned long long t0 = __builtin_amdgcn_s_memrealtime();
    bool ok = false;
    unsigned pub = 0;
    while (true) {
      if (!ok) {
        unsigned a = __hip_atomic_load(f0, __ATOMIC_RELAXED, __HIP_MEMORY_SCOPE_SYSTEM);
        unsigned c = __hip_atomic_load(f0 + 32, __ATOMIC_RELAXED, __HIP_MEMORY_SCOPE_SYSTEM);
        ok = a >= call && c >= call;
      }
      unsigned long long miss = __builtin_amdgcn_ballot_w64(!ok);
      if (miss == 0) break;
      unsigned g = (unsigned)__builtin_ctzll(miss) >> 2;
      if (g > pub) {
        pub = g;
        asm volatile("s_waitcnt lgkmcnt(0)" ::: "memory");
        *(volatile AS3 unsigned short*)(size_t)(lb + L_RDY) = (unsigned short)((1u << g) - 1);
      }
      if (__hip_atomic_load(tmo, __ATOMIC_RELAXED, __HIP_MEMORY_SCOPE_AGENT) == call
          || __builtin_amdgcn_s_memrealtime() - t0 > DEADLINE_TICKS) {
        if (lane == 0) __hip_atomic_store(tmo, call, __ATOMIC_RELAXED, __HIP_MEMORY_SCOPE_AGENT);
        break;
      }
      __builtin_amdgcn_s_sleep(2);
    }
    asm volatile("s_waitcnt lgkmcnt(0)" ::: "memory");
    *(volatile AS3 unsigned short*)(size_t)(lb + L_RDY) = (unsigned short)0x7fff;
    STAMP(14);
    asm volatile("s_waitcnt vmcnt(0)" ::: "memory");
    if (__hip_atomic_load(tmo, __ATOMIC_RELAXED, __HIP_MEMORY_SCOPE_AGENT) == call) scale = __builtin_nanf("");
    *(volatile AS3 float*)(size_t)(lb + L_SCALE) = scale;
    asm volatile("s_waitcnt lgkmcnt(0)" ::: "memory");
    *(volatile AS3 unsigned short*)(size_t)(lb + L_RDY) = (unsigned short)0xffff;
    return;
  }
  if (wv == 4) {
    u32x4 srdW = mk_srd(W, (unsigned)KK * NN);
    unsigned vo[8];
    const unsigned r3 = lane >> 3;
    #pragma unroll
    for (int i = 0; i < 8; ++i)
      vo[i] = (i * 8 + r3) * NN + n0 + (((lane & 7) ^ (((r3 >> 1) & 3) | ((i & 2) << 1))) << 4) - (i & 3) * 1024;
    const unsigned tl = step_of(lane, rank) << 20;
    for (unsigned g = 0; g < 16; ++g) {
      if (g == 1) __builtin_amdgcn_s_barrier();
      if (g >= 4) {
        while (true) {
          u32x4 p = *(volatile AS3 u32x4*)(size_t)(lb + L_PROG);
          unsigned mn = min(min(p[0], p[1]), min(p[2], p[3]));
          if (rfl(mn) >= g - 3) break;
          __builtin_amdgcn_s_sleep(2);
        }
      }
      #pragma unroll
      for (int j = 0; j < 4; ++j) {
        unsigned so = __builtin_amdgcn_readlane(tl, g * 4 + j);
        asm volatile("s_nop 4" : "+s"(so));
        const unsigned m = lb + L_RING + ((g & 3) * 4 + j) * 8192;
        setm0(m);
        glds<0>(vo[0], srdW, so); glds<1024>(vo[1], srdW, so); glds<2048>(vo[2], srdW, so); glds<3072>(vo[3], srdW, so);
        setm0(m + 4096);
        glds<0>(vo[4], srdW, so); glds<1024>(vo[5], srdW, so); glds<2048>(vo[6], srdW, so); glds<3072>(vo[7], srdW, so);
        if (j == 2) {
          asm volatile("s_waitcnt vmcnt(24)" ::: "memory");
          if (g) *(volatile AS3 unsigned short*)(size_t)(lb + L_RDY + 2) = (unsigned short)((1u << g) - 1);
        }
      }
    }
    asm volatile("s_waitcnt vmcnt(0)" ::: "memory");
    *(volatile AS3 unsigned short*)(size_t)(lb + L_RDY + 2) = (unsigned short)0xffff;
    return;
  }

  const unsigned wid = wv;
  if (wid == 0 && lane < 8) *(AS3 unsigned*)(size_t)(lb + L_CTL + lane * 4) = 0;
  asm volatile("s_waitcnt lgkmcnt(0)" ::: "memory");
  __builtin_amdgcn_s_barrier();
  if (wid == 0) STAMP(0);
  u32x4 srdX = mk_srd(ws, 2 * STG);
  const unsigned voffX = (mh * 16 + wid * 4) * 1024 + lane * 16;
  const unsigned g4 = lane >> 4, q8 = (lane & 15) >> 1;
  const unsigned a0 = lb + L_RING + (g4 * 16 + q8) * 128 + ((((q8 >> 1) & 3) | ((g4 & 1) << 2)) << 4) + (lane & 1) * 8;
  unsigned ad[8];
  #pragma unroll
  for (int c = 0; c < 8; ++c) ad[c] = a0 ^ (c << 4);
  unsigned actl = lb + L_RDY;
  asm volatile("" : "+v"(actl));
  unsigned aprog = lb + L_PROG + wid * 4;
  asm volatile("" : "+v"(aprog));
  v4i xr[4][4];
  #pragma unroll
  for (int i = 0; i < 4; ++i)
    #pragma unroll
    for (int m = 0; m < 4; ++m) xr[i][m] = (v4i){0, 0, 0, 0};
  v4i acc[4][8];
  #pragma unroll
  for (int m = 0; m < 4; ++m)
    #pragma unroll
    for (int c = 0; c < 8; ++c) acc[m][c] = (v4i){0, 0, 0, 0};
  wait_ready(0, 0x00000001u, actl);
  unsigned sx[4];
  {
    u32x4 s4 = *(volatile AS3 u32x4*)(size_t)(lb + L_SX);
    sx[0] = rfl(s4[0]); sx[1] = rfl(s4[1]); sx[2] = rfl(s4[2]); sx[3] = rfl(s4[3]);
    asm volatile("s_nop 4" : "+s"(sx[0]), "+s"(sx[1]), "+s"(sx[2]), "+s"(sx[3]));
  }
  #pragma unroll
  for (int t = 0; t < 3; ++t) {
    xload<0>(xr[t][0], voffX, srdX, sx[t]); xload<1024>(xr[t][1], voffX, srdX, sx[t]);
    xload<2048>(xr[t][2], voffX, srdX, sx[t]); xload<3072>(xr[t][3], voffX, srdX, sx[t]);
  }
  wait_ready(0, 0x00010000u, actl);
  if (wid == 0) STAMP(1);
  v2i wf[8][2];
  #pragma unroll
  for (int c = 0; c < 7; ++c) {
    wf[c][0] = __builtin_amdgcn_ds_read_tr8_b64_v2i32(LDSP(ad[c]));
    wf[c][1] = __builtin_amdgcn_ds_read_tr8_b64_v2i32(LDSP(ad[c] + 1024));
  }
  unsigned rdyv = *(volatile AS3 unsigned*)(size_t)actl;
  unsigned rdy = rfl(rdyv);
  unsigned need = 0x00020002u;
  unsigned prog = 0;
  unsigned asx = lb + L_SX + 16;
  asm volatile("" : "+v"(asx), "+v"(prog));
  u32x4 s4;
  SB();
  for (int G = 0; G < 8; ++G) {
    #pragma unroll
    for (int i = 0; i < 8; ++i) {
      asm volatile("s_waitcnt vmcnt(8)" : "+v"(xr[i & 3][0]), "+v"(xr[i & 3][1]), "+v"(xr[i & 3][2]), "+v"(xr[i & 3][3]) :: "memory");
      if ((i & 3) == 0) { wait_ready(rdy, need, actl); need <<= 1; }
      SB();
      const unsigned sX = sx[(i + 3) & 3];
      #pragma unroll
      for (int c = 0; c < 8; ++c) {
        const int pc = (c + 7) & 7;
        const int rslot = (c == 0) ? i : ((i + 1) & 7);
        if (c == 0) { wf[7][0] = __builtin_amdgcn_ds_read_tr8_b64_v2i32(LDSP(ad[7] + rslot * 8192)); SB(); }
        v4i a = {wf[c][0][0], wf[c][0][1], wf[c][1][0], wf[c][1][1]};
        acc[0][c] = __builtin_amdgcn_mfma_i32_16x16x64_i8(a, xr[i & 3][0], acc[0][c], 0, 0, 0);
        if (c == 0) { wf[7][1] = __builtin_amdgcn_ds_read_tr8_b64_v2i32(LDSP(ad[7] + rslot * 8192 + 1024)); if (i == 7) ad[7] ^= 0x10000; }
        else { if (i == 7) ad[pc] ^= 0x10000; wf[pc][0] = __builtin_amdgcn_ds_read_tr8_b64_v2i32(LDSP(ad[pc] + rslot * 8192)); }
        SB();
        acc[1][c] = __builtin_amdgcn_mfma_i32_16x16x64_i8(a, xr[i & 3][1], acc[1][c], 0, 0, 0);
        if (c != 0) wf[pc][1] = __builtin_amdgcn_ds_read_tr8_b64_v2i32(LDSP(ad[pc] + rslot * 8192 + 1024));
        SB();
        acc[2][c] = __builtin_amdgcn_mfma_i32_16x16x64_i8(a, xr[i & 3][2], acc[2][c], 0, 0, 0);
        if (c == 1) xload<0>(xr[(i + 3) & 3][0], voffX, srdX, sX);
        if (c == 3) xload<1024>(xr[(i + 3) & 3][1], voffX, srdX, sX);
        if (c == 5) xload<2048>(xr[(i + 3) & 3][2], voffX, srdX, sX);
        if (c == 7) xload<3072>(xr[(i + 3) & 3][3], voffX, srdX, sX);
        if ((i & 3) == 0) {
          if (c == 0) { *(volatile AS3 unsigned*)(size_t)aprog = prog; prog += 1; }
          if (c == 2) s4 = *(volatile AS3 u32x4*)(size_t)(asx + (i >> 2) * 16);
          if (c == 6) { sx[0] = rfl(s4[0]); sx[1] = rfl(s4[1]); }
        }
        if ((i & 3) == 1 && c == 0) { sx[2] = rfl(s4[2]); sx[3] = rfl(s4[3]); if (i == 5) asx += 32; }
        if ((i & 3) == 3 && c == 4) rdyv = *(volatile AS3 unsigned*)(size_t)actl;
        if ((i & 3) == 3 && c == 6) rdy = rfl(rdyv);
        SB();
        acc[3][c] = __builtin_amdgcn_mfma_i32_16x16x64_i8(a, xr[i & 3][3], acc[3][c], 0, 0, 0);
        SB();
      }
    }
  }
  SB();
  asm volatile("s_waitcnt vmcnt(0)" :: "v"(xr[0][0]), "v"(xr[0][1]), "v"(xr[0][2]), "v"(xr[0][3]), "v"(xr[1][0]), "v"(xr[1][1]), "v"(xr[1][2]), "v"(xr[1][3]),
               "v"(xr[2][0]), "v"(xr[2][1]), "v"(xr[2][2]), "v"(xr[2][3]), "v"(xr[3][0]), "v"(xr[3][1]), "v"(xr[3][2]), "v"(xr[3][3]) : "memory");
  SB();
  if (wid == 0) STAMP(2);
  const float scale = *(volatile AS3 float*)(size_t)(lb + L_SCALE);
  #pragma unroll
  for (int m = 0; m < 4; ++m) {
    float* p = out + (size_t)(mh * 256 + (wid * 4 + m) * 16 + (lane & 15)) * NN + n0 + (lane >> 4) * 4;
    #pragma unroll
    for (int c = 0; c < 8; ++c) {
      f32x4 v = {acc[m][c][0] * scale, acc[m][c][1] * scale, acc[m][c][2] * scale, acc[m][c][3] * scale};
      asm volatile("global_store_dwordx4 %0, %1, off offset:%2 sc0 sc1\n\ts_nop 1" :: "v"(p), "v"(v), "i"(c * 64) : "memory");
    }
  }
}

extern "C" void kernel_launch(int rank, int nranks, ncclComm_t comm, hipStream_t stream,
                              void* const* d_in, const int* in_sizes, int n_in,
                              void* d_out, int out_size, const int* in_dtypes, int out_dtype,
                              void* const* d_out_all, void* const* d_ws_all, size_t ws_bytes) {
  if (nranks != NR || (unsigned)rank >= (unsigned)NR || n_in < 4 || ws_bytes < WS_NEED
      || in_sizes[0] != KK * KL || in_sizes[1] != KK * NN || out_size != ML * NN) {
    fprintf(stderr, "kernel_launch: unexpected shapes\n");
    return;
  }
  Args a;
  for (int p = 0; p < NR; ++p) a.ws[p] = (char*)d_ws_all[(rank + p) % NR];
  a2a_gemm<<<256, 448, 0, stream>>>((const int8_t*)d_in[0], (const int8_t*)d_in[1], (const float*)d_in[2], (const float*)d_in[3],
                                    (float*)d_out, a, (unsigned)rank);
  HIP_CHECK(hipGetLastError());
}
```

```cpp
#include <hip/hip_runtime.h>
#include <rccl/rccl.h>
#include <cstdio>
#include <cstdint>

#define HIP_CHECK(expr) do { hipError_t _e = (expr); if (_e != hipSuccess) { \
    fprintf(stderr, "HIP error %s at %s:%d\n", hipGetErrorString(_e), __FILE__, __LINE__); \
    return; } } while (0)

typedef int v4i __attribute__((ext_vector_type(4)));
typedef int v2i __attribute__((ext_vector_type(2)));
typedef unsigned u32x4 __attribute__((ext_vector_type(4)));
typedef float f32x4 __attribute__((ext_vector_type(4)));

constexpr int NR = 8;
constexpr int NN = 16384, KK = 4096, ML = 512, KL = 512;
constexpr unsigned STG = ML * KK;
constexpr unsigned OFF_FLAG = 2 * STG;
constexpr unsigned OFF_CNT = OFF_FLAG + 65536;
constexpr unsigned OFF_TMO = OFF_CNT + 131072;
constexpr unsigned OFF_DBG = OFF_TMO + 4096;
constexpr size_t WS_NEED = OFF_DBG + 262144;
constexpr unsigned long long DEADLINE_TICKS = 50000000ull;
#define SEND_DELAY 300

#define LEAD 2
#define REP 16
#define POLLN "24"
#define POLLS "1"
#define WPOL ""
constexpr unsigned L_RING = 0;
constexpr unsigned L_CTL = 131072;
constexpr unsigned L_RDY = L_CTL;
constexpr unsigned L_SCALE = L_CTL + 4;
constexpr unsigned L_PROG = L_CTL + 16;
constexpr unsigned L_SIZE = L_CTL + 512;

#define AS3 __attribute__((address_space(3)))
#define LDSP(p) ((AS3 v2i*)(size_t)(p))
#define SB() __builtin_amdgcn_sched_barrier(0)
__device__ __forceinline__ unsigned rfl(unsigned x) { return __builtin_amdgcn_readfirstlane(x); }
__device__ __forceinline__ u32x4 mk_srd(const void* p, unsigned bytes) {
  unsigned long long a = (unsigned long long)p;
  u32x4 r; r[0] = rfl((unsigned)a); r[1] = rfl((unsigned)(a >> 32) & 0xffff); r[2] = bytes; r[3] = 0x00020000; return r;
}
template<int OFF> __device__ __forceinline__ void xload(v4i& d, unsigned voff, u32x4 srd, unsigned soff) {
  asm volatile("buffer_load_dwordx4 %0, %1, %2, %3 offen offset:%4 sc0 sc1" : "+v"(d) : "v"(voff), "s"(srd), "s"(soff), "i"(OFF) : "memory");
}
__device__ __forceinline__ void setm0(unsigned v) { asm volatile("s_mov_b32 m0, %0\n\ts_nop 0" :: "s"(v) : "memory"); }
template<int OFF> __device__ __forceinline__ void glds(unsigned voff, u32x4 srd, unsigned soff) {
  asm volatile("buffer_load_dwordx4 %0, %1, %2 offen offset:%3 " WPOL " lds" :: "v"(voff), "s"(srd), "s"(soff), "i"(OFF) : "memory");
}
__device__ __forceinline__ void wait_ready(unsigned& cur, unsigned need, unsigned addr) {
  unsigned vt, st;
  asm volatile(
    "s_andn2_b32 %1, %3, %2\n\t"
    "s_cbranch_scc0 2f\n"
    "1:\n\t"
    "ds_read_b32 %0, %4\n\t"
    "s_waitcnt lgkmcnt(0)\n\t"
    "v_readfirstlane_b32 %2, %0\n\t"
    "s_nop 3\n\t"
    "s_andn2_b32 %1, %3, %2\n\t"
    "s_cbranch_scc0 2f\n\t"
    "s_sleep 1\n\t"
    "s_branch 1b\n"
    "2:"
    : "=&v"(vt), "=&s"(st), "+s"(cur) : "s"(need), "v"(addr) : "scc", "memory");
}
template<int OFF> __device__ __forceinline__ void xload_plain(v4i& d, unsigned voff, u32x4 srd) {
  asm volatile("buffer_load_dwordx4 %0, %1, %2, 0 offen offset:%3" : "+v"(d) : "v"(voff), "s"(srd), "i"(OFF) : "memory");
}
__device__ __forceinline__ void pfetch(unsigned& d, unsigned voff, u32x4 srd, unsigned soff) {
  asm volatile("s_nop 4\n\tbuffer_load_dword %0, %1, %2, %3 offen sc0 sc1" : "+v"(d) : "v"(voff), "s"(srd), "s"(soff) : "memory");
}
#define STAMP(i)
#define CSTAMP(i)

struct Args { char* ws[NR]; };

__device__ __forceinline__ unsigned step_of(unsigned u, unsigned rank) {
  if (u < 8) return rank * 8 + u;
  unsigned v = (u - 8) >> 1, sp = v / 7, i = v - sp * 7;
  return ((rank + 1 + i) & 7) * 8 + sp * 2 + (u & 1);
}

__global__ __launch_bounds__(448, 2) void a2a_gemm(const int8_t* __restrict__ X, const int8_t* __restrict__ W,
    const float* __restrict__ sxp, const float* __restrict__ swp, float* __restrict__ out, Args args, unsigned rank) {
  __shared__ __attribute__((aligned(1024))) char lds[L_SIZE];
  const unsigned b = blockIdx.x, xcd = b & 7, jj = b >> 3, nt = xcd * 16 + (jj >> 1), mh = jj & 1;
  const unsigned wv = rfl(threadIdx.x >> 6), lane = threadIdx.x & 63;
  const unsigned n0 = nt * 128;
  const unsigned lb = (unsigned)(size_t)(AS3 char*)lds;
  char* ws = args.ws[0];

  if (wv == 0) {
    const unsigned qi = b >> 5, q = (rank + qi) & 7, sp = (b & 31) >> 3, c8 = b & 7;
    if (qi == 0) {
      __builtin_amdgcn_s_barrier();
      return;
    }
    __builtin_amdgcn_s_setprio(3);
    unsigned long long t0 = __builtin_amdgcn_s_memrealtime();
    const int8_t* src = X + (size_t)(q * ML + c8 * 64 + (lane & 15)) * KL + sp * 128 + (lane >> 4) * 16;
    u32x4 f[8];
    #pragma unroll
    for (int i = 0; i < 8; ++i) f[i] = *(const u32x4*)(src + (i >> 1) * 16 * KL + (i & 1) * 64);
    unsigned call = __hip_atomic_fetch_add((unsigned*)(ws + OFF_CNT + (b * 2) * 256) + lane, 1u, __ATOMIC_RELAXED, __HIP_MEMORY_SCOPE_AGENT);
    char* dws = args.ws[qi & 7];
    __builtin_amdgcn_s_barrier();
    asm volatile("s_waitcnt vmcnt(0)" : "+v"(f[0]), "+v"(f[1]), "+v"(f[2]), "+v"(f[3]), "+v"(f[4]), "+v"(f[5]), "+v"(f[6]), "+v"(f[7]), "+v"(call) :: "memory");
    call = rfl(call) + 1;
    const unsigned v = sp * 7 + ((7 - qi) & 7);
    char* dst = dws + (call & 1) * STG + (8 + v * 2) * 32768 + c8 * 4096 + lane * 16;
    STAMP(8);
    while (__builtin_amdgcn_s_memrealtime() - t0 < (unsigned long long)(sp * SEND_DELAY)) __builtin_amdgcn_s_sleep(1);
    #pragma unroll
    for (int i = 0; i < 4; ++i)
      asm volatile("global_store_dwordx4 %0, %1, off offset:%2 sc0 sc1\n\ts_nop 1" :: "v"(dst), "v"(f[i * 2]), "i"(i * 1024) : "memory");
    dst += 32768;
    #pragma unroll
    for (int i = 0; i < 4; ++i)
      asm volatile("global_store_dwordx4 %0, %1, off offset:%2 sc0 sc1\n\ts_nop 1" :: "v"(dst), "v"(f[i * 2 + 1]), "i"(i * 1024) : "memory");
    asm volatile("s_waitcnt vmcnt(0)" ::: "memory");
    if (lane < REP) __hip_atomic_store((unsigned*)(dws + OFF_FLAG + (c8 >> 2) * 8192 + lane * 512) + v * 4 + (c8 & 3), call, __ATOMIC_RELAXED, __HIP_MEMORY_SCOPE_SYSTEM);
    STAMP(9);
    return;
  }
  if (wv == 2) {
    unsigned call = __hip_atomic_fetch_add((unsigned*)(ws + OFF_CNT + (b * 2 + 1) * 256) + lane, 1u, __ATOMIC_RELAXED, __HIP_MEMORY_SCOPE_AGENT);
    unsigned pd = 0;
    pfetch(pd, (jj * 64 + lane) * 128, mk_srd(X, KK * KL), rank * (ML * KL));
    float scale = sxp[0] * swp[0];
    __builtin_amdgcn_s_barrier();
    SB();
    const u32x4 srdW = mk_srd(W, (unsigned)KK * NN);
    const unsigned wb = (srdW[0] >> 7) + xcd * 16;
    unsigned badm = 0;
    for (unsigned l = 0; l < 16; ++l) if (((wb + l) & 7) == 3 || ((wb + l) & 127) == 63) badm |= 1u << l;
    if ((badm >> (jj >> 1)) & 1) badm = 0;
    {
      const unsigned so = (step_of(2 + jj, rank) << 20) + xcd * 2048;
      for (unsigned mk = badm; mk; mk &= mk - 1) pfetch(pd, lane * NN, srdW, so + __builtin_ctz(mk) * 128);
    }
    asm volatile("s_waitcnt vmcnt(0)" : "+v"(call), "+v"(pd) :: "memory");
    call = rfl(call) + 1;
    const unsigned par = call & 1;
    const char* fb = ws + OFF_FLAG + mh * 8192 + ((jj >> 1) & (REP - 1)) * 512;
    unsigned* tmo = (unsigned*)(ws + OFF_TMO);
    const u32x4 srdS = mk_srd(ws + par * STG, STG);
    unsigned long long t0 = __builtin_amdgcn_s_memrealtime();
    unsigned dead = 0;
    for (unsigned g = 2; g < 16; ++g) {
      while (!dead) {
        unsigned it;
        asm volatile("s_mov_b32 %0, " POLLN "\n"
                     "1:\n\t"
                     "s_load_dwordx8 s[64:71], %1, 0x0 glc\n\t"
                     "s_waitcnt lgkmcnt(0)\n\t"
                     "s_min_u32 s64, s64, s65\n\ts_min_u32 s66, s66, s67\n\ts_min_u32 s68, s68, s69\n\ts_min_u32 s70, s70, s71\n\t"
                     "s_min_u32 s64, s64, s66\n\ts_min_u32 s68, s68, s70\n\ts_min_u32 s64, s64, s68\n\t"
                     "s_cmp_ge_u32 s64, %2\n\t"
                     "s_cbranch_scc1 2f\n\t"
                     "s_sleep " POLLS "\n\t"
                     "s_sub_u32 %0, %0, 1\n\t"
                     "s_cmp_lg_u32 %0, 0\n\t"
                     "s_cbranch_scc1 1b\n"
                     "2:"
                     : "=&s"(it) : "s"(fb), "s"(call) : "s64", "s65", "s66", "s67", "s68", "s69", "s70", "s71", "scc", "memory");
        if (it) break;
        STAMP(7);
        unsigned fv = __hip_atomic_load((const unsigned*)fb + (lane & 7), __ATOMIC_RELAXED, __HIP_MEMORY_SCOPE_SYSTEM);
        unsigned tv = __hip_atomic_load(tmo, __ATOMIC_RELAXED, __HIP_MEMORY_SCOPE_AGENT);
        tv = rfl(tv);
        if (__builtin_amdgcn_ballot_w64(fv < call) == 0) { STAMP(6); break; }
        if (tv == call || __builtin_amdgcn_s_memrealtime() - t0 > DEADLINE_TICKS) {
          __hip_atomic_store(tmo, call, __ATOMIC_RELAXED, __HIP_MEMORY_SCOPE_AGENT);
          dead = 1;
        }
      }
      fb += 32;
      if (g < 15) *(volatile AS3 unsigned short*)(size_t)(lb + L_RDY) = (unsigned short)(((2u << g) - 2) | par);
      if (g == 3 && jj < 30) {
        const unsigned so = (step_of(34 + jj, rank) << 20) + xcd * 2048;
        for (unsigned mk = badm; mk; mk &= mk - 1) pfetch(pd, lane * NN, srdW, so + __builtin_ctz(mk) * 128);
      }
    }
    STAMP(14);
    asm volatile("s_waitcnt vmcnt(0)" : "+v"(pd) :: "memory");
    if (__hip_atomic_load(tmo, __ATOMIC_RELAXED, __HIP_MEMORY_SCOPE_AGENT) == call) scale = __builtin_nanf("");
    *(volatile AS3 float*)(size_t)(lb + L_SCALE) = scale;
    asm volatile("s_waitcnt lgkmcnt(0)" ::: "memory");
    *(volatile AS3 unsigned short*)(size_t)(lb + L_RDY) = (unsigned short)(0xfffe | par);
    return;
  }
  if (wv == 1) {
    u32x4 srdW = mk_srd(W, (unsigned)KK * NN);
    unsigned vo[8];
    const unsigned r3 = lane >> 3;
    #pragma unroll
    for (int i = 0; i < 8; ++i)
      vo[i] = (i * 8 + r3) * NN + n0 + (((lane & 7) ^ (((r3 >> 1) & 3) | ((i & 2) << 1))) << 4) - (i & 3) * 1024;
    const unsigned tl = step_of(lane, rank) << 20;
    for (unsigned g = 0; g < 16; ++g) {
      if (g == 1) __builtin_amdgcn_s_barrier();
      if (g > LEAD) {
        while (true) {
          u32x4 p = *(volatile AS3 u32x4*)(size_t)(lb + L_PROG);
          unsigned mn = min(min(p[0], p[1]), min(p[2], p[3]));
          if (rfl(mn) >= g - LEAD) break;
          __builtin_amdgcn_s_sleep(2);
        }
      }
      #pragma unroll
      for (int j = 0; j < 4; ++j) {
        unsigned so = __builtin_amdgcn_readlane(tl, g * 4 + j);
        asm volatile("s_nop 4" : "+s"(so));
        const unsigned m = lb + L_RING + ((g & 3) * 4 + j) * 8192;
        setm0(m);
        glds<0>(vo[0], srdW, so); glds<1024>(vo[1], srdW, so); glds<2048>(vo[2], srdW, so); glds<3072>(vo[3], srdW, so);
        setm0(m + 4096);
        glds<0>(vo[4], srdW, so); glds<1024>(vo[5], srdW, so); glds<2048>(vo[6], srdW, so); glds<3072>(vo[7], srdW, so);
        if (j == 0 && g == 1) {
          asm volatile("s_waitcnt vmcnt(8)" ::: "memory");
          *(volatile AS3 unsigned short*)(size_t)(lb + L_RDY + 2) = (unsigned short)1;
        }
        if (j == 2) {
          asm volatile("s_waitcnt vmcnt(24)" ::: "memory");
          if (g) *(volatile AS3 unsigned short*)(size_t)(lb + L_RDY + 2) = (unsigned short)((1u << g) - 1);
        }
      }
    }
    asm volatile("s_waitcnt vmcnt(0)" ::: "memory");
    *(volatile AS3 unsigned short*)(size_t)(lb + L_RDY + 2) = (unsigned short)0xffff;
    return;
  }

  const unsigned wid = wv - 3;
  if (wid == 0 && lane < 8) *(AS3 unsigned*)(size_t)(lb + L_CTL + lane * 4) = lane == 0 ? 2u : 0u;
  u32x4 srdX = mk_srd(X + (size_t)rank * ML * KL, ML * KL);
  unsigned voff[4];
  #pragma unroll
  for (int m = 0; m < 4; ++m) voff[m] = (mh * 256 + wid * 64 + m * 16 + (lane & 15)) * KL + (lane >> 4) * 16;
  v4i xr[4][4];
  #pragma unroll
  for (int i = 0; i < 4; ++i)
    #pragma unroll
    for (int m = 0; m < 4; ++m) xr[i][m] = (v4i){0, 0, 0, 0};
  asm volatile("s_waitcnt lgkmcnt(0)" ::: "memory");
  __builtin_amdgcn_s_barrier();
  #pragma unroll
  for (int m = 0; m < 4; ++m) xload_plain<0>(xr[0][m], voff[m], srdX);
  #pragma unroll
  for (int m = 0; m < 4; ++m) xload_plain<64>(xr[1][m], voff[m], srdX);
  #pragma unroll
  for (int m = 0; m < 4; ++m) xload_plain<128>(xr[2][m], voff[m], srdX);
  if (wid == 0) STAMP(0);
  unsigned voffX = (mh * 16 + wid * 4) * 1024 + lane * 16;
  asm volatile("" : "+v"(voffX));
  const unsigned wslo = rfl((unsigned)(size_t)ws), wshi = rfl((unsigned)((size_t)ws >> 32) & 0xffff);
  const unsigned g4 = lane >> 4, q8 = (lane & 15) >> 1;
  const unsigned a0 = lb + L_RING + (g4 * 16 + q8) * 128 + ((((q8 >> 1) & 3) | ((g4 & 1) << 2)) << 4) + (lane & 1) * 8;
  unsigned ad[8];
  #pragma unroll
  for (int c = 0; c < 8; ++c) ad[c] = a0 ^ (c << 4);
  unsigned actl = lb + L_RDY;
  asm volatile("" : "+v"(actl));
  unsigned aprog = lb + L_PROG + wid * 4;
  asm volatile("" : "+v"(aprog));
  v4i acc[4][8];
  #pragma unroll
  for (int m = 0; m < 4; ++m)
    #pragma unroll
    for (int c = 0; c < 8; ++c) acc[m][c] = (v4i){0, 0, 0, 0};
  #pragma unroll
  for (int m = 0; m < 4; ++m)
    asm volatile("" : "+v"(acc[m][0]), "+v"(acc[m][1]), "+v"(acc[m][2]), "+v"(acc[m][3]), "+v"(acc[m][4]), "+v"(acc[m][5]), "+v"(acc[m][6]), "+v"(acc[m][7]), "+v"(xr[3][m]));
  asm volatile("" : "+v"(ad[0]), "+v"(ad[1]), "+v"(ad[2]), "+v"(ad[3]), "+v"(ad[4]), "+v"(ad[5]), "+v"(ad[6]), "+v"(ad[7]));
  unsigned rdy = 0;
  wait_ready(rdy, 0x00010000u, actl);
  if (wid == 0) { STAMP(1); CSTAMP(4); }
  v2i wf[8][2];
  #pragma unroll
  for (int c = 0; c < 7; ++c) {
    wf[c][0] = __builtin_amdgcn_ds_read_tr8_b64_v2i32(LDSP(ad[c]));
    wf[c][1] = __builtin_amdgcn_ds_read_tr8_b64_v2i32(LDSP(ad[c] + 1024));
  }
  unsigned rdyv;
  unsigned need = 2;
  unsigned prog = 1;
  unsigned sX = 192, sinc = 64;
  asm volatile("" : "+v"(prog));
  SB();
  for (int G = 0; G < 8; ++G) {
    #pragma unroll
    for (int i = 0; i < 8; ++i) {
      asm volatile("s_waitcnt vmcnt(8)" : "+v"(xr[i & 3][0]), "+v"(xr[i & 3][1]), "+v"(xr[i & 3][2]), "+v"(xr[i & 3][3]) :: "memory");
      if ((i & 3) == 1) wait_ready(rdy, need & 0xffff, actl);
      if ((i & 3) == 3) { wait_ready(rdy, need << 16, actl); need <<= 1; }
      if (i == 5) {
        unsigned long long sb = (((unsigned long long)wshi << 32) | wslo) + ((rdy & 1) << 21);
        srdX[0] = (unsigned)sb; srdX[1] = (unsigned)(sb >> 32); srdX[2] = STG;
        sX = (G + 1) * 262144; sinc = 32768;
      }
      SB();
      #pragma unroll
      for (int c = 0; c < 8; ++c) {
        const int pc = (c + 7) & 7;
        const int rslot = (c == 0) ? i : ((i + 1) & 7);
        if (c == 0) { wf[7][0] = __builtin_amdgcn_ds_read_tr8_b64_v2i32(LDSP(ad[7] + rslot * 8192)); SB(); }
        v4i a = {wf[c][0][0], wf[c][0][1], wf[c][1][0], wf[c][1][1]};
        acc[0][c] = __builtin_amdgcn_mfma_i32_16x16x64_i8(a, xr[i & 3][0], acc[0][c], 0, 0, 0);
        if (c == 0) { wf[7][1] = __builtin_amdgcn_ds_read_tr8_b64_v2i32(LDSP(ad[7] + rslot * 8192 + 1024)); if (i == 7) ad[7] ^= 0x10000; }
        else { if (i == 7) ad[pc] ^= 0x10000; wf[pc][0] = __builtin_amdgcn_ds_read_tr8_b64_v2i32(LDSP(ad[pc] + rslot * 8192)); }
        SB();
        acc[1][c] = __builtin_amdgcn_mfma_i32_16x16x64_i8(a, xr[i & 3][1], acc[1][c], 0, 0, 0);
        if (c != 0) wf[pc][1] = __builtin_amdgcn_ds_read_tr8_b64_v2i32(LDSP(ad[pc] + rslot * 8192 + 1024));
        SB();
        acc[2][c] = __builtin_amdgcn_mfma_i32_16x16x64_i8(a, xr[i & 3][2], acc[2][c], 0, 0, 0);
        if (i < 5) {
          if (c & 1) xload<0>(xr[(i + 3) & 3][c >> 1], voff[c >> 1], srdX, sX);
        } else {
          if (c == 1) xload<0>(xr[(i + 3) & 3][0], voffX, srdX, sX);
          if (c == 3) xload<1024>(xr[(i + 3) & 3][1], voffX, srdX, sX);
          if (c == 5) xload<2048>(xr[(i + 3) & 3][2], voffX, srdX, sX);
          if (c == 7) xload<3072>(xr[(i + 3) & 3][3], voffX, srdX, sX);
        }
        if (c == 7) sX += sinc;
        if (i == 6 && !(c & 1)) asm volatile("v_add_u32 %0, %1, %2" : "=v"(voff[c >> 1]) : "i"((c >> 1) * 1024), "v"(voffX));
        if ((i & 3) == 0) {
          if (c == 0) { *(volatile AS3 unsigned*)(size_t)aprog = prog; prog += 1; }
          if (c == 2) rdyv = *(volatile AS3 unsigned*)(size_t)actl;
          if (c == 6) rdy = rfl(rdyv);
        }
        SB();
        acc[3][c] = __builtin_amdgcn_mfma_i32_16x16x64_i8(a, xr[i & 3][3], acc[3][c], 0, 0, 0);
        SB();
      }
    }
  }
  SB();
  asm volatile("s_waitcnt vmcnt(0)" :: "v"(xr[0][0]), "v"(xr[0][1]), "v"(xr[0][2]), "v"(xr[0][3]), "v"(xr[1][0]), "v"(xr[1][1]), "v"(xr[1][2]), "v"(xr[1][3]),
               "v"(xr[2][0]), "v"(xr[2][1]), "v"(xr[2][2]), "v"(xr[2][3]), "v"(xr[3][0]), "v"(xr[3][1]), "v"(xr[3][2]), "v"(xr[3][3]) : "memory");
  SB();
  if (wid == 0) { STAMP(2); CSTAMP(5); }
  const float scale = *(volatile AS3 float*)(size_t)(lb + L_SCALE);
  #pragma unroll
  for (int m = 0; m < 4; ++m) {
    float* p = out + (size_t)(mh * 256 + (wid * 4 + m) * 16 + (lane & 15)) * NN + n0 + (lane >> 4) * 4;
    #pragma unroll
    for (int c = 0; c < 8; ++c) {
      f32x4 v = {acc[m][c][0] * scale, acc[m][c][1] * scale, acc[m][c][2] * scale, acc[m][c][3] * scale};
      asm volatile("global_store_dwordx4 %0, %1, off offset:%2 sc0 sc1\n\ts_nop 1" :: "v"(p), "v"(v), "i"(c * 64) : "memory");
    }
  }
}

extern "C" void kernel_launch(int rank, int nranks, ncclComm_t comm, hipStream_t stream,
                              void* const* d_in, const int* in_sizes, int n_in,
                              void* d_out, int out_size, const int* in_dtypes, int out_dtype,
                              void* const* d_out_all, void* const* d_ws_all, size_t ws_bytes) {
  if (nranks != NR || (unsigned)rank >= (unsigned)NR || n_in < 4 || ws_bytes < WS_NEED
      || in_sizes[0] != KK * KL || in_sizes[1] != KK * NN || out_size != ML * NN) {
    fprintf(stderr, "kernel_launch: unexpected shapes\n");
    return;
  }
  Args a;
  for (int p = 0; p < NR; ++p) a.ws[p] = (char*)d_ws_all[(rank + p) % NR];
  a2a_gemm<<<256, 448, 0, stream>>>((const int8_t*)d_in[0], (const int8_t*)d_in[1], (const float*)d_in[2], (const float*)d_in[3],
                                    (float*)d_out, a, (unsigned)rank);
  HIP_CHECK(hipGetLastError());
}
```

```cpp
#include <hip/hip_runtime.h>
#include <rccl/rccl.h>
#include <cstdio>
#include <cstdint>
#include <utility>

#define HIP_CHECK(expr) do { hipError_t _e = (expr); if (_e != hipSuccess) { \
    fprintf(stderr, "HIP error %s at %s:%d\n", hipGetErrorString(_e), __FILE__, __LINE__); \
    return; } } while (0)

typedef int v4i __attribute__((ext_vector_type(4)));
typedef unsigned u32x2 __attribute__((ext_vector_type(2)));
typedef unsigned u32x4 __attribute__((ext_vector_type(4)));
typedef float f32x4 __attribute__((ext_vector_type(4)));

constexpr int NR = 8;
constexpr int NN = 16384, KK = 4096, ML = 512, KL = 512;
constexpr unsigned STG = ML * KK;
constexpr unsigned OFF_FLAG = 2 * STG;
constexpr unsigned OFF_CNT = OFF_FLAG + 65536;
constexpr unsigned OFF_TMO = OFF_CNT + 131072;
constexpr unsigned OFF_DBG = OFF_TMO + 4096;
constexpr size_t WS_NEED = OFF_DBG + 262144;
constexpr unsigned long long DEADLINE_TICKS = 50000000ull;
#define SEND_DELAY 300

#define G1E 1
#define SWP 2
#define PFS 4
#define PFE 0
#define BD 3
#define PFD 1
#define CPB 0x7600
#define CPX 0
#define PACE 0
#define PCS 5
#define PFPOL "sc0 sc1"
#define REP 16
#define POLLN "24"
#define POLLS "1"
#define WPOL ""
constexpr unsigned L_RING = 0;
constexpr unsigned L_CTL = 131072;
constexpr unsigned L_RDY = L_CTL;
constexpr unsigned L_SCALE = L_CTL + 4;
constexpr unsigned L_PROG = L_CTL + 16;
constexpr unsigned L_EF = L_CTL + 32;
constexpr unsigned L_FR = L_CTL + 48;
constexpr unsigned L_OUT = L_CTL + 512;
constexpr unsigned L_SIZE = L_OUT + 16384;
#define EPOL "sc1"
#define ETHR 11
template<class F, int... I> __device__ __forceinline__ void sfor_(F&& fn, std::integer_sequence<int, I...>) { (fn(std::integral_constant<int, I>{}), ...); }
template<int N, class F> __device__ __forceinline__ void sfor(F&& fn) { sfor_(fn, std::make_integer_sequence<int, N>{}); }

#define AS3 __attribute__((address_space(3)))
#define SB() __builtin_amdgcn_sched_barrier(0)
__device__ __forceinline__ unsigned rfl(unsigned x) { return __builtin_amdgcn_readfirstlane(x); }
__device__ __forceinline__ u32x4 mk_srd(const void* p, unsigned bytes) {
  unsigned long long a = (unsigned long long)p;
  u32x4 r; r[0] = rfl((unsigned)a); r[1] = rfl((unsigned)(a >> 32) & 0xffff); r[2] = bytes; r[3] = 0x00020000; return r;
}
__device__ __forceinline__ void setm0(unsigned v) { asm volatile("s_mov_b32 m0, %0\n\ts_nop 0" :: "s"(v) : "memory"); }
template<int OFF> __device__ __forceinline__ void glds(unsigned voff, u32x4 srd, unsigned soff) {
  asm volatile("buffer_load_dwordx4 %0, %1, %2 offen offset:%3 " WPOL " lds" :: "v"(voff), "s"(srd), "s"(soff), "i"(OFF) : "memory");
}
template<int N> __device__ __forceinline__ void wait_vm() { asm volatile("s_waitcnt vmcnt(%0)" :: "i"(N) : "memory"); }
template<int OFF> __device__ __forceinline__ void ostore(f32x4 v, unsigned voff, u32x4 srd, unsigned soff) {
  asm volatile("buffer_store_dwordx4 %0, %1, %2, %3 offen offset:%4 " EPOL "\n\ts_nop 1" :: "v"(v), "v"(voff), "s"(srd), "s"(soff), "i"(OFF) : "memory");
}
__device__ __forceinline__ void pfetch(unsigned voff, u32x4 srd, unsigned soff) {
  asm volatile("s_nop 4\n\tbuffer_load_dword v255, %0, %1, %2 offen " PFPOL :: "v"(voff), "s"(srd), "s"(soff) : "memory", "v255");
}
#define STAMP(i)
#define ASTAMP(ins, off)

struct Args { char* ws[NR]; };

__device__ __forceinline__ unsigned step_of(unsigned u, unsigned rank) {
  if (u < 8) return rank * 8 + u;
  unsigned v = u - 8, sl = v / 7, i = v - sl * 7;
  return ((rank + 1 + i) & 7) * 8 + sl;
}

__global__ __launch_bounds__(512, 2) void a2a_gemm(const int8_t* __restrict__ X, const int8_t* __restrict__ W,
    const float* __restrict__ sxp, const float* __restrict__ swp, float* __restrict__ out, Args args, unsigned rank) {
  __shared__ __attribute__((aligned(1024))) char lds[L_SIZE];
  const unsigned b = blockIdx.x, xcd = b & 7, jj = b >> 3, nt = xcd * 16 + (jj >> 1), mh = jj & 1;
  const unsigned wv = rfl(threadIdx.x >> 6), lane = threadIdx.x & 63;
  const unsigned n0 = nt * 128;
  const unsigned lb = (unsigned)(size_t)(AS3 char*)lds;
  char* ws = args.ws[0];

  if (wv == 7) {
    if (lane < 16) *(volatile AS3 unsigned*)(size_t)(lb + L_CTL + lane * 4) = lane == 0 ? 2u : 0u;
    asm volatile("s_waitcnt lgkmcnt(0)" ::: "memory");
  }
  asm volatile("s_barrier" ::: "memory");
  if (wv < 3 || wv == 7) {
  if (wv == 0) {
    const unsigned qi = b >> 5, q = (rank + qi) & 7, sp = (b & 31) >> 3, c8 = b & 7;
    if (qi == 0) {
    } else {
    __builtin_amdgcn_s_setprio(3);
    unsigned long long t0 = __builtin_amdgcn_s_memrealtime();
    const int8_t* src = X + (size_t)(q * ML + c8 * 64 + (lane & 15)) * KL + sp * 128 + (lane >> 4) * 16;
    u32x4 f[8];
    #pragma unroll
    for (int i = 0; i < 8; ++i) f[i] = *(const u32x4*)(src + (i >> 1) * 16 * KL + (i & 1) * 64);
    unsigned call = __hip_atomic_fetch_add((unsigned*)(ws + OFF_CNT + (b * 2) * 256) + lane, 1u, __ATOMIC_RELAXED, __HIP_MEMORY_SCOPE_AGENT);
    char* dws = args.ws[qi & 7];
    asm volatile("s_waitcnt vmcnt(0)" : "+v"(f[0]), "+v"(f[1]), "+v"(f[2]), "+v"(f[3]), "+v"(f[4]), "+v"(f[5]), "+v"(f[6]), "+v"(f[7]), "+v"(call) :: "memory");
    call = rfl(call) + 1;
    const unsigned v = sp * 14 + ((7 - qi) & 7);
    char* dst = dws + (call & 1) * STG + (8 + v) * 32768 + c8 * 4096 + lane * 16;
    unsigned* fl = (unsigned*)(dws + OFF_FLAG + (c8 >> 2) * 16384 + (lane & (REP - 1)) * 1024) + v * 4 + (c8 & 3);
    STAMP(8);
    while (__builtin_amdgcn_s_memrealtime() - t0 < (unsigned long long)(sp * SEND_DELAY)) __builtin_amdgcn_s_sleep(1);
    #pragma unroll
    for (int e = 0; e < 2; ++e) {
      #pragma unroll
      for (int i = 0; i < 4; ++i)
        asm volatile("global_store_dwordx4 %0, %1, off offset:%2 sc0 sc1\n\ts_nop 1" :: "v"(dst + e * 7 * 32768), "v"(f[i * 2 + e]), "i"(i * 1024) : "memory");
      asm volatile("s_waitcnt vmcnt(0)" ::: "memory");
      if (lane < REP) __hip_atomic_store(fl + e * 28, call, __ATOMIC_RELAXED, __HIP_MEMORY_SCOPE_SYSTEM);
    }
    STAMP(9);
    __builtin_amdgcn_s_setprio(0);
    }
  }
  if (wv == 2) {
    unsigned call = __hip_atomic_fetch_add((unsigned*)(ws + OFF_CNT + (b * 2 + 1) * 256) + lane, 1u, __ATOMIC_RELAXED, __HIP_MEMORY_SCOPE_AGENT);
    pfetch((jj * 64 + lane) * 128, mk_srd(X, KK * KL), rank * (ML * KL));
    float scale = sxp[0] * swp[0];
    SB();
    asm volatile("s_waitcnt vmcnt(0)" : "+v"(call) :: "memory");
    call = rfl(call) + 1;
    const unsigned par = call & 1;
    const char* fb = ws + OFF_FLAG + mh * 16384 + ((jj >> 1) & (REP - 1)) * 1024;
    unsigned* tmo = (unsigned*)(ws + OFF_TMO);
    unsigned long long t0 = __builtin_amdgcn_s_memrealtime();
    unsigned dead = 0;
    for (unsigned g = 2; g < 16; ++g) {
      while (!dead) {
        unsigned it;
        asm volatile("s_mov_b32 %0, " POLLN "\n"
                     "1:\n\t"
                     "s_load_dwordx16 s[64:79], %1, 0x0 glc\n\t"
                     "s_waitcnt lgkmcnt(0)\n\t"
                     "s_min_u32 s64, s64, s72\n\ts_min_u32 s65, s65, s73\n\ts_min_u32 s66, s66, s74\n\ts_min_u32 s67, s67, s75\n\t"
                     "s_min_u32 s68, s68, s76\n\ts_min_u32 s69, s69, s77\n\ts_min_u32 s70, s70, s78\n\ts_min_u32 s71, s71, s79\n\t"
                     "s_min_u32 s64, s64, s65\n\ts_min_u32 s66, s66, s67\n\ts_min_u32 s68, s68, s69\n\ts_min_u32 s70, s70, s71\n\t"
                     "s_min_u32 s64, s64, s66\n\ts_min_u32 s68, s68, s70\n\ts_min_u32 s64, s64, s68\n\t"
                     "s_cmp_ge_u32 s64, %2\n\t"
                     "s_cbranch_scc1 2f\n\t"
                     "s_sleep " POLLS "\n\t"
                     "s_sub_u32 %0, %0, 1\n\t"
                     "s_cmp_lg_u32 %0, 0\n\t"
                     "s_cbranch_scc1 1b\n"
                     "2:"
                     : "=&s"(it) : "s"(fb), "s"(call) : "s64", "s65", "s66", "s67", "s68", "s69", "s70", "s71", "s72", "s73", "s74", "s75", "s76", "s77", "s78", "s79", "scc", "memory");
        if (it) break;
        unsigned fv = __hip_atomic_load((const unsigned*)fb + (lane & 15), __ATOMIC_RELAXED, __HIP_MEMORY_SCOPE_SYSTEM);
        unsigned tv = __hip_atomic_load(tmo, __ATOMIC_RELAXED, __HIP_MEMORY_SCOPE_AGENT);
        tv = rfl(tv);
        if (__builtin_amdgcn_ballot_w64(fv < call) == 0) break;
        if (tv == call || __builtin_amdgcn_s_memrealtime() - t0 > DEADLINE_TICKS) {
          __hip_atomic_store(tmo, call, __ATOMIC_RELAXED, __HIP_MEMORY_SCOPE_AGENT);
          dead = 1;
        }
      }
      fb += 64;
      if (g < 12) *(volatile AS3 unsigned short*)(size_t)(lb + L_RDY) = (unsigned short)(((2u << g) - 2) | par);
    }
    STAMP(14);
    asm volatile("s_waitcnt vmcnt(0)" ::: "memory");
    if (__hip_atomic_load(tmo, __ATOMIC_RELAXED, __HIP_MEMORY_SCOPE_AGENT) == call) scale = __builtin_nanf("");
    *(volatile AS3 float*)(size_t)(lb + L_SCALE) = scale;
    asm volatile("s_waitcnt lgkmcnt(0)" ::: "memory");
    *(volatile AS3 unsigned short*)(size_t)(lb + L_RDY) = (unsigned short)(0xfffe | par);
  }
  if (wv == 1) {
    u32x4 srdW = mk_srd(W, (unsigned)KK * NN);
    unsigned vo[8];
    const unsigned r3 = lane >> 3;
    #pragma unroll
    for (int i = 0; i < 8; ++i)
      vo[i] = (i * 8 + r3) * NN + n0 + (((lane & 7) ^ (((r3 >> 1) & 3) | ((i & 2) << 1))) << 4) - (i & 3) * 1024;
    const unsigned tl = step_of(lane, rank) << 20;
    unsigned badm = 0;
    for (unsigned g = 0; g < 16; ++g) {
      if (g >= 2) {
        while (true) {
          u32x4 p = *(volatile AS3 u32x4*)(size_t)(lb + L_PROG);
          unsigned mn = min(min(p[0], p[1]), min(p[2], p[3]));
          if (rfl(mn) >= (g < 4 ? g - 1 : g - 2)) break;
          __builtin_amdgcn_s_sleep(2);
        }
      }
      #pragma unroll
      for (int j = 0; j < 4; ++j) {
        const unsigned jx = j ^ (mh * SWP);
        unsigned so = __builtin_amdgcn_readlane(tl, g * 4 + jx);
        asm volatile("s_nop 4" : "+s"(so));
        const unsigned m = lb + L_RING + ((g & 3) * 4 + jx) * 8192;
        setm0(m);
#define PC1 if (PACE == 1 && g >= PCS) __builtin_amdgcn_s_sleep(1);
#define PC2 if (PACE <= 2 && PACE && g >= PCS) __builtin_amdgcn_s_sleep(1);
#define PC4 if (PACE && g >= PCS) __builtin_amdgcn_s_sleep(PACE <= 2 ? 1 : PACE);
        glds<0>(vo[0], srdW, so); PC1 glds<1024>(vo[1], srdW, so); PC2 glds<2048>(vo[2], srdW, so); PC1 glds<3072>(vo[3], srdW, so); PC4
        setm0(m + 4096);
        glds<0>(vo[4], srdW, so); PC1 glds<1024>(vo[5], srdW, so); PC2 glds<2048>(vo[6], srdW, so); PC1 glds<3072>(vo[7], srdW, so); PC4
        if (j == G1E - 1 && g == 1) {
          wait_vm<G1E * 8>();
          *(volatile AS3 unsigned short*)(size_t)(lb + L_RDY + 2) = (unsigned short)1;
        }
        if (j == 2 && g >= 4) {
          asm volatile("s_waitcnt vmcnt(24)" ::: "memory");
          *(volatile AS3 unsigned short*)(size_t)(lb + L_RDY + 2) = (unsigned short)((1u << g) - 1);
        }
      }
      if (g == 1) {
        asm volatile("s_waitcnt vmcnt(0)" ::: "memory");
        *(volatile AS3 unsigned short*)(size_t)(lb + L_RDY + 2) = (unsigned short)3;
        #pragma unroll
        for (int e = 0; e < PFE; ++e) pfetch(lane * NN + n0, srdW, __builtin_amdgcn_readlane(tl, 8 + (e >> 1) * 4 + (e & 1) + mh * 2));
      }
      if (g >= PFS && g + PFD < 16) {
        pfetch(lane * NN + n0, srdW, __builtin_amdgcn_readlane(tl, (g + PFD) * 4 + mh * 2));
        pfetch(lane * NN + n0, srdW, __builtin_amdgcn_readlane(tl, (g + PFD) * 4 + 1 + mh * 2));
      }
      if (g == 2) {
        wait_vm<(PFS <= 2 ? 2 : 0)>();
        *(volatile AS3 unsigned short*)(size_t)(lb + L_RDY + 2) = (unsigned short)7;
      }
      if (g == 0) {
        const unsigned wb = (srdW[0] >> 7) + xcd * 16;
        badm = (unsigned)__builtin_amdgcn_ballot_w64(((wb + lane) & 7) == 3 || ((wb + lane) & 127) == 63) & 0xffff;
        if ((badm >> (jj >> 1)) & 1) badm = 0;
      }
      const int gb = 2 + (int)(jj >> 2) - BD;
      if ((int)g == (gb < 1 ? 1 : gb) || ((int)g == gb + 8 && jj < 24)) {
        const unsigned so = __builtin_amdgcn_readlane(tl, ((int)g == gb + 8 ? 40 : 8) + jj) + xcd * 2048;
        for (unsigned mk = badm; mk; mk &= mk - 1) pfetch(lane * NN, srdW, so + __builtin_ctz(mk) * 128);
      }
    }
    asm volatile("s_waitcnt vmcnt(0)" ::: "memory");
    *(volatile AS3 unsigned short*)(size_t)(lb + L_RDY + 2) = (unsigned short)0xffff;
  }
  {
    const unsigned wid = wv & 3;
    const unsigned rd = lb + L_OUT + wid * 4096 + (lane >> 3) * 128 + (((lane & 7) ^ (lane >> 3)) << 4);
    const unsigned vo = (mh * 256 + wid * 64 + (lane >> 3)) * (NN * 4) + n0 * 4 + (lane & 7) * 16;
    const u32x4 srdO = mk_srd(out, ML * NN * 4);
    while (*(volatile AS3 unsigned*)(size_t)(lb + L_PROG + wid * 4) < 13) __builtin_amdgcn_s_sleep(8);
    v4i d[32];
    float scale = 0.f;
    auto R = [&](auto K) __attribute__((always_inline)) {
      constexpr int k = K;
      while (*(volatile AS3 unsigned*)(size_t)(lb + L_EF + wid * 4) <= k) __builtin_amdgcn_s_sleep(1);
      if (k == 0) { scale = *(volatile AS3 float*)(size_t)(lb + L_SCALE); if (wid == 0) STAMP(6); }
      #pragma unroll
      for (int i = 0; i < 4; ++i) d[k * 4 + i] = *(volatile AS3 v4i*)(size_t)(rd + i * 1024);
      if (k < 7) *(volatile AS3 unsigned*)(size_t)(lb + L_FR + wid * 4) = (2u << k) - 1;
    };
    auto S = [&](auto J, auto THR) __attribute__((always_inline)) {
      constexpr int j = J, k = j >> 2;
      f32x4 v = {d[j][0] * scale, d[j][1] * scale, d[j][2] * scale, d[j][3] * scale};
      if constexpr (THR >= 0) wait_vm<THR>();
      ostore<(k & 3) * 128>(v, vo, srdO, ((k >> 2) * 32 + (j & 3) * 8) * (NN * 4));
    };
#define IC(x) std::integral_constant<int, (x)>{}
    R(IC(0)); S(IC(0), IC(-1)); S(IC(1), IC(-1)); S(IC(2), IC(-1)); S(IC(3), IC(-1));
    R(IC(1)); S(IC(4), IC(-1)); S(IC(5), IC(-1));
    R(IC(2)); S(IC(6), IC(-1)); S(IC(7), IC(-1));
    R(IC(3));
    sfor<8>([&](auto I) __attribute__((always_inline)) { S(IC(8 + I), IC(ETHR)); });
    if (wid == 0) STAMP(7);
    sfor<4>([&](auto K) __attribute__((always_inline)) {
      R(IC(4 + K));
      sfor<4>([&](auto I) __attribute__((always_inline)) { S(IC(16 + 4 * K + I), IC(K < 2 ? ETHR : -1)); });
    });
    return;
  }
  }

  const unsigned wid = wv - 3;
  const u32x4 srdX = mk_srd(X + (size_t)(rank * ML + mh * 256 + wid * 64) * KL, 64 * KL);
  const unsigned xa = lb + L_RING + 65536 + wid * 8192;
  const unsigned dvo = (lane >> 3) * KL + (((lane & 7) ^ (lane >> 3)) << 4);
  const unsigned xa0 = xa + (lane & 15) * 128 + (((lane >> 4) ^ (lane & 7)) << 4);
  const unsigned voffX = (mh * 16 + wid * 4) * 1024 + lane * 16;
  const u32x2 wsb = {rfl((unsigned)(size_t)ws), rfl((unsigned)((size_t)ws >> 32) & 0xffff)};
  const unsigned g4 = lane >> 4, q8 = (lane & 15) >> 1;
  const unsigned a0 = lb + L_RING + (g4 * 16 + q8) * 128 + ((((q8 >> 1) & 3) | ((g4 & 1) << 2)) << 4) + (lane & 1) * 8;
  const unsigned aw0 = lb + L_OUT + wid * 4096 + (lane & 15) * 128 + (((lane >> 4) ^ (lane & 7)) << 4);
  const char* dbg = ws + OFF_DBG + (wid ? 131072 : 0) + b * 128;
  asm volatile(
    "s_nop 4\n" "s_add_u32 m0, s46, 0\n" "s_movk_i32 s55, 0\n" "buffer_load_dwordx4 v248, s[40:43], s55 offen offset:0 lds\n" "s_movk_i32 s55, 3072\n"
    "buffer_load_dwordx4 v248, s[40:43], s55 offen offset:1024 lds\n" "s_movk_i32 s55, 6144\n" "buffer_load_dwordx4 v248, s[40:43], s55 offen offset:2048 lds\n" "s_movk_i32 s55, 9216\n"
    "buffer_load_dwordx4 v248, s[40:43], s55 offen offset:3072 lds\n" "s_add_u32 m0, s46, 4096\n" "s_movk_i32 s55, 16384\n" "buffer_load_dwordx4 v248, s[40:43], s55 offen offset:0 lds\n"
    "s_movk_i32 s55, 19456\n" "buffer_load_dwordx4 v248, s[40:43], s55 offen offset:1024 lds\n" "s_movk_i32 s55, 22528\n" "buffer_load_dwordx4 v248, s[40:43], s55 offen offset:2048 lds\n"
    "s_movk_i32 s55, 25600\n" "buffer_load_dwordx4 v248, s[40:43], s55 offen offset:3072 lds\n" "s_add_u32 m0, s46, 32768\n" "s_movk_i32 s55, 128\n"
    "buffer_load_dwordx4 v248, s[40:43], s55 offen offset:0 lds\n" "s_movk_i32 s55, 3200\n" "buffer_load_dwordx4 v248, s[40:43], s55 offen offset:1024 lds\n" "s_movk_i32 s55, 6272\n"
    "buffer_load_dwordx4 v248, s[40:43], s55 offen offset:2048 lds\n" "s_movk_i32 s55, 9344\n" "buffer_load_dwordx4 v248, s[40:43], s55 offen offset:3072 lds\n" "s_add_u32 m0, s46, 36864\n"
    "s_movk_i32 s55, 16512\n" "buffer_load_dwordx4 v248, s[40:43], s55 offen offset:0 lds\n" "s_movk_i32 s55, 19584\n" "buffer_load_dwordx4 v248, s[40:43], s55 offen offset:1024 lds\n"
    "s_movk_i32 s55, 22656\n" "buffer_load_dwordx4 v248, s[40:43], s55 offen offset:2048 lds\n" "s_movk_i32 s55, 25728\n" "buffer_load_dwordx4 v248, s[40:43], s55 offen offset:3072 lds\n"
    "v_xor_b32 v225, 16, v224\n" "v_xor_b32 v226, 32, v224\n" "v_xor_b32 v227, 48, v224\n" "v_xor_b32 v228, 64, v224\n" "v_xor_b32 v229, 80, v224\n" "v_xor_b32 v230, 96, v224\n"
    "v_xor_b32 v231, 112, v224\n" "v_xor_b32 v250, 64, v249\n" "v_xor_b32 v246, 64, v245\n" "v_mov_b32 v243, 1\n" "s_mov_b32 s51, 0\n" "s_mov_b32 s52, 8\n" "s_mov_b32 s53, 4\n" "s_mov_b32 s54, 0\n"
    "s_mov_b32 s56, 0\n" "s_mov_b32 s50, 0x40000\n" ASTAMP(s_memrealtime, 0) "s_waitcnt vmcnt(8)\n" "ds_read_b128 v[128:131], v249 offset:0\n" "ds_read_b128 v[132:135], v249 offset:2048\n"
    "ds_read_b128 v[136:139], v249 offset:4096\n" "ds_read_b128 v[140:143], v249 offset:6144\n" "ds_read_b128 v[144:147], v250 offset:0\n" "ds_read_b128 v[148:151], v250 offset:2048\n"
    "ds_read_b128 v[152:155], v250 offset:4096\n" "ds_read_b128 v[156:159], v250 offset:6144\n" ASTAMP(s_memrealtime, 96) "s_andn2_b32 s55, 0x10000, s51\n" "s_cbranch_scc0 L2_%=\n" "L1_%=:\n"
    "ds_read_b32 v251, v241 offset:0\n" "s_waitcnt lgkmcnt(0)\n" "v_readfirstlane_b32 s51, v251\n" "s_nop 3\n" "s_andn2_b32 s55, 0x10000, s51\n" "s_cbranch_scc0 L2_%=\n" "s_sleep 1\n"
    "s_branch L1_%=\n" "L2_%=:\n" ASTAMP(s_memrealtime, 8) ASTAMP(s_memtime, 32) "ds_read_b64_tr_b8 v[192:193], v224 offset:0\n" "ds_read_b64_tr_b8 v[194:195], v224 offset:1024\n"
    "ds_read_b64_tr_b8 v[196:197], v225 offset:0\n" "ds_read_b64_tr_b8 v[198:199], v225 offset:1024\n" "ds_read_b64_tr_b8 v[200:201], v226 offset:0\n" "ds_read_b64_tr_b8 v[202:203], v226 offset:1024\n"
    "ds_read_b64_tr_b8 v[204:205], v227 offset:0\n" "ds_read_b64_tr_b8 v[206:207], v227 offset:1024\n" "ds_read_b64_tr_b8 v[208:209], v228 offset:0\n" "ds_read_b64_tr_b8 v[210:211], v228 offset:1024\n"
    "ds_read_b64_tr_b8 v[212:213], v229 offset:0\n" "ds_read_b64_tr_b8 v[214:215], v229 offset:1024\n" "ds_read_b64_tr_b8 v[216:217], v230 offset:0\n" "ds_read_b64_tr_b8 v[218:219], v230 offset:1024\n"
    "s_waitcnt lgkmcnt(12)\n" "v_mfma_i32_16x16x64_i8 v[0:3], v[192:195], v[128:131], 0\n" "ds_read_b64_tr_b8 v[220:221], v231 offset:0\n" "v_mfma_i32_16x16x64_i8 v[32:35], v[192:195], v[132:135], 0\n"
    "ds_read_b64_tr_b8 v[222:223], v231 offset:1024\n" "v_mfma_i32_16x16x64_i8 v[64:67], v[192:195], v[136:139], 0\n" "s_add_u32 m0, s46, 0\n" "s_movk_i32 s55, 256\n"
    "buffer_load_dwordx4 v248, s[40:43], s55 offen offset:0 lds\n" "v_mfma_i32_16x16x64_i8 v[96:99], v[192:195], v[140:143], 0\n" "s_waitcnt lgkmcnt(12)\n"
    "v_mfma_i32_16x16x64_i8 v[100:103], v[196:199], v[140:143], 0\n" "ds_read_b64_tr_b8 v[192:193], v224 offset:8192\n" "v_mfma_i32_16x16x64_i8 v[68:71], v[196:199], v[136:139], 0\n"
    "ds_read_b64_tr_b8 v[194:195], v224 offset:9216\n" "v_mfma_i32_16x16x64_i8 v[36:39], v[196:199], v[132:135], 0\n" "s_movk_i32 s55, 3328\n"
    "buffer_load_dwordx4 v248, s[40:43], s55 offen offset:1024 lds\n" "v_mfma_i32_16x16x64_i8 v[4:7], v[196:199], v[128:131], 0\n" "s_waitcnt lgkmcnt(12)\n"
    "v_mfma_i32_16x16x64_i8 v[8:11], v[200:203], v[128:131], 0\n" "ds_read_b64_tr_b8 v[196:197], v225 offset:8192\n" "v_mfma_i32_16x16x64_i8 v[40:43], v[200:203], v[132:135], 0\n"
    "ds_read_b64_tr_b8 v[198:199], v225 offset:9216\n" "v_mfma_i32_16x16x64_i8 v[72:75], v[200:203], v[136:139], 0\n" "s_movk_i32 s55, 6400\n"
    "buffer_load_dwordx4 v248, s[40:43], s55 offen offset:2048 lds\n" "v_mfma_i32_16x16x64_i8 v[104:107], v[200:203], v[140:143], 0\n" "s_waitcnt lgkmcnt(12)\n"
    "v_mfma_i32_16x16x64_i8 v[108:111], v[204:207], v[140:143], 0\n" "ds_read_b64_tr_b8 v[200:201], v226 offset:8192\n" "v_mfma_i32_16x16x64_i8 v[76:79], v[204:207], v[136:139], 0\n"
    "ds_read_b64_tr_b8 v[202:203], v226 offset:9216\n" "v_mfma_i32_16x16x64_i8 v[44:47], v[204:207], v[132:135], 0\n" "ds_read_b32 v244, v241\n" "s_movk_i32 s55, 9472\n"
    "buffer_load_dwordx4 v248, s[40:43], s55 offen offset:3072 lds\n" "v_mfma_i32_16x16x64_i8 v[12:15], v[204:207], v[128:131], 0\n" "s_waitcnt lgkmcnt(13)\n"
    "v_mfma_i32_16x16x64_i8 v[16:19], v[208:211], v[128:131], 0\n" "ds_read_b64_tr_b8 v[204:205], v227 offset:8192\n" "v_mfma_i32_16x16x64_i8 v[48:51], v[208:211], v[132:135], 0\n"
    "ds_read_b64_tr_b8 v[206:207], v227 offset:9216\n" "v_mfma_i32_16x16x64_i8 v[80:83], v[208:211], v[136:139], 0\n" "s_add_u32 m0, s46, 4096\n" "s_movk_i32 s55, 16640\n"
    "buffer_load_dwordx4 v248, s[40:43], s55 offen offset:0 lds\n" "s_waitcnt vmcnt(5)\n" "ds_read_b128 v[160:163], v249 offset:32768\n" "v_mfma_i32_16x16x64_i8 v[112:115], v[208:211], v[140:143], 0\n"
    "s_waitcnt lgkmcnt(14)\n" "v_mfma_i32_16x16x64_i8 v[116:119], v[212:215], v[140:143], 0\n" "ds_read_b64_tr_b8 v[208:209], v228 offset:8192\n"
    "v_mfma_i32_16x16x64_i8 v[84:87], v[212:215], v[136:139], 0\n" "ds_read_b64_tr_b8 v[210:211], v228 offset:9216\n" "v_mfma_i32_16x16x64_i8 v[52:55], v[212:215], v[132:135], 0\n"
    "s_movk_i32 s55, 19712\n" "buffer_load_dwordx4 v248, s[40:43], s55 offen offset:1024 lds\n" "ds_read_b128 v[164:167], v249 offset:34816\n"
    "v_mfma_i32_16x16x64_i8 v[20:23], v[212:215], v[128:131], 0\n" "s_waitcnt lgkmcnt(15)\n" "v_mfma_i32_16x16x64_i8 v[24:27], v[216:219], v[128:131], 0\n"
    "ds_read_b64_tr_b8 v[212:213], v229 offset:8192\n" "v_mfma_i32_16x16x64_i8 v[56:59], v[216:219], v[132:135], 0\n" "ds_read_b64_tr_b8 v[214:215], v229 offset:9216\n"
    "v_mfma_i32_16x16x64_i8 v[88:91], v[216:219], v[136:139], 0\n" "s_movk_i32 s55, 22784\n" "buffer_load_dwordx4 v248, s[40:43], s55 offen offset:2048 lds\n"
    "ds_read_b128 v[168:171], v249 offset:36864\n" "v_mfma_i32_16x16x64_i8 v[120:123], v[216:219], v[140:143], 0\n" "s_waitcnt lgkmcnt(15)\n"
    "v_mfma_i32_16x16x64_i8 v[124:127], v[220:223], v[140:143], 0\n" "ds_read_b64_tr_b8 v[216:217], v230 offset:8192\n" "v_mfma_i32_16x16x64_i8 v[92:95], v[220:223], v[136:139], 0\n"
    "ds_read_b64_tr_b8 v[218:219], v230 offset:9216\n" "v_mfma_i32_16x16x64_i8 v[60:63], v[220:223], v[132:135], 0\n" "s_waitcnt lgkmcnt(11)\n" "v_readfirstlane_b32 s51, v244\n"
    "s_movk_i32 s55, 25856\n" "buffer_load_dwordx4 v248, s[40:43], s55 offen offset:3072 lds\n" "ds_read_b128 v[172:175], v249 offset:38912\n"
    "v_mfma_i32_16x16x64_i8 v[28:31], v[220:223], v[128:131], 0\n" "v_mfma_i32_16x16x64_i8 v[0:3], v[192:195], v[144:147], v[0:3]\n" "ds_read_b64_tr_b8 v[220:221], v231 offset:8192\n"
    "v_mfma_i32_16x16x64_i8 v[32:35], v[192:195], v[148:151], v[32:35]\n" "ds_read_b64_tr_b8 v[222:223], v231 offset:9216\n" "v_mfma_i32_16x16x64_i8 v[64:67], v[192:195], v[152:155], v[64:67]\n"
    "ds_read_b128 v[176:179], v250 offset:32768\n" "v_mfma_i32_16x16x64_i8 v[96:99], v[192:195], v[156:159], v[96:99]\n" "v_mfma_i32_16x16x64_i8 v[100:103], v[196:199], v[156:159], v[100:103]\n"
    "ds_read_b64_tr_b8 v[192:193], v224 offset:16384\n" "v_mfma_i32_16x16x64_i8 v[68:71], v[196:199], v[152:155], v[68:71]\n" "ds_read_b64_tr_b8 v[194:195], v224 offset:17408\n"
    "v_mfma_i32_16x16x64_i8 v[36:39], v[196:199], v[148:151], v[36:39]\n" "ds_read_b128 v[180:183], v250 offset:34816\n" "v_mfma_i32_16x16x64_i8 v[4:7], v[196:199], v[144:147], v[4:7]\n"
    "v_mfma_i32_16x16x64_i8 v[8:11], v[200:203], v[144:147], v[8:11]\n" "ds_read_b64_tr_b8 v[196:197], v225 offset:16384\n" "v_mfma_i32_16x16x64_i8 v[40:43], v[200:203], v[148:151], v[40:43]\n"
    "ds_read_b64_tr_b8 v[198:199], v225 offset:17408\n" "v_mfma_i32_16x16x64_i8 v[72:75], v[200:203], v[152:155], v[72:75]\n" "ds_read_b128 v[184:187], v250 offset:36864\n"
    "v_mfma_i32_16x16x64_i8 v[104:107], v[200:203], v[156:159], v[104:107]\n" "s_waitcnt lgkmcnt(15)\n" "v_mfma_i32_16x16x64_i8 v[108:111], v[204:207], v[156:159], v[108:111]\n"
    "ds_read_b64_tr_b8 v[200:201], v226 offset:16384\n" "v_mfma_i32_16x16x64_i8 v[76:79], v[204:207], v[152:155], v[76:79]\n" "ds_read_b64_tr_b8 v[202:203], v226 offset:17408\n"
    "v_mfma_i32_16x16x64_i8 v[44:47], v[204:207], v[148:151], v[44:47]\n" "ds_read_b128 v[188:191], v250 offset:38912\n" "v_mfma_i32_16x16x64_i8 v[12:15], v[204:207], v[144:147], v[12:15]\n"
    "v_mfma_i32_16x16x64_i8 v[16:19], v[208:211], v[144:147], v[16:19]\n" "ds_read_b64_tr_b8 v[204:205], v227 offset:16384\n" "v_mfma_i32_16x16x64_i8 v[48:51], v[208:211], v[148:151], v[48:51]\n"
    "ds_read_b64_tr_b8 v[206:207], v227 offset:17408\n" "v_mfma_i32_16x16x64_i8 v[80:83], v[208:211], v[152:155], v[80:83]\n" "s_waitcnt lgkmcnt(2)\n" "s_add_u32 m0, s46, 32768\n"
    "s_movk_i32 s55, 384\n" "buffer_load_dwordx4 v248, s[40:43], s55 offen offset:0 lds\n" "v_mfma_i32_16x16x64_i8 v[112:115], v[208:211], v[156:159], v[112:115]\n"
    "v_mfma_i32_16x16x64_i8 v[116:119], v[212:215], v[156:159], v[116:119]\n" "ds_read_b64_tr_b8 v[208:209], v228 offset:16384\n" "v_mfma_i32_16x16x64_i8 v[84:87], v[212:215], v[152:155], v[84:87]\n"
    "ds_read_b64_tr_b8 v[210:211], v228 offset:17408\n" "v_mfma_i32_16x16x64_i8 v[52:55], v[212:215], v[148:151], v[52:55]\n" "s_movk_i32 s55, 3456\n"
    "buffer_load_dwordx4 v248, s[40:43], s55 offen offset:1024 lds\n" "v_mfma_i32_16x16x64_i8 v[20:23], v[212:215], v[144:147], v[20:23]\n"
    "v_mfma_i32_16x16x64_i8 v[24:27], v[216:219], v[144:147], v[24:27]\n" "ds_read_b64_tr_b8 v[212:213], v229 offset:16384\n" "v_mfma_i32_16x16x64_i8 v[56:59], v[216:219], v[148:151], v[56:59]\n"
    "ds_read_b64_tr_b8 v[214:215], v229 offset:17408\n" "v_mfma_i32_16x16x64_i8 v[88:91], v[216:219], v[152:155], v[88:91]\n" "s_movk_i32 s55, 6528\n"
    "buffer_load_dwordx4 v248, s[40:43], s55 offen offset:2048 lds\n" "v_mfma_i32_16x16x64_i8 v[120:123], v[216:219], v[156:159], v[120:123]\n"
    "v_mfma_i32_16x16x64_i8 v[124:127], v[220:223], v[156:159], v[124:127]\n" "ds_read_b64_tr_b8 v[216:217], v230 offset:16384\n" "v_mfma_i32_16x16x64_i8 v[92:95], v[220:223], v[152:155], v[92:95]\n"
    "ds_read_b64_tr_b8 v[218:219], v230 offset:17408\n" "v_mfma_i32_16x16x64_i8 v[60:63], v[220:223], v[148:151], v[60:63]\n" "s_movk_i32 s55, 9600\n"
    "buffer_load_dwordx4 v248, s[40:43], s55 offen offset:3072 lds\n" "v_mfma_i32_16x16x64_i8 v[28:31], v[220:223], v[144:147], v[28:31]\n"
    "v_mfma_i32_16x16x64_i8 v[0:3], v[192:195], v[160:163], v[0:3]\n" "ds_read_b64_tr_b8 v[220:221], v231 offset:16384\n" "v_mfma_i32_16x16x64_i8 v[32:35], v[192:195], v[164:167], v[32:35]\n"
    "ds_read_b64_tr_b8 v[222:223], v231 offset:17408\n" "v_mfma_i32_16x16x64_i8 v[64:67], v[192:195], v[168:171], v[64:67]\n" "s_add_u32 m0, s46, 36864\n" "s_movk_i32 s55, 16768\n"
    "buffer_load_dwordx4 v248, s[40:43], s55 offen offset:0 lds\n" "s_waitcnt vmcnt(5)\n" "ds_read_b128 v[128:131], v249 offset:0\n"
    "v_mfma_i32_16x16x64_i8 v[96:99], v[192:195], v[172:175], v[96:99]\n" "v_mfma_i32_16x16x64_i8 v[100:103], v[196:199], v[172:175], v[100:103]\n" "ds_read_b64_tr_b8 v[192:193], v224 offset:24576\n"
    "v_mfma_i32_16x16x64_i8 v[68:71], v[196:199], v[168:171], v[68:71]\n" "ds_read_b64_tr_b8 v[194:195], v224 offset:25600\n" "v_mfma_i32_16x16x64_i8 v[36:39], v[196:199], v[164:167], v[36:39]\n"
    "s_movk_i32 s55, 19840\n" "buffer_load_dwordx4 v248, s[40:43], s55 offen offset:1024 lds\n" "ds_read_b128 v[132:135], v249 offset:2048\n"
    "v_mfma_i32_16x16x64_i8 v[4:7], v[196:199], v[160:163], v[4:7]\n" "v_mfma_i32_16x16x64_i8 v[8:11], v[200:203], v[160:163], v[8:11]\n" "ds_read_b64_tr_b8 v[196:197], v225 offset:24576\n"
    "v_mfma_i32_16x16x64_i8 v[40:43], v[200:203], v[164:167], v[40:43]\n" "ds_read_b64_tr_b8 v[198:199], v225 offset:25600\n" "v_mfma_i32_16x16x64_i8 v[72:75], v[200:203], v[168:171], v[72:75]\n"
    "s_movk_i32 s55, 22912\n" "buffer_load_dwordx4 v248, s[40:43], s55 offen offset:2048 lds\n" "ds_read_b128 v[136:139], v249 offset:4096\n"
    "v_mfma_i32_16x16x64_i8 v[104:107], v[200:203], v[172:175], v[104:107]\n" "s_waitcnt lgkmcnt(15)\n" "v_mfma_i32_16x16x64_i8 v[108:111], v[204:207], v[172:175], v[108:111]\n"
    "ds_read_b64_tr_b8 v[200:201], v226 offset:24576\n" "v_mfma_i32_16x16x64_i8 v[76:79], v[204:207], v[168:171], v[76:79]\n" "ds_read_b64_tr_b8 v[202:203], v226 offset:25600\n"
    "v_mfma_i32_16x16x64_i8 v[44:47], v[204:207], v[164:167], v[44:47]\n" "ds_read_b32 v244, v241\n" "s_movk_i32 s55, 25984\n" "buffer_load_dwordx4 v248, s[40:43], s55 offen offset:3072 lds\n"
    "ds_read_b128 v[140:143], v249 offset:6144\n" "v_mfma_i32_16x16x64_i8 v[12:15], v[204:207], v[160:163], v[12:15]\n" "s_waitcnt lgkmcnt(15)\n"
    "v_mfma_i32_16x16x64_i8 v[16:19], v[208:211], v[160:163], v[16:19]\n" "ds_read_b64_tr_b8 v[204:205], v227 offset:24576\n" "v_mfma_i32_16x16x64_i8 v[48:51], v[208:211], v[164:167], v[48:51]\n"
    "ds_read_b64_tr_b8 v[206:207], v227 offset:25600\n" "v_mfma_i32_16x16x64_i8 v[80:83], v[208:211], v[168:171], v[80:83]\n" "ds_read_b128 v[144:147], v250 offset:0\n"
    "v_mfma_i32_16x16x64_i8 v[112:115], v[208:211], v[172:175], v[112:115]\n" "v_mfma_i32_16x16x64_i8 v[116:119], v[212:215], v[172:175], v[116:119]\n"
    "ds_read_b64_tr_b8 v[208:209], v228 offset:24576\n" "v_mfma_i32_16x16x64_i8 v[84:87], v[212:215], v[168:171], v[84:87]\n" "ds_read_b64_tr_b8 v[210:211], v228 offset:25600\n"
    "v_mfma_i32_16x16x64_i8 v[52:55], v[212:215], v[164:167], v[52:55]\n" "ds_read_b128 v[148:151], v250 offset:2048\n" "v_mfma_i32_16x16x64_i8 v[20:23], v[212:215], v[160:163], v[20:23]\n"
    "s_waitcnt lgkmcnt(15)\n" "v_mfma_i32_16x16x64_i8 v[24:27], v[216:219], v[160:163], v[24:27]\n" "ds_read_b64_tr_b8 v[212:213], v229 offset:24576\n"
    "v_mfma_i32_16x16x64_i8 v[56:59], v[216:219], v[164:167], v[56:59]\n" "ds_read_b64_tr_b8 v[214:215], v229 offset:25600\n" "v_mfma_i32_16x16x64_i8 v[88:91], v[216:219], v[168:171], v[88:91]\n"
    "ds_read_b128 v[152:155], v250 offset:4096\n" "v_mfma_i32_16x16x64_i8 v[120:123], v[216:219], v[172:175], v[120:123]\n" "v_mfma_i32_16x16x64_i8 v[124:127], v[220:223], v[172:175], v[124:127]\n"
    "ds_read_b64_tr_b8 v[216:217], v230 offset:24576\n" "v_mfma_i32_16x16x64_i8 v[92:95], v[220:223], v[168:171], v[92:95]\n" "ds_read_b64_tr_b8 v[218:219], v230 offset:25600\n"
    "v_mfma_i32_16x16x64_i8 v[60:63], v[220:223], v[164:167], v[60:63]\n" "s_waitcnt lgkmcnt(12)\n" "v_readfirstlane_b32 s51, v244\n" "ds_read_b128 v[156:159], v250 offset:6144\n"
    "ds_write_b32 v242, v243\n" "v_add_u32 v243, 1, v243\n" "v_mfma_i32_16x16x64_i8 v[28:31], v[220:223], v[160:163], v[28:31]\n" "s_andn2_b32 s55, 0x20000, s51\n" "s_cbranch_scc0 L4_%=\n" "L3_%=:\n"
    "ds_read_b32 v251, v241 offset:0\n" "s_waitcnt lgkmcnt(0)\n" "v_readfirstlane_b32 s51, v251\n" "s_nop 3\n" "s_andn2_b32 s55, 0x20000, s51\n" "s_cbranch_scc0 L4_%=\n" "s_sleep 1\n"
    "s_branch L3_%=\n" "L4_%=:\n" "v_mfma_i32_16x16x64_i8 v[0:3], v[192:195], v[176:179], v[0:3]\n" "ds_read_b64_tr_b8 v[220:221], v231 offset:24576\n"
    "v_mfma_i32_16x16x64_i8 v[32:35], v[192:195], v[180:183], v[32:35]\n" "ds_read_b64_tr_b8 v[222:223], v231 offset:25600\n" "v_mfma_i32_16x16x64_i8 v[64:67], v[192:195], v[184:187], v[64:67]\n"
    "v_mfma_i32_16x16x64_i8 v[96:99], v[192:195], v[188:191], v[96:99]\n" "v_mfma_i32_16x16x64_i8 v[100:103], v[196:199], v[188:191], v[100:103]\n" "ds_read_b64_tr_b8 v[192:193], v224 offset:32768\n"
    "v_mfma_i32_16x16x64_i8 v[68:71], v[196:199], v[184:187], v[68:71]\n" "ds_read_b64_tr_b8 v[194:195], v224 offset:33792\n" "v_mfma_i32_16x16x64_i8 v[36:39], v[196:199], v[180:183], v[36:39]\n"
    "v_mfma_i32_16x16x64_i8 v[4:7], v[196:199], v[176:179], v[4:7]\n" "v_mfma_i32_16x16x64_i8 v[8:11], v[200:203], v[176:179], v[8:11]\n" "ds_read_b64_tr_b8 v[196:197], v225 offset:32768\n"
    "v_mfma_i32_16x16x64_i8 v[40:43], v[200:203], v[180:183], v[40:43]\n" "ds_read_b64_tr_b8 v[198:199], v225 offset:33792\n" "v_mfma_i32_16x16x64_i8 v[72:75], v[200:203], v[184:187], v[72:75]\n"
    "v_mfma_i32_16x16x64_i8 v[104:107], v[200:203], v[188:191], v[104:107]\n" "s_waitcnt lgkmcnt(15)\n" "v_mfma_i32_16x16x64_i8 v[108:111], v[204:207], v[188:191], v[108:111]\n"
    "ds_read_b64_tr_b8 v[200:201], v226 offset:32768\n" "v_mfma_i32_16x16x64_i8 v[76:79], v[204:207], v[184:187], v[76:79]\n" "ds_read_b64_tr_b8 v[202:203], v226 offset:33792\n"
    "v_mfma_i32_16x16x64_i8 v[44:47], v[204:207], v[180:183], v[44:47]\n" "v_mfma_i32_16x16x64_i8 v[12:15], v[204:207], v[176:179], v[12:15]\n" "s_waitcnt lgkmcnt(15)\n"
    "v_mfma_i32_16x16x64_i8 v[16:19], v[208:211], v[176:179], v[16:19]\n" "ds_read_b64_tr_b8 v[204:205], v227 offset:32768\n" "v_mfma_i32_16x16x64_i8 v[48:51], v[208:211], v[180:183], v[48:51]\n"
    "ds_read_b64_tr_b8 v[206:207], v227 offset:33792\n" "v_mfma_i32_16x16x64_i8 v[80:83], v[208:211], v[184:187], v[80:83]\n" "v_mfma_i32_16x16x64_i8 v[112:115], v[208:211], v[188:191], v[112:115]\n"
    "s_waitcnt lgkmcnt(15)\n" "v_mfma_i32_16x16x64_i8 v[116:119], v[212:215], v[188:191], v[116:119]\n" "ds_read_b64_tr_b8 v[208:209], v228 offset:32768\n"
    "v_mfma_i32_16x16x64_i8 v[84:87], v[212:215], v[184:187], v[84:87]\n" "ds_read_b64_tr_b8 v[210:211], v228 offset:33792\n" "v_mfma_i32_16x16x64_i8 v[52:55], v[212:215], v[180:183], v[52:55]\n"
    "v_mfma_i32_16x16x64_i8 v[20:23], v[212:215], v[176:179], v[20:23]\n" "s_waitcnt lgkmcnt(14)\n" "v_mfma_i32_16x16x64_i8 v[24:27], v[216:219], v[176:179], v[24:27]\n"
    "ds_read_b64_tr_b8 v[212:213], v229 offset:32768\n" "v_mfma_i32_16x16x64_i8 v[56:59], v[216:219], v[180:183], v[56:59]\n" "ds_read_b64_tr_b8 v[214:215], v229 offset:33792\n"
    "v_mfma_i32_16x16x64_i8 v[88:91], v[216:219], v[184:187], v[88:91]\n" "v_mfma_i32_16x16x64_i8 v[120:123], v[216:219], v[188:191], v[120:123]\n" "s_waitcnt lgkmcnt(12)\n"
    "v_mfma_i32_16x16x64_i8 v[124:127], v[220:223], v[188:191], v[124:127]\n" "ds_read_b64_tr_b8 v[216:217], v230 offset:32768\n" "v_mfma_i32_16x16x64_i8 v[92:95], v[220:223], v[184:187], v[92:95]\n"
    "ds_read_b64_tr_b8 v[218:219], v230 offset:33792\n" "v_mfma_i32_16x16x64_i8 v[60:63], v[220:223], v[180:183], v[60:63]\n" "v_mfma_i32_16x16x64_i8 v[28:31], v[220:223], v[176:179], v[28:31]\n"
    "s_waitcnt lgkmcnt(12)\n" "v_mfma_i32_16x16x64_i8 v[0:3], v[192:195], v[128:131], v[0:3]\n" "ds_read_b64_tr_b8 v[220:221], v231 offset:32768\n"
    "v_mfma_i32_16x16x64_i8 v[32:35], v[192:195], v[132:135], v[32:35]\n" "ds_read_b64_tr_b8 v[222:223], v231 offset:33792\n" "v_mfma_i32_16x16x64_i8 v[64:67], v[192:195], v[136:139], v[64:67]\n"
    "s_waitcnt vmcnt(0)\n" "ds_read_b128 v[160:163], v249 offset:32768\n" "v_mfma_i32_16x16x64_i8 v[96:99], v[192:195], v[140:143], v[96:99]\n" "s_waitcnt lgkmcnt(13)\n"
    "v_mfma_i32_16x16x64_i8 v[100:103], v[196:199], v[140:143], v[100:103]\n" "ds_read_b64_tr_b8 v[192:193], v224 offset:40960\n" "v_mfma_i32_16x16x64_i8 v[68:71], v[196:199], v[136:139], v[68:71]\n"
    "ds_read_b64_tr_b8 v[194:195], v224 offset:41984\n" "v_mfma_i32_16x16x64_i8 v[36:39], v[196:199], v[132:135], v[36:39]\n" "ds_read_b128 v[164:167], v249 offset:34816\n"
    "v_mfma_i32_16x16x64_i8 v[4:7], v[196:199], v[128:131], v[4:7]\n" "s_waitcnt lgkmcnt(14)\n" "v_mfma_i32_16x16x64_i8 v[8:11], v[200:203], v[128:131], v[8:11]\n"
    "ds_read_b64_tr_b8 v[196:197], v225 offset:40960\n" "v_mfma_i32_16x16x64_i8 v[40:43], v[200:203], v[132:135], v[40:43]\n" "ds_read_b64_tr_b8 v[198:199], v225 offset:41984\n"
    "v_mfma_i32_16x16x64_i8 v[72:75], v[200:203], v[136:139], v[72:75]\n" "ds_read_b128 v[168:171], v249 offset:36864\n" "v_mfma_i32_16x16x64_i8 v[104:107], v[200:203], v[140:143], v[104:107]\n"
    "s_waitcnt lgkmcnt(15)\n" "v_mfma_i32_16x16x64_i8 v[108:111], v[204:207], v[140:143], v[108:111]\n" "ds_read_b64_tr_b8 v[200:201], v226 offset:40960\n"
    "v_mfma_i32_16x16x64_i8 v[76:79], v[204:207], v[136:139], v[76:79]\n" "ds_read_b64_tr_b8 v[202:203], v226 offset:41984\n" "v_mfma_i32_16x16x64_i8 v[44:47], v[204:207], v[132:135], v[44:47]\n"
    "ds_read_b32 v244, v241\n" "ds_read_b128 v[172:175], v249 offset:38912\n" "v_mfma_i32_16x16x64_i8 v[12:15], v[204:207], v[128:131], v[12:15]\n" "s_waitcnt lgkmcnt(15)\n"
    "v_mfma_i32_16x16x64_i8 v[16:19], v[208:211], v[128:131], v[16:19]\n" "ds_read_b64_tr_b8 v[204:205], v227 offset:40960\n" "v_mfma_i32_16x16x64_i8 v[48:51], v[208:211], v[132:135], v[48:51]\n"
    "ds_read_b64_tr_b8 v[206:207], v227 offset:41984\n" "v_mfma_i32_16x16x64_i8 v[80:83], v[208:211], v[136:139], v[80:83]\n" "ds_read_b128 v[176:179], v250 offset:32768\n"
    "v_mfma_i32_16x16x64_i8 v[112:115], v[208:211], v[140:143], v[112:115]\n" "v_mfma_i32_16x16x64_i8 v[116:119], v[212:215], v[140:143], v[116:119]\n"
    "ds_read_b64_tr_b8 v[208:209], v228 offset:40960\n" "v_mfma_i32_16x16x64_i8 v[84:87], v[212:215], v[136:139], v[84:87]\n" "ds_read_b64_tr_b8 v[210:211], v228 offset:41984\n"
    "v_mfma_i32_16x16x64_i8 v[52:55], v[212:215], v[132:135], v[52:55]\n" "ds_read_b128 v[180:183], v250 offset:34816\n" "v_mfma_i32_16x16x64_i8 v[20:23], v[212:215], v[128:131], v[20:23]\n"
    "s_waitcnt lgkmcnt(15)\n" "v_mfma_i32_16x16x64_i8 v[24:27], v[216:219], v[128:131], v[24:27]\n" "ds_read_b64_tr_b8 v[212:213], v229 offset:40960\n"
    "v_mfma_i32_16x16x64_i8 v[56:59], v[216:219], v[132:135], v[56:59]\n" "ds_read_b64_tr_b8 v[214:215], v229 offset:41984\n" "v_mfma_i32_16x16x64_i8 v[88:91], v[216:219], v[136:139], v[88:91]\n"
    "ds_read_b128 v[184:187], v250 offset:36864\n" "v_mfma_i32_16x16x64_i8 v[120:123], v[216:219], v[140:143], v[120:123]\n" "v_mfma_i32_16x16x64_i8 v[124:127], v[220:223], v[140:143], v[124:127]\n"
    "ds_read_b64_tr_b8 v[216:217], v230 offset:40960\n" "v_mfma_i32_16x16x64_i8 v[92:95], v[220:223], v[136:139], v[92:95]\n" "ds_read_b64_tr_b8 v[218:219], v230 offset:41984\n"
    "v_mfma_i32_16x16x64_i8 v[60:63], v[220:223], v[132:135], v[60:63]\n" "s_waitcnt lgkmcnt(12)\n" "v_readfirstlane_b32 s51, v244\n" "ds_read_b128 v[188:191], v250 offset:38912\n"
    "ds_write_b32 v242, v243\n" "v_add_u32 v243, 1, v243\n" "v_mfma_i32_16x16x64_i8 v[28:31], v[220:223], v[128:131], v[28:31]\n" "s_bitcmp1_b32 s51, 2\n" "s_cbranch_scc0 L5_%=\n"
    "s_and_b32 s55, s51, 1\n" "s_lshl_b32 s55, s55, 21\n" "s_add_u32 s40, s44, s55\n" "s_addc_u32 s41, s45, 0\n" "s_mov_b32 s42, 0x200000\n" "s_mov_b32 s56, 1\n"
    "buffer_load_dwordx4 v[128:131], v240, s[40:43], s50 offen offset:0 sc0 sc1\n" "buffer_load_dwordx4 v[132:135], v240, s[40:43], s50 offen offset:1024 sc0 sc1\n"
    "buffer_load_dwordx4 v[136:139], v240, s[40:43], s50 offen offset:2048 sc0 sc1\n" "buffer_load_dwordx4 v[140:143], v240, s[40:43], s50 offen offset:3072 sc0 sc1\n" "s_add_u32 s50, s50, 0x8000\n"
    "L5_%=:\n" "v_mfma_i32_16x16x64_i8 v[0:3], v[192:195], v[144:147], v[0:3]\n" "ds_read_b64_tr_b8 v[220:221], v231 offset:40960\n"
    "v_mfma_i32_16x16x64_i8 v[32:35], v[192:195], v[148:151], v[32:35]\n" "ds_read_b64_tr_b8 v[222:223], v231 offset:41984\n" "v_mfma_i32_16x16x64_i8 v[64:67], v[192:195], v[152:155], v[64:67]\n"
    "v_mfma_i32_16x16x64_i8 v[96:99], v[192:195], v[156:159], v[96:99]\n" "v_mfma_i32_16x16x64_i8 v[100:103], v[196:199], v[156:159], v[100:103]\n" "ds_read_b64_tr_b8 v[192:193], v224 offset:49152\n"
    "v_mfma_i32_16x16x64_i8 v[68:71], v[196:199], v[152:155], v[68:71]\n" "ds_read_b64_tr_b8 v[194:195], v224 offset:50176\n" "v_mfma_i32_16x16x64_i8 v[36:39], v[196:199], v[148:151], v[36:39]\n"
    "v_mfma_i32_16x16x64_i8 v[4:7], v[196:199], v[144:147], v[4:7]\n" "v_mfma_i32_16x16x64_i8 v[8:11], v[200:203], v[144:147], v[8:11]\n" "ds_read_b64_tr_b8 v[196:197], v225 offset:49152\n"
    "v_mfma_i32_16x16x64_i8 v[40:43], v[200:203], v[148:151], v[40:43]\n" "ds_read_b64_tr_b8 v[198:199], v225 offset:50176\n" "v_mfma_i32_16x16x64_i8 v[72:75], v[200:203], v[152:155], v[72:75]\n"
    "v_mfma_i32_16x16x64_i8 v[104:107], v[200:203], v[156:159], v[104:107]\n" "s_waitcnt lgkmcnt(15)\n" "v_mfma_i32_16x16x64_i8 v[108:111], v[204:207], v[156:159], v[108:111]\n"
    "ds_read_b64_tr_b8 v[200:201], v226 offset:49152\n" "v_mfma_i32_16x16x64_i8 v[76:79], v[204:207], v[152:155], v[76:79]\n" "ds_read_b64_tr_b8 v[202:203], v226 offset:50176\n"
    "v_mfma_i32_16x16x64_i8 v[44:47], v[204:207], v[148:151], v[44:47]\n" "ds_read_b32 v244, v241\n" "v_mfma_i32_16x16x64_i8 v[12:15], v[204:207], v[144:147], v[12:15]\n" "s_waitcnt lgkmcnt(15)\n"
    "v_mfma_i32_16x16x64_i8 v[16:19], v[208:211], v[144:147], v[16:19]\n" "ds_read_b64_tr_b8 v[204:205], v227 offset:49152\n" "v_mfma_i32_16x16x64_i8 v[48:51], v[208:211], v[148:151], v[48:51]\n"
    "ds_read_b64_tr_b8 v[206:207], v227 offset:50176\n" "v_mfma_i32_16x16x64_i8 v[80:83], v[208:211], v[152:155], v[80:83]\n" "v_mfma_i32_16x16x64_i8 v[112:115], v[208:211], v[156:159], v[112:115]\n"
    "s_waitcnt lgkmcnt(15)\n" "v_mfma_i32_16x16x64_i8 v[116:119], v[212:215], v[156:159], v[116:119]\n" "ds_read_b64_tr_b8 v[208:209], v228 offset:49152\n"
    "v_mfma_i32_16x16x64_i8 v[84:87], v[212:215], v[152:155], v[84:87]\n" "ds_read_b64_tr_b8 v[210:211], v228 offset:50176\n" "v_mfma_i32_16x16x64_i8 v[52:55], v[212:215], v[148:151], v[52:55]\n"
    "v_mfma_i32_16x16x64_i8 v[20:23], v[212:215], v[144:147], v[20:23]\n" "s_waitcnt lgkmcnt(15)\n" "v_mfma_i32_16x16x64_i8 v[24:27], v[216:219], v[144:147], v[24:27]\n"
    "ds_read_b64_tr_b8 v[212:213], v229 offset:49152\n" "v_mfma_i32_16x16x64_i8 v[56:59], v[216:219], v[148:151], v[56:59]\n" "ds_read_b64_tr_b8 v[214:215], v229 offset:50176\n"
    "v_mfma_i32_16x16x64_i8 v[88:91], v[216:219], v[152:155], v[88:91]\n" "v_mfma_i32_16x16x64_i8 v[120:123], v[216:219], v[156:159], v[120:123]\n" "s_waitcnt lgkmcnt(13)\n"
    "v_mfma_i32_16x16x64_i8 v[124:127], v[220:223], v[156:159], v[124:127]\n" "ds_read_b64_tr_b8 v[216:217], v230 offset:49152\n" "v_mfma_i32_16x16x64_i8 v[92:95], v[220:223], v[152:155], v[92:95]\n"
    "ds_read_b64_tr_b8 v[218:219], v230 offset:50176\n" "v_mfma_i32_16x16x64_i8 v[60:63], v[220:223], v[148:151], v[60:63]\n" "s_waitcnt lgkmcnt(8)\n" "v_readfirstlane_b32 s51, v244\n"
    "v_mfma_i32_16x16x64_i8 v[28:31], v[220:223], v[144:147], v[28:31]\n" "s_bitcmp1_b32 s51, 2\n" "s_cbranch_scc0 L7_%=\n" "s_cmp_lg_u32 s56, 0\n" "s_cbranch_scc1 L8_%=\n" "s_and_b32 s55, s51, 1\n"
    "s_lshl_b32 s55, s55, 21\n" "s_add_u32 s40, s44, s55\n" "s_addc_u32 s41, s45, 0\n" "s_mov_b32 s42, 0x200000\n" "s_mov_b32 s56, 1\n"
    "buffer_load_dwordx4 v[128:131], v240, s[40:43], s50 offen offset:0 sc0 sc1\n" "buffer_load_dwordx4 v[132:135], v240, s[40:43], s50 offen offset:1024 sc0 sc1\n"
    "buffer_load_dwordx4 v[136:139], v240, s[40:43], s50 offen offset:2048 sc0 sc1\n" "buffer_load_dwordx4 v[140:143], v240, s[40:43], s50 offen offset:3072 sc0 sc1\n" "s_add_u32 s50, s50, 0x8000\n"
    "L8_%=:\n" "buffer_load_dwordx4 v[144:147], v240, s[40:43], s50 offen offset:0 sc0 sc1\n" "buffer_load_dwordx4 v[148:151], v240, s[40:43], s50 offen offset:1024 sc0 sc1\n"
    "buffer_load_dwordx4 v[152:155], v240, s[40:43], s50 offen offset:2048 sc0 sc1\n" "buffer_load_dwordx4 v[156:159], v240, s[40:43], s50 offen offset:3072 sc0 sc1\n" "s_add_u32 s50, s50, 0x8000\n"
    "L7_%=:\n" "v_mfma_i32_16x16x64_i8 v[0:3], v[192:195], v[160:163], v[0:3]\n" "ds_read_b64_tr_b8 v[220:221], v231 offset:49152\n"
    "v_mfma_i32_16x16x64_i8 v[32:35], v[192:195], v[164:167], v[32:35]\n" "ds_read_b64_tr_b8 v[222:223], v231 offset:50176\n" "v_mfma_i32_16x16x64_i8 v[64:67], v[192:195], v[168:171], v[64:67]\n"
    "v_mfma_i32_16x16x64_i8 v[96:99], v[192:195], v[172:175], v[96:99]\n" "v_mfma_i32_16x16x64_i8 v[100:103], v[196:199], v[172:175], v[100:103]\n" "ds_read_b64_tr_b8 v[192:193], v224 offset:57344\n"
    "v_mfma_i32_16x16x64_i8 v[68:71], v[196:199], v[168:171], v[68:71]\n" "ds_read_b64_tr_b8 v[194:195], v224 offset:58368\n" "v_mfma_i32_16x16x64_i8 v[36:39], v[196:199], v[164:167], v[36:39]\n"
    "v_mfma_i32_16x16x64_i8 v[4:7], v[196:199], v[160:163], v[4:7]\n" "v_mfma_i32_16x16x64_i8 v[8:11], v[200:203], v[160:163], v[8:11]\n" "ds_read_b64_tr_b8 v[196:197], v225 offset:57344\n"
    "v_mfma_i32_16x16x64_i8 v[40:43], v[200:203], v[164:167], v[40:43]\n" "ds_read_b64_tr_b8 v[198:199], v225 offset:58368\n" "v_mfma_i32_16x16x64_i8 v[72:75], v[200:203], v[168:171], v[72:75]\n"
    "v_mfma_i32_16x16x64_i8 v[104:107], v[200:203], v[172:175], v[104:107]\n" "s_waitcnt lgkmcnt(12)\n" "v_mfma_i32_16x16x64_i8 v[108:111], v[204:207], v[172:175], v[108:111]\n"
    "ds_read_b64_tr_b8 v[200:201], v226 offset:57344\n" "v_mfma_i32_16x16x64_i8 v[76:79], v[204:207], v[168:171], v[76:79]\n" "ds_read_b64_tr_b8 v[202:203], v226 offset:58368\n"
    "v_mfma_i32_16x16x64_i8 v[44:47], v[204:207], v[164:167], v[44:47]\n" "ds_read_b32 v244, v241\n" "v_mfma_i32_16x16x64_i8 v[12:15], v[204:207], v[160:163], v[12:15]\n" "s_waitcnt lgkmcnt(13)\n"
    "v_mfma_i32_16x16x64_i8 v[16:19], v[208:211], v[160:163], v[16:19]\n" "ds_read_b64_tr_b8 v[204:205], v227 offset:57344\n" "v_mfma_i32_16x16x64_i8 v[48:51], v[208:211], v[164:167], v[48:51]\n"
    "ds_read_b64_tr_b8 v[206:207], v227 offset:58368\n" "v_mfma_i32_16x16x64_i8 v[80:83], v[208:211], v[168:171], v[80:83]\n" "v_mfma_i32_16x16x64_i8 v[112:115], v[208:211], v[172:175], v[112:115]\n"
    "s_waitcnt lgkmcnt(13)\n" "v_mfma_i32_16x16x64_i8 v[116:119], v[212:215], v[172:175], v[116:119]\n" "ds_read_b64_tr_b8 v[208:209], v228 offset:57344\n"
    "v_mfma_i32_16x16x64_i8 v[84:87], v[212:215], v[168:171], v[84:87]\n" "ds_read_b64_tr_b8 v[210:211], v228 offset:58368\n" "v_mfma_i32_16x16x64_i8 v[52:55], v[212:215], v[164:167], v[52:55]\n"
    "v_mfma_i32_16x16x64_i8 v[20:23], v[212:215], v[160:163], v[20:23]\n" "s_waitcnt lgkmcnt(13)\n" "v_mfma_i32_16x16x64_i8 v[24:27], v[216:219], v[160:163], v[24:27]\n"
    "ds_read_b64_tr_b8 v[212:213], v229 offset:57344\n" "v_mfma_i32_16x16x64_i8 v[56:59], v[216:219], v[164:167], v[56:59]\n" "ds_read_b64_tr_b8 v[214:215], v229 offset:58368\n"
    "v_mfma_i32_16x16x64_i8 v[88:91], v[216:219], v[168:171], v[88:91]\n" "v_mfma_i32_16x16x64_i8 v[120:123], v[216:219], v[172:175], v[120:123]\n" "s_waitcnt lgkmcnt(13)\n"
    "v_mfma_i32_16x16x64_i8 v[124:127], v[220:223], v[172:175], v[124:127]\n" "ds_read_b64_tr_b8 v[216:217], v230 offset:57344\n" "v_mfma_i32_16x16x64_i8 v[92:95], v[220:223], v[168:171], v[92:95]\n"
    "ds_read_b64_tr_b8 v[218:219], v230 offset:58368\n" "v_mfma_i32_16x16x64_i8 v[60:63], v[220:223], v[164:167], v[60:63]\n" "s_waitcnt lgkmcnt(8)\n" "v_readfirstlane_b32 s51, v244\n"
    "v_mfma_i32_16x16x64_i8 v[28:31], v[220:223], v[160:163], v[28:31]\n" "s_bitcmp1_b32 s51, 2\n" "s_cbranch_scc0 L9_%=\n" "s_cmp_lg_u32 s56, 0\n" "s_cbranch_scc1 L10_%=\n" "s_and_b32 s55, s51, 1\n"
    "s_lshl_b32 s55, s55, 21\n" "s_add_u32 s40, s44, s55\n" "s_addc_u32 s41, s45, 0\n" "s_mov_b32 s42, 0x200000\n" "s_mov_b32 s56, 1\n"
    "buffer_load_dwordx4 v[128:131], v240, s[40:43], s50 offen offset:0 sc0 sc1\n" "buffer_load_dwordx4 v[132:135], v240, s[40:43], s50 offen offset:1024 sc0 sc1\n"
    "buffer_load_dwordx4 v[136:139], v240, s[40:43], s50 offen offset:2048 sc0 sc1\n" "buffer_load_dwordx4 v[140:143], v240, s[40:43], s50 offen offset:3072 sc0 sc1\n" "s_add_u32 s50, s50, 0x8000\n"
    "buffer_load_dwordx4 v[144:147], v240, s[40:43], s50 offen offset:0 sc0 sc1\n" "buffer_load_dwordx4 v[148:151], v240, s[40:43], s50 offen offset:1024 sc0 sc1\n"
    "buffer_load_dwordx4 v[152:155], v240, s[40:43], s50 offen offset:2048 sc0 sc1\n" "buffer_load_dwordx4 v[156:159], v240, s[40:43], s50 offen offset:3072 sc0 sc1\n" "s_add_u32 s50, s50, 0x8000\n"
    "L10_%=:\n" "buffer_load_dwordx4 v[160:163], v240, s[40:43], s50 offen offset:0 sc0 sc1\n" "buffer_load_dwordx4 v[164:167], v240, s[40:43], s50 offen offset:1024 sc0 sc1\n"
    "buffer_load_dwordx4 v[168:171], v240, s[40:43], s50 offen offset:2048 sc0 sc1\n" "buffer_load_dwordx4 v[172:175], v240, s[40:43], s50 offen offset:3072 sc0 sc1\n" "s_add_u32 s50, s50, 0x8000\n"
    "L9_%=:\n" "s_andn2_b32 s55, 0x40000, s51\n" "s_cbranch_scc0 L12_%=\n" "L11_%=:\n" "ds_read_b32 v251, v241 offset:0\n" "s_waitcnt lgkmcnt(0)\n" "v_readfirstlane_b32 s51, v251\n" "s_nop 3\n"
    "s_andn2_b32 s55, 0x40000, s51\n" "s_cbranch_scc0 L12_%=\n" "s_sleep 1\n" "s_branch L11_%=\n" "L12_%=:\n" "v_mfma_i32_16x16x64_i8 v[0:3], v[192:195], v[176:179], v[0:3]\n"
    "ds_read_b64_tr_b8 v[220:221], v231 offset:57344\n" "v_mfma_i32_16x16x64_i8 v[32:35], v[192:195], v[180:183], v[32:35]\n" "ds_read_b64_tr_b8 v[222:223], v231 offset:58368\n"
    "v_mfma_i32_16x16x64_i8 v[64:67], v[192:195], v[184:187], v[64:67]\n" "v_mfma_i32_16x16x64_i8 v[96:99], v[192:195], v[188:191], v[96:99]\n"
    "v_mfma_i32_16x16x64_i8 v[100:103], v[196:199], v[188:191], v[100:103]\n" "v_xor_b32 v224, 0x10000, v224\n" "ds_read_b64_tr_b8 v[192:193], v224 offset:0\n"
    "v_mfma_i32_16x16x64_i8 v[68:71], v[196:199], v[184:187], v[68:71]\n" "ds_read_b64_tr_b8 v[194:195], v224 offset:1024\n" "v_mfma_i32_16x16x64_i8 v[36:39], v[196:199], v[180:183], v[36:39]\n"
    "v_mfma_i32_16x16x64_i8 v[4:7], v[196:199], v[176:179], v[4:7]\n" "v_mfma_i32_16x16x64_i8 v[8:11], v[200:203], v[176:179], v[8:11]\n" "v_xor_b32 v225, 0x10000, v225\n"
    "ds_read_b64_tr_b8 v[196:197], v225 offset:0\n" "v_mfma_i32_16x16x64_i8 v[40:43], v[200:203], v[180:183], v[40:43]\n" "ds_read_b64_tr_b8 v[198:199], v225 offset:1024\n"
    "v_mfma_i32_16x16x64_i8 v[72:75], v[200:203], v[184:187], v[72:75]\n" "v_mfma_i32_16x16x64_i8 v[104:107], v[200:203], v[188:191], v[104:107]\n" "s_waitcnt lgkmcnt(12)\n"
    "v_mfma_i32_16x16x64_i8 v[108:111], v[204:207], v[188:191], v[108:111]\n" "v_xor_b32 v226, 0x10000, v226\n" "ds_read_b64_tr_b8 v[200:201], v226 offset:0\n"
    "v_mfma_i32_16x16x64_i8 v[76:79], v[204:207], v[184:187], v[76:79]\n" "ds_read_b64_tr_b8 v[202:203], v226 offset:1024\n" "v_mfma_i32_16x16x64_i8 v[44:47], v[204:207], v[180:183], v[44:47]\n"
    "ds_read_b32 v244, v241\n" "v_mfma_i32_16x16x64_i8 v[12:15], v[204:207], v[176:179], v[12:15]\n" "s_waitcnt lgkmcnt(13)\n" "v_mfma_i32_16x16x64_i8 v[16:19], v[208:211], v[176:179], v[16:19]\n"
    "v_xor_b32 v227, 0x10000, v227\n" "ds_read_b64_tr_b8 v[204:205], v227 offset:0\n" "v_mfma_i32_16x16x64_i8 v[48:51], v[208:211], v[180:183], v[48:51]\n"
    "ds_read_b64_tr_b8 v[206:207], v227 offset:1024\n" "v_mfma_i32_16x16x64_i8 v[80:83], v[208:211], v[184:187], v[80:83]\n" "v_mfma_i32_16x16x64_i8 v[112:115], v[208:211], v[188:191], v[112:115]\n"
    "s_waitcnt lgkmcnt(13)\n" "v_mfma_i32_16x16x64_i8 v[116:119], v[212:215], v[188:191], v[116:119]\n" "v_xor_b32 v228, 0x10000, v228\n" "ds_read_b64_tr_b8 v[208:209], v228 offset:0\n"
    "v_mfma_i32_16x16x64_i8 v[84:87], v[212:215], v[184:187], v[84:87]\n" "ds_read_b64_tr_b8 v[210:211], v228 offset:1024\n" "v_mfma_i32_16x16x64_i8 v[52:55], v[212:215], v[180:183], v[52:55]\n"
    "v_mfma_i32_16x16x64_i8 v[20:23], v[212:215], v[176:179], v[20:23]\n" "s_waitcnt lgkmcnt(13)\n" "v_mfma_i32_16x16x64_i8 v[24:27], v[216:219], v[176:179], v[24:27]\n"
    "v_xor_b32 v229, 0x10000, v229\n" "ds_read_b64_tr_b8 v[212:213], v229 offset:0\n" "v_mfma_i32_16x16x64_i8 v[56:59], v[216:219], v[180:183], v[56:59]\n"
    "ds_read_b64_tr_b8 v[214:215], v229 offset:1024\n" "v_mfma_i32_16x16x64_i8 v[88:91], v[216:219], v[184:187], v[88:91]\n" "v_mfma_i32_16x16x64_i8 v[120:123], v[216:219], v[188:191], v[120:123]\n"
    "s_waitcnt lgkmcnt(13)\n" "v_mfma_i32_16x16x64_i8 v[124:127], v[220:223], v[188:191], v[124:127]\n" "v_xor_b32 v230, 0x10000, v230\n" "ds_read_b64_tr_b8 v[216:217], v230 offset:0\n"
    "v_mfma_i32_16x16x64_i8 v[92:95], v[220:223], v[184:187], v[92:95]\n" "ds_read_b64_tr_b8 v[218:219], v230 offset:1024\n" "v_mfma_i32_16x16x64_i8 v[60:63], v[220:223], v[180:183], v[60:63]\n"
    "s_waitcnt lgkmcnt(8)\n" "v_readfirstlane_b32 s51, v244\n" "v_mfma_i32_16x16x64_i8 v[28:31], v[220:223], v[176:179], v[28:31]\n" "s_cmp_lg_u32 s56, 0\n" "s_cbranch_scc1 L13_%=\n"
    "s_andn2_b32 s55, 4, s51\n" "s_cbranch_scc0 L16_%=\n" "L15_%=:\n" "ds_read_b32 v251, v241 offset:0\n" "s_waitcnt lgkmcnt(0)\n" "v_readfirstlane_b32 s51, v251\n" "s_nop 3\n"
    "s_andn2_b32 s55, 4, s51\n" "s_cbranch_scc0 L16_%=\n" "s_sleep 1\n" "s_branch L15_%=\n" "L16_%=:\n" "s_and_b32 s55, s51, 1\n" "s_lshl_b32 s55, s55, 21\n" "s_add_u32 s40, s44, s55\n"
    "s_addc_u32 s41, s45, 0\n" "s_mov_b32 s42, 0x200000\n" "buffer_load_dwordx4 v[128:131], v240, s[40:43], s50 offen offset:0 sc0 sc1\n"
    "buffer_load_dwordx4 v[132:135], v240, s[40:43], s50 offen offset:1024 sc0 sc1\n" "buffer_load_dwordx4 v[136:139], v240, s[40:43], s50 offen offset:2048 sc0 sc1\n"
    "buffer_load_dwordx4 v[140:143], v240, s[40:43], s50 offen offset:3072 sc0 sc1\n" "s_add_u32 s50, s50, 0x8000\n" "buffer_load_dwordx4 v[144:147], v240, s[40:43], s50 offen offset:0 sc0 sc1\n"
    "buffer_load_dwordx4 v[148:151], v240, s[40:43], s50 offen offset:1024 sc0 sc1\n" "buffer_load_dwordx4 v[152:155], v240, s[40:43], s50 offen offset:2048 sc0 sc1\n"
    "buffer_load_dwordx4 v[156:159], v240, s[40:43], s50 offen offset:3072 sc0 sc1\n" "s_add_u32 s50, s50, 0x8000\n" "buffer_load_dwordx4 v[160:163], v240, s[40:43], s50 offen offset:0 sc0 sc1\n"
    "buffer_load_dwordx4 v[164:167], v240, s[40:43], s50 offen offset:1024 sc0 sc1\n" "buffer_load_dwordx4 v[168:171], v240, s[40:43], s50 offen offset:2048 sc0 sc1\n"
    "buffer_load_dwordx4 v[172:175], v240, s[40:43], s50 offen offset:3072 sc0 sc1\n" "s_add_u32 s50, s50, 0x8000\n" "L13_%=:\n" ASTAMP(s_memrealtime, 88) "LOOP_%=:\n"
    "s_waitcnt vmcnt(11) lgkmcnt(12)\n" "v_mfma_i32_16x16x64_i8 v[0:3], v[192:195], v[128:131], v[0:3]\n" "v_xor_b32 v231, 0x10000, v231\n" "ds_read_b64_tr_b8 v[220:221], v231 offset:0\n"
    "s_waitcnt vmcnt(10)\n" "v_mfma_i32_16x16x64_i8 v[32:35], v[192:195], v[132:135], v[32:35]\n" "ds_read_b64_tr_b8 v[222:223], v231 offset:1024\n" "s_waitcnt vmcnt(9)\n"
    "v_mfma_i32_16x16x64_i8 v[64:67], v[192:195], v[136:139], v[64:67]\n" "ds_write_b32 v242, v243\n" "v_add_u32 v243, 1, v243\n" "s_waitcnt vmcnt(8)\n"
    "v_mfma_i32_16x16x64_i8 v[96:99], v[192:195], v[140:143], v[96:99]\n" "s_waitcnt lgkmcnt(13)\n" "v_mfma_i32_16x16x64_i8 v[100:103], v[196:199], v[140:143], v[100:103]\n"
    "ds_read_b64_tr_b8 v[192:193], v224 offset:8192\n" "v_mfma_i32_16x16x64_i8 v[68:71], v[196:199], v[136:139], v[68:71]\n" "ds_read_b64_tr_b8 v[194:195], v224 offset:9216\n"
    "v_mfma_i32_16x16x64_i8 v[36:39], v[196:199], v[132:135], v[36:39]\n" "buffer_load_dwordx4 v[176:179], v240, s[40:43], s50 offen offset:0 sc0 sc1\n"
    "v_mfma_i32_16x16x64_i8 v[4:7], v[196:199], v[128:131], v[4:7]\n" "s_waitcnt lgkmcnt(13)\n" "v_mfma_i32_16x16x64_i8 v[8:11], v[200:203], v[128:131], v[8:11]\n"
    "ds_read_b64_tr_b8 v[196:197], v225 offset:8192\n" "v_mfma_i32_16x16x64_i8 v[40:43], v[200:203], v[132:135], v[40:43]\n" "ds_read_b64_tr_b8 v[198:199], v225 offset:9216\n"
    "v_mfma_i32_16x16x64_i8 v[72:75], v[200:203], v[136:139], v[72:75]\n" "v_mfma_i32_16x16x64_i8 v[104:107], v[200:203], v[140:143], v[104:107]\n" "s_waitcnt lgkmcnt(13)\n"
    "v_mfma_i32_16x16x64_i8 v[108:111], v[204:207], v[140:143], v[108:111]\n" "ds_read_b64_tr_b8 v[200:201], v226 offset:8192\n" "v_mfma_i32_16x16x64_i8 v[76:79], v[204:207], v[136:139], v[76:79]\n"
    "ds_read_b64_tr_b8 v[202:203], v226 offset:9216\n" "v_mfma_i32_16x16x64_i8 v[44:47], v[204:207], v[132:135], v[44:47]\n"
    "buffer_load_dwordx4 v[180:183], v240, s[40:43], s50 offen offset:1024 sc0 sc1\n" "ds_read_b32 v244, v241\n" "v_mfma_i32_16x16x64_i8 v[12:15], v[204:207], v[128:131], v[12:15]\n"
    "s_waitcnt lgkmcnt(14)\n" "v_mfma_i32_16x16x64_i8 v[16:19], v[208:211], v[128:131], v[16:19]\n" "ds_read_b64_tr_b8 v[204:205], v227 offset:8192\n"
    "v_mfma_i32_16x16x64_i8 v[48:51], v[208:211], v[132:135], v[48:51]\n" "ds_read_b64_tr_b8 v[206:207], v227 offset:9216\n" "v_mfma_i32_16x16x64_i8 v[80:83], v[208:211], v[136:139], v[80:83]\n"
    "v_mfma_i32_16x16x64_i8 v[112:115], v[208:211], v[140:143], v[112:115]\n" "s_waitcnt lgkmcnt(14)\n" "v_mfma_i32_16x16x64_i8 v[116:119], v[212:215], v[140:143], v[116:119]\n"
    "ds_read_b64_tr_b8 v[208:209], v228 offset:8192\n" "v_mfma_i32_16x16x64_i8 v[84:87], v[212:215], v[136:139], v[84:87]\n" "ds_read_b64_tr_b8 v[210:211], v228 offset:9216\n"
    "v_mfma_i32_16x16x64_i8 v[52:55], v[212:215], v[132:135], v[52:55]\n" "buffer_load_dwordx4 v[184:187], v240, s[40:43], s50 offen offset:2048 sc0 sc1\n"
    "v_mfma_i32_16x16x64_i8 v[20:23], v[212:215], v[128:131], v[20:23]\n" "s_waitcnt lgkmcnt(14)\n" "v_mfma_i32_16x16x64_i8 v[24:27], v[216:219], v[128:131], v[24:27]\n"
    "ds_read_b64_tr_b8 v[212:213], v229 offset:8192\n" "v_mfma_i32_16x16x64_i8 v[56:59], v[216:219], v[132:135], v[56:59]\n" "ds_read_b64_tr_b8 v[214:215], v229 offset:9216\n"
    "v_mfma_i32_16x16x64_i8 v[88:91], v[216:219], v[136:139], v[88:91]\n" "v_mfma_i32_16x16x64_i8 v[120:123], v[216:219], v[140:143], v[120:123]\n" "s_waitcnt lgkmcnt(14)\n"
    "v_mfma_i32_16x16x64_i8 v[124:127], v[220:223], v[140:143], v[124:127]\n" "ds_read_b64_tr_b8 v[216:217], v230 offset:8192\n" "v_mfma_i32_16x16x64_i8 v[92:95], v[220:223], v[136:139], v[92:95]\n"
    "ds_read_b64_tr_b8 v[218:219], v230 offset:9216\n" "v_mfma_i32_16x16x64_i8 v[60:63], v[220:223], v[132:135], v[60:63]\n"
    "buffer_load_dwordx4 v[188:191], v240, s[40:43], s50 offen offset:3072 sc0 sc1\n" "s_add_u32 s50, s50, 0x8000\n" "s_waitcnt lgkmcnt(8)\n" "v_readfirstlane_b32 s51, v244\n"
    "v_mfma_i32_16x16x64_i8 v[28:31], v[220:223], v[128:131], v[28:31]\n" "s_andn2_b32 s55, s52, s51\n" "s_cbranch_scc0 L18_%=\n" "L17_%=:\n" "ds_read_b32 v251, v241 offset:0\n"
    "s_waitcnt lgkmcnt(0)\n" "v_readfirstlane_b32 s51, v251\n" "s_nop 3\n" "s_andn2_b32 s55, s52, s51\n" "s_cbranch_scc0 L18_%=\n" "s_sleep 1\n" "s_branch L17_%=\n" "L18_%=:\n" "s_waitcnt vmcnt(11)\n"
    "v_mfma_i32_16x16x64_i8 v[0:3], v[192:195], v[144:147], v[0:3]\n" "ds_read_b64_tr_b8 v[220:221], v231 offset:8192\n" "s_waitcnt vmcnt(10)\n"
    "v_mfma_i32_16x16x64_i8 v[32:35], v[192:195], v[148:151], v[32:35]\n" "ds_read_b64_tr_b8 v[222:223], v231 offset:9216\n" "s_waitcnt vmcnt(9)\n"
    "v_mfma_i32_16x16x64_i8 v[64:67], v[192:195], v[152:155], v[64:67]\n" "s_waitcnt vmcnt(8)\n" "v_mfma_i32_16x16x64_i8 v[96:99], v[192:195], v[156:159], v[96:99]\n"
    "v_mfma_i32_16x16x64_i8 v[100:103], v[196:199], v[156:159], v[100:103]\n" "ds_read_b64_tr_b8 v[192:193], v224 offset:16384\n" "v_mfma_i32_16x16x64_i8 v[68:71], v[196:199], v[152:155], v[68:71]\n"
    "ds_read_b64_tr_b8 v[194:195], v224 offset:17408\n" "v_mfma_i32_16x16x64_i8 v[36:39], v[196:199], v[148:151], v[36:39]\n"
    "buffer_load_dwordx4 v[128:131], v240, s[40:43], s50 offen offset:0 sc0 sc1\n" "v_mfma_i32_16x16x64_i8 v[4:7], v[196:199], v[144:147], v[4:7]\n"
    "v_mfma_i32_16x16x64_i8 v[8:11], v[200:203], v[144:147], v[8:11]\n" "ds_read_b64_tr_b8 v[196:197], v225 offset:16384\n" "v_mfma_i32_16x16x64_i8 v[40:43], v[200:203], v[148:151], v[40:43]\n"
    "ds_read_b64_tr_b8 v[198:199], v225 offset:17408\n" "v_mfma_i32_16x16x64_i8 v[72:75], v[200:203], v[152:155], v[72:75]\n" "v_mfma_i32_16x16x64_i8 v[104:107], v[200:203], v[156:159], v[104:107]\n"
    "s_waitcnt lgkmcnt(12)\n" "v_mfma_i32_16x16x64_i8 v[108:111], v[204:207], v[156:159], v[108:111]\n" "ds_read_b64_tr_b8 v[200:201], v226 offset:16384\n"
    "v_mfma_i32_16x16x64_i8 v[76:79], v[204:207], v[152:155], v[76:79]\n" "ds_read_b64_tr_b8 v[202:203], v226 offset:17408\n" "v_mfma_i32_16x16x64_i8 v[44:47], v[204:207], v[148:151], v[44:47]\n"
    "buffer_load_dwordx4 v[132:135], v240, s[40:43], s50 offen offset:1024 sc0 sc1\n" "v_mfma_i32_16x16x64_i8 v[12:15], v[204:207], v[144:147], v[12:15]\n" "s_waitcnt lgkmcnt(12)\n"
    "v_mfma_i32_16x16x64_i8 v[16:19], v[208:211], v[144:147], v[16:19]\n" "ds_read_b64_tr_b8 v[204:205], v227 offset:16384\n" "v_mfma_i32_16x16x64_i8 v[48:51], v[208:211], v[148:151], v[48:51]\n"
    "ds_read_b64_tr_b8 v[206:207], v227 offset:17408\n" "v_mfma_i32_16x16x64_i8 v[80:83], v[208:211], v[152:155], v[80:83]\n" "v_mfma_i32_16x16x64_i8 v[112:115], v[208:211], v[156:159], v[112:115]\n"
    "s_waitcnt lgkmcnt(12)\n" "v_mfma_i32_16x16x64_i8 v[116:119], v[212:215], v[156:159], v[116:119]\n" "ds_read_b64_tr_b8 v[208:209], v228 offset:16384\n"
    "v_mfma_i32_16x16x64_i8 v[84:87], v[212:215], v[152:155], v[84:87]\n" "ds_read_b64_tr_b8 v[210:211], v228 offset:17408\n" "v_mfma_i32_16x16x64_i8 v[52:55], v[212:215], v[148:151], v[52:55]\n"
    "buffer_load_dwordx4 v[136:139], v240, s[40:43], s50 offen offset:2048 sc0 sc1\n" "v_mfma_i32_16x16x64_i8 v[20:23], v[212:215], v[144:147], v[20:23]\n" "s_waitcnt lgkmcnt(12)\n"
    "v_mfma_i32_16x16x64_i8 v[24:27], v[216:219], v[144:147], v[24:27]\n" "ds_read_b64_tr_b8 v[212:213], v229 offset:16384\n" "v_mfma_i32_16x16x64_i8 v[56:59], v[216:219], v[148:151], v[56:59]\n"
    "ds_read_b64_tr_b8 v[214:215], v229 offset:17408\n" "v_mfma_i32_16x16x64_i8 v[88:91], v[216:219], v[152:155], v[88:91]\n" "v_mfma_i32_16x16x64_i8 v[120:123], v[216:219], v[156:159], v[120:123]\n"
    "s_waitcnt lgkmcnt(12)\n" "v_mfma_i32_16x16x64_i8 v[124:127], v[220:223], v[156:159], v[124:127]\n" "ds_read_b64_tr_b8 v[216:217], v230 offset:16384\n"
    "v_mfma_i32_16x16x64_i8 v[92:95], v[220:223], v[152:155], v[92:95]\n" "ds_read_b64_tr_b8 v[218:219], v230 offset:17408\n" "v_mfma_i32_16x16x64_i8 v[60:63], v[220:223], v[148:151], v[60:63]\n"
    "buffer_load_dwordx4 v[140:143], v240, s[40:43], s50 offen offset:3072 sc0 sc1\n" "s_add_u32 s50, s50, 0x8000\n" "v_mfma_i32_16x16x64_i8 v[28:31], v[220:223], v[144:147], v[28:31]\n"
    "s_waitcnt vmcnt(11) lgkmcnt(12)\n" "v_mfma_i32_16x16x64_i8 v[0:3], v[192:195], v[160:163], v[0:3]\n" "ds_read_b64_tr_b8 v[220:221], v231 offset:16384\n" "s_waitcnt vmcnt(10)\n"
    "v_mfma_i32_16x16x64_i8 v[32:35], v[192:195], v[164:167], v[32:35]\n" "ds_read_b64_tr_b8 v[222:223], v231 offset:17408\n" "s_waitcnt vmcnt(9)\n"
    "v_mfma_i32_16x16x64_i8 v[64:67], v[192:195], v[168:171], v[64:67]\n" "s_waitcnt vmcnt(8)\n" "v_mfma_i32_16x16x64_i8 v[96:99], v[192:195], v[172:175], v[96:99]\n" "s_waitcnt lgkmcnt(12)\n"
    "v_mfma_i32_16x16x64_i8 v[100:103], v[196:199], v[172:175], v[100:103]\n" "ds_read_b64_tr_b8 v[192:193], v224 offset:24576\n" "v_mfma_i32_16x16x64_i8 v[68:71], v[196:199], v[168:171], v[68:71]\n"
    "ds_read_b64_tr_b8 v[194:195], v224 offset:25600\n" "v_mfma_i32_16x16x64_i8 v[36:39], v[196:199], v[164:167], v[36:39]\n"
    "buffer_load_dwordx4 v[144:147], v240, s[40:43], s50 offen offset:0 sc0 sc1\n" "v_mfma_i32_16x16x64_i8 v[4:7], v[196:199], v[160:163], v[4:7]\n" "s_waitcnt lgkmcnt(12)\n"
    "v_mfma_i32_16x16x64_i8 v[8:11], v[200:203], v[160:163], v[8:11]\n" "ds_read_b64_tr_b8 v[196:197], v225 offset:24576\n" "v_mfma_i32_16x16x64_i8 v[40:43], v[200:203], v[164:167], v[40:43]\n"
    "ds_read_b64_tr_b8 v[198:199], v225 offset:25600\n" "v_mfma_i32_16x16x64_i8 v[72:75], v[200:203], v[168:171], v[72:75]\n" "v_mfma_i32_16x16x64_i8 v[104:107], v[200:203], v[172:175], v[104:107]\n"
    "s_waitcnt lgkmcnt(12)\n" "v_mfma_i32_16x16x64_i8 v[108:111], v[204:207], v[172:175], v[108:111]\n" "ds_read_b64_tr_b8 v[200:201], v226 offset:24576\n"
    "v_mfma_i32_16x16x64_i8 v[76:79], v[204:207], v[168:171], v[76:79]\n" "ds_read_b64_tr_b8 v[202:203], v226 offset:25600\n" "v_mfma_i32_16x16x64_i8 v[44:47], v[204:207], v[164:167], v[44:47]\n"
    "buffer_load_dwordx4 v[148:151], v240, s[40:43], s50 offen offset:1024 sc0 sc1\n" "ds_read_b32 v244, v241\n" "v_mfma_i32_16x16x64_i8 v[12:15], v[204:207], v[160:163], v[12:15]\n"
    "s_waitcnt lgkmcnt(13)\n" "v_mfma_i32_16x16x64_i8 v[16:19], v[208:211], v[160:163], v[16:19]\n" "ds_read_b64_tr_b8 v[204:205], v227 offset:24576\n"
    "v_mfma_i32_16x16x64_i8 v[48:51], v[208:211], v[164:167], v[48:51]\n" "ds_read_b64_tr_b8 v[206:207], v227 offset:25600\n" "v_mfma_i32_16x16x64_i8 v[80:83], v[208:211], v[168:171], v[80:83]\n"
    "v_mfma_i32_16x16x64_i8 v[112:115], v[208:211], v[172:175], v[112:115]\n" "s_waitcnt lgkmcnt(13)\n" "v_mfma_i32_16x16x64_i8 v[116:119], v[212:215], v[172:175], v[116:119]\n"
    "ds_read_b64_tr_b8 v[208:209], v228 offset:24576\n" "v_mfma_i32_16x16x64_i8 v[84:87], v[212:215], v[168:171], v[84:87]\n" "ds_read_b64_tr_b8 v[210:211], v228 offset:25600\n"
    "v_mfma_i32_16x16x64_i8 v[52:55], v[212:215], v[164:167], v[52:55]\n" "buffer_load_dwordx4 v[152:155], v240, s[40:43], s50 offen offset:2048 sc0 sc1\n"
    "v_mfma_i32_16x16x64_i8 v[20:23], v[212:215], v[160:163], v[20:23]\n" "s_waitcnt lgkmcnt(13)\n" "v_mfma_i32_16x16x64_i8 v[24:27], v[216:219], v[160:163], v[24:27]\n"
    "ds_read_b64_tr_b8 v[212:213], v229 offset:24576\n" "v_mfma_i32_16x16x64_i8 v[56:59], v[216:219], v[164:167], v[56:59]\n" "ds_read_b64_tr_b8 v[214:215], v229 offset:25600\n"
    "v_mfma_i32_16x16x64_i8 v[88:91], v[216:219], v[168:171], v[88:91]\n" "v_mfma_i32_16x16x64_i8 v[120:123], v[216:219], v[172:175], v[120:123]\n" "s_waitcnt lgkmcnt(13)\n"
    "v_mfma_i32_16x16x64_i8 v[124:127], v[220:223], v[172:175], v[124:127]\n" "ds_read_b64_tr_b8 v[216:217], v230 offset:24576\n" "v_mfma_i32_16x16x64_i8 v[92:95], v[220:223], v[168:171], v[92:95]\n"
    "ds_read_b64_tr_b8 v[218:219], v230 offset:25600\n" "v_mfma_i32_16x16x64_i8 v[60:63], v[220:223], v[164:167], v[60:63]\n"
    "buffer_load_dwordx4 v[156:159], v240, s[40:43], s50 offen offset:3072 sc0 sc1\n" "s_add_u32 s50, s50, 0x8000\n" "s_waitcnt lgkmcnt(8)\n" "v_readfirstlane_b32 s51, v244\n"
    "v_mfma_i32_16x16x64_i8 v[28:31], v[220:223], v[160:163], v[28:31]\n" "s_lshl_b32 s57, s52, 16\n" "s_lshl_b32 s52, s52, 1\n" "s_andn2_b32 s55, s57, s51\n" "s_cbranch_scc0 L20_%=\n" "L19_%=:\n"
    "ds_read_b32 v251, v241 offset:0\n" "s_waitcnt lgkmcnt(0)\n" "v_readfirstlane_b32 s51, v251\n" "s_nop 3\n" "s_andn2_b32 s55, s57, s51\n" "s_cbranch_scc0 L20_%=\n" "s_sleep 1\n" "s_branch L19_%=\n"
    "L20_%=:\n" "s_waitcnt vmcnt(11)\n" "v_mfma_i32_16x16x64_i8 v[0:3], v[192:195], v[176:179], v[0:3]\n" "ds_read_b64_tr_b8 v[220:221], v231 offset:24576\n" "s_waitcnt vmcnt(10)\n"
    "v_mfma_i32_16x16x64_i8 v[32:35], v[192:195], v[180:183], v[32:35]\n" "ds_read_b64_tr_b8 v[222:223], v231 offset:25600\n" "s_waitcnt vmcnt(9)\n"
    "v_mfma_i32_16x16x64_i8 v[64:67], v[192:195], v[184:187], v[64:67]\n" "s_waitcnt vmcnt(8)\n" "v_mfma_i32_16x16x64_i8 v[96:99], v[192:195], v[188:191], v[96:99]\n"
    "v_mfma_i32_16x16x64_i8 v[100:103], v[196:199], v[188:191], v[100:103]\n" "ds_read_b64_tr_b8 v[192:193], v224 offset:32768\n" "v_mfma_i32_16x16x64_i8 v[68:71], v[196:199], v[184:187], v[68:71]\n"
    "ds_read_b64_tr_b8 v[194:195], v224 offset:33792\n" "v_mfma_i32_16x16x64_i8 v[36:39], v[196:199], v[180:183], v[36:39]\n"
    "buffer_load_dwordx4 v[160:163], v240, s[40:43], s50 offen offset:0 sc0 sc1\n" "v_mfma_i32_16x16x64_i8 v[4:7], v[196:199], v[176:179], v[4:7]\n"
    "v_mfma_i32_16x16x64_i8 v[8:11], v[200:203], v[176:179], v[8:11]\n" "ds_read_b64_tr_b8 v[196:197], v225 offset:32768\n" "v_mfma_i32_16x16x64_i8 v[40:43], v[200:203], v[180:183], v[40:43]\n"
    "ds_read_b64_tr_b8 v[198:199], v225 offset:33792\n" "v_mfma_i32_16x16x64_i8 v[72:75], v[200:203], v[184:187], v[72:75]\n" "v_mfma_i32_16x16x64_i8 v[104:107], v[200:203], v[188:191], v[104:107]\n"
    "s_waitcnt lgkmcnt(12)\n" "v_mfma_i32_16x16x64_i8 v[108:111], v[204:207], v[188:191], v[108:111]\n" "ds_read_b64_tr_b8 v[200:201], v226 offset:32768\n"
    "v_mfma_i32_16x16x64_i8 v[76:79], v[204:207], v[184:187], v[76:79]\n" "ds_read_b64_tr_b8 v[202:203], v226 offset:33792\n" "v_mfma_i32_16x16x64_i8 v[44:47], v[204:207], v[180:183], v[44:47]\n"
    "buffer_load_dwordx4 v[164:167], v240, s[40:43], s50 offen offset:1024 sc0 sc1\n" "v_mfma_i32_16x16x64_i8 v[12:15], v[204:207], v[176:179], v[12:15]\n" "s_waitcnt lgkmcnt(12)\n"
    "v_mfma_i32_16x16x64_i8 v[16:19], v[208:211], v[176:179], v[16:19]\n" "ds_read_b64_tr_b8 v[204:205], v227 offset:32768\n" "v_mfma_i32_16x16x64_i8 v[48:51], v[208:211], v[180:183], v[48:51]\n"
    "ds_read_b64_tr_b8 v[206:207], v227 offset:33792\n" "v_mfma_i32_16x16x64_i8 v[80:83], v[208:211], v[184:187], v[80:83]\n" "v_mfma_i32_16x16x64_i8 v[112:115], v[208:211], v[188:191], v[112:115]\n"
    "s_waitcnt lgkmcnt(12)\n" "v_mfma_i32_16x16x64_i8 v[116:119], v[212:215], v[188:191], v[116:119]\n" "ds_read_b64_tr_b8 v[208:209], v228 offset:32768\n"
    "v_mfma_i32_16x16x64_i8 v[84:87], v[212:215], v[184:187], v[84:87]\n" "ds_read_b64_tr_b8 v[210:211], v228 offset:33792\n" "v_mfma_i32_16x16x64_i8 v[52:55], v[212:215], v[180:183], v[52:55]\n"
    "buffer_load_dwordx4 v[168:171], v240, s[40:43], s50 offen offset:2048 sc0 sc1\n" "v_mfma_i32_16x16x64_i8 v[20:23], v[212:215], v[176:179], v[20:23]\n" "s_waitcnt lgkmcnt(12)\n"
    "v_mfma_i32_16x16x64_i8 v[24:27], v[216:219], v[176:179], v[24:27]\n" "ds_read_b64_tr_b8 v[212:213], v229 offset:32768\n" "v_mfma_i32_16x16x64_i8 v[56:59], v[216:219], v[180:183], v[56:59]\n"
    "ds_read_b64_tr_b8 v[214:215], v229 offset:33792\n" "v_mfma_i32_16x16x64_i8 v[88:91], v[216:219], v[184:187], v[88:91]\n" "v_mfma_i32_16x16x64_i8 v[120:123], v[216:219], v[188:191], v[120:123]\n"
    "s_waitcnt lgkmcnt(12)\n" "v_mfma_i32_16x16x64_i8 v[124:127], v[220:223], v[188:191], v[124:127]\n" "ds_read_b64_tr_b8 v[216:217], v230 offset:32768\n"
    "v_mfma_i32_16x16x64_i8 v[92:95], v[220:223], v[184:187], v[92:95]\n" "ds_read_b64_tr_b8 v[218:219], v230 offset:33792\n" "v_mfma_i32_16x16x64_i8 v[60:63], v[220:223], v[180:183], v[60:63]\n"
    "buffer_load_dwordx4 v[172:175], v240, s[40:43], s50 offen offset:3072 sc0 sc1\n" "s_add_u32 s50, s50, 0x8000\n" "v_mfma_i32_16x16x64_i8 v[28:31], v[220:223], v[176:179], v[28:31]\n"
    "s_waitcnt vmcnt(11) lgkmcnt(12)\n" "v_mfma_i32_16x16x64_i8 v[0:3], v[192:195], v[128:131], v[0:3]\n" "ds_read_b64_tr_b8 v[220:221], v231 offset:32768\n" "s_waitcnt vmcnt(10)\n"
    "v_mfma_i32_16x16x64_i8 v[32:35], v[192:195], v[132:135], v[32:35]\n" "ds_read_b64_tr_b8 v[222:223], v231 offset:33792\n" "s_waitcnt vmcnt(9)\n"
    "v_mfma_i32_16x16x64_i8 v[64:67], v[192:195], v[136:139], v[64:67]\n" "ds_write_b32 v242, v243\n" "v_add_u32 v243, 1, v243\n" "s_waitcnt vmcnt(8)\n"
    "v_mfma_i32_16x16x64_i8 v[96:99], v[192:195], v[140:143], v[96:99]\n" "s_waitcnt lgkmcnt(13)\n" "v_mfma_i32_16x16x64_i8 v[100:103], v[196:199], v[140:143], v[100:103]\n"
    "ds_read_b64_tr_b8 v[192:193], v224 offset:40960\n" "v_mfma_i32_16x16x64_i8 v[68:71], v[196:199], v[136:139], v[68:71]\n" "ds_read_b64_tr_b8 v[194:195], v224 offset:41984\n"
    "v_mfma_i32_16x16x64_i8 v[36:39], v[196:199], v[132:135], v[36:39]\n" "buffer_load_dwordx4 v[176:179], v240, s[40:43], s50 offen offset:0 sc0 sc1\n"
    "v_mfma_i32_16x16x64_i8 v[4:7], v[196:199], v[128:131], v[4:7]\n" "s_waitcnt lgkmcnt(13)\n" "v_mfma_i32_16x16x64_i8 v[8:11], v[200:203], v[128:131], v[8:11]\n"
    "ds_read_b64_tr_b8 v[196:197], v225 offset:40960\n" "v_mfma_i32_16x16x64_i8 v[40:43], v[200:203], v[132:135], v[40:43]\n" "ds_read_b64_tr_b8 v[198:199], v225 offset:41984\n"
    "v_mfma_i32_16x16x64_i8 v[72:75], v[200:203], v[136:139], v[72:75]\n" "v_mfma_i32_16x16x64_i8 v[104:107], v[200:203], v[140:143], v[104:107]\n" "s_waitcnt lgkmcnt(13)\n"
    "v_mfma_i32_16x16x64_i8 v[108:111], v[204:207], v[140:143], v[108:111]\n" "ds_read_b64_tr_b8 v[200:201], v226 offset:40960\n" "v_mfma_i32_16x16x64_i8 v[76:79], v[204:207], v[136:139], v[76:79]\n"
    "ds_read_b64_tr_b8 v[202:203], v226 offset:41984\n" "v_mfma_i32_16x16x64_i8 v[44:47], v[204:207], v[132:135], v[44:47]\n"
    "buffer_load_dwordx4 v[180:183], v240, s[40:43], s50 offen offset:1024 sc0 sc1\n" "ds_read_b32 v244, v241\n" "v_mfma_i32_16x16x64_i8 v[12:15], v[204:207], v[128:131], v[12:15]\n"
    "s_waitcnt lgkmcnt(14)\n" "v_mfma_i32_16x16x64_i8 v[16:19], v[208:211], v[128:131], v[16:19]\n" "ds_read_b64_tr_b8 v[204:205], v227 offset:40960\n"
    "v_mfma_i32_16x16x64_i8 v[48:51], v[208:211], v[132:135], v[48:51]\n" "ds_read_b64_tr_b8 v[206:207], v227 offset:41984\n" "v_mfma_i32_16x16x64_i8 v[80:83], v[208:211], v[136:139], v[80:83]\n"
    "v_mfma_i32_16x16x64_i8 v[112:115], v[208:211], v[140:143], v[112:115]\n" "s_waitcnt lgkmcnt(14)\n" "v_mfma_i32_16x16x64_i8 v[116:119], v[212:215], v[140:143], v[116:119]\n"
    "ds_read_b64_tr_b8 v[208:209], v228 offset:40960\n" "v_mfma_i32_16x16x64_i8 v[84:87], v[212:215], v[136:139], v[84:87]\n" "ds_read_b64_tr_b8 v[210:211], v228 offset:41984\n"
    "v_mfma_i32_16x16x64_i8 v[52:55], v[212:215], v[132:135], v[52:55]\n" "buffer_load_dwordx4 v[184:187], v240, s[40:43], s50 offen offset:2048 sc0 sc1\n"
    "v_mfma_i32_16x16x64_i8 v[20:23], v[212:215], v[128:131], v[20:23]\n" "s_waitcnt lgkmcnt(14)\n" "v_mfma_i32_16x16x64_i8 v[24:27], v[216:219], v[128:131], v[24:27]\n"
    "ds_read_b64_tr_b8 v[212:213], v229 offset:40960\n" "v_mfma_i32_16x16x64_i8 v[56:59], v[216:219], v[132:135], v[56:59]\n" "ds_read_b64_tr_b8 v[214:215], v229 offset:41984\n"
    "v_mfma_i32_16x16x64_i8 v[88:91], v[216:219], v[136:139], v[88:91]\n" "v_mfma_i32_16x16x64_i8 v[120:123], v[216:219], v[140:143], v[120:123]\n" "s_waitcnt lgkmcnt(14)\n"
    "v_mfma_i32_16x16x64_i8 v[124:127], v[220:223], v[140:143], v[124:127]\n" "ds_read_b64_tr_b8 v[216:217], v230 offset:40960\n" "v_mfma_i32_16x16x64_i8 v[92:95], v[220:223], v[136:139], v[92:95]\n"
    "ds_read_b64_tr_b8 v[218:219], v230 offset:41984\n" "v_mfma_i32_16x16x64_i8 v[60:63], v[220:223], v[132:135], v[60:63]\n"
    "buffer_load_dwordx4 v[188:191], v240, s[40:43], s50 offen offset:3072 sc0 sc1\n" "s_add_u32 s50, s50, 0x8000\n" "s_waitcnt lgkmcnt(8)\n" "v_readfirstlane_b32 s51, v244\n"
    "v_mfma_i32_16x16x64_i8 v[28:31], v[220:223], v[128:131], v[28:31]\n" "s_andn2_b32 s55, s52, s51\n" "s_cbranch_scc0 L22_%=\n" "L21_%=:\n" "ds_read_b32 v251, v241 offset:0\n"
    "s_waitcnt lgkmcnt(0)\n" "v_readfirstlane_b32 s51, v251\n" "s_nop 3\n" "s_andn2_b32 s55, s52, s51\n" "s_cbranch_scc0 L22_%=\n" "s_sleep 1\n" "s_branch L21_%=\n" "L22_%=:\n" "s_waitcnt vmcnt(11)\n"
    "v_mfma_i32_16x16x64_i8 v[0:3], v[192:195], v[144:147], v[0:3]\n" "ds_read_b64_tr_b8 v[220:221], v231 offset:40960\n" "s_waitcnt vmcnt(10)\n"
    "v_mfma_i32_16x16x64_i8 v[32:35], v[192:195], v[148:151], v[32:35]\n" "ds_read_b64_tr_b8 v[222:223], v231 offset:41984\n" "s_waitcnt vmcnt(9)\n"
    "v_mfma_i32_16x16x64_i8 v[64:67], v[192:195], v[152:155], v[64:67]\n" "s_waitcnt vmcnt(8)\n" "v_mfma_i32_16x16x64_i8 v[96:99], v[192:195], v[156:159], v[96:99]\n"
    "v_mfma_i32_16x16x64_i8 v[100:103], v[196:199], v[156:159], v[100:103]\n" "ds_read_b64_tr_b8 v[192:193], v224 offset:49152\n" "v_mfma_i32_16x16x64_i8 v[68:71], v[196:199], v[152:155], v[68:71]\n"
    "ds_read_b64_tr_b8 v[194:195], v224 offset:50176\n" "v_mfma_i32_16x16x64_i8 v[36:39], v[196:199], v[148:151], v[36:39]\n"
    "buffer_load_dwordx4 v[128:131], v240, s[40:43], s50 offen offset:0 sc0 sc1\n" "v_mfma_i32_16x16x64_i8 v[4:7], v[196:199], v[144:147], v[4:7]\n"
    "v_mfma_i32_16x16x64_i8 v[8:11], v[200:203], v[144:147], v[8:11]\n" "ds_read_b64_tr_b8 v[196:197], v225 offset:49152\n" "v_mfma_i32_16x16x64_i8 v[40:43], v[200:203], v[148:151], v[40:43]\n"
    "ds_read_b64_tr_b8 v[198:199], v225 offset:50176\n" "v_mfma_i32_16x16x64_i8 v[72:75], v[200:203], v[152:155], v[72:75]\n" "v_mfma_i32_16x16x64_i8 v[104:107], v[200:203], v[156:159], v[104:107]\n"
    "s_waitcnt lgkmcnt(12)\n" "v_mfma_i32_16x16x64_i8 v[108:111], v[204:207], v[156:159], v[108:111]\n" "ds_read_b64_tr_b8 v[200:201], v226 offset:49152\n"
    "v_mfma_i32_16x16x64_i8 v[76:79], v[204:207], v[152:155], v[76:79]\n" "ds_read_b64_tr_b8 v[202:203], v226 offset:50176\n" "v_mfma_i32_16x16x64_i8 v[44:47], v[204:207], v[148:151], v[44:47]\n"
    "buffer_load_dwordx4 v[132:135], v240, s[40:43], s50 offen offset:1024 sc0 sc1\n" "v_mfma_i32_16x16x64_i8 v[12:15], v[204:207], v[144:147], v[12:15]\n" "s_waitcnt lgkmcnt(12)\n"
    "v_mfma_i32_16x16x64_i8 v[16:19], v[208:211], v[144:147], v[16:19]\n" "ds_read_b64_tr_b8 v[204:205], v227 offset:49152\n" "v_mfma_i32_16x16x64_i8 v[48:51], v[208:211], v[148:151], v[48:51]\n"
    "ds_read_b64_tr_b8 v[206:207], v227 offset:50176\n" "v_mfma_i32_16x16x64_i8 v[80:83], v[208:211], v[152:155], v[80:83]\n" "v_mfma_i32_16x16x64_i8 v[112:115], v[208:211], v[156:159], v[112:115]\n"
    "s_waitcnt lgkmcnt(12)\n" "v_mfma_i32_16x16x64_i8 v[116:119], v[212:215], v[156:159], v[116:119]\n" "ds_read_b64_tr_b8 v[208:209], v228 offset:49152\n"
    "v_mfma_i32_16x16x64_i8 v[84:87], v[212:215], v[152:155], v[84:87]\n" "ds_read_b64_tr_b8 v[210:211], v228 offset:50176\n" "v_mfma_i32_16x16x64_i8 v[52:55], v[212:215], v[148:151], v[52:55]\n"
    "buffer_load_dwordx4 v[136:139], v240, s[40:43], s50 offen offset:2048 sc0 sc1\n" "v_mfma_i32_16x16x64_i8 v[20:23], v[212:215], v[144:147], v[20:23]\n" "s_waitcnt lgkmcnt(12)\n"
    "v_mfma_i32_16x16x64_i8 v[24:27], v[216:219], v[144:147], v[24:27]\n" "ds_read_b64_tr_b8 v[212:213], v229 offset:49152\n" "v_mfma_i32_16x16x64_i8 v[56:59], v[216:219], v[148:151], v[56:59]\n"
    "ds_read_b64_tr_b8 v[214:215], v229 offset:50176\n" "v_mfma_i32_16x16x64_i8 v[88:91], v[216:219], v[152:155], v[88:91]\n" "v_mfma_i32_16x16x64_i8 v[120:123], v[216:219], v[156:159], v[120:123]\n"
    "s_waitcnt lgkmcnt(12)\n" "v_mfma_i32_16x16x64_i8 v[124:127], v[220:223], v[156:159], v[124:127]\n" "ds_read_b64_tr_b8 v[216:217], v230 offset:49152\n"
    "v_mfma_i32_16x16x64_i8 v[92:95], v[220:223], v[152:155], v[92:95]\n" "ds_read_b64_tr_b8 v[218:219], v230 offset:50176\n" "v_mfma_i32_16x16x64_i8 v[60:63], v[220:223], v[148:151], v[60:63]\n"
    "buffer_load_dwordx4 v[140:143], v240, s[40:43], s50 offen offset:3072 sc0 sc1\n" "s_add_u32 s50, s50, 0x8000\n" "v_mfma_i32_16x16x64_i8 v[28:31], v[220:223], v[144:147], v[28:31]\n"
    "s_waitcnt vmcnt(11) lgkmcnt(12)\n" "v_mfma_i32_16x16x64_i8 v[0:3], v[192:195], v[160:163], v[0:3]\n" "ds_read_b64_tr_b8 v[220:221], v231 offset:49152\n" "s_waitcnt vmcnt(10)\n"
    "v_mfma_i32_16x16x64_i8 v[32:35], v[192:195], v[164:167], v[32:35]\n" "ds_read_b64_tr_b8 v[222:223], v231 offset:50176\n" "s_waitcnt vmcnt(9)\n"
    "v_mfma_i32_16x16x64_i8 v[64:67], v[192:195], v[168:171], v[64:67]\n" "s_waitcnt vmcnt(8)\n" "v_mfma_i32_16x16x64_i8 v[96:99], v[192:195], v[172:175], v[96:99]\n" "s_waitcnt lgkmcnt(12)\n"
    "v_mfma_i32_16x16x64_i8 v[100:103], v[196:199], v[172:175], v[100:103]\n" "ds_read_b64_tr_b8 v[192:193], v224 offset:57344\n" "v_mfma_i32_16x16x64_i8 v[68:71], v[196:199], v[168:171], v[68:71]\n"
    "ds_read_b64_tr_b8 v[194:195], v224 offset:58368\n" "v_mfma_i32_16x16x64_i8 v[36:39], v[196:199], v[164:167], v[36:39]\n"
    "buffer_load_dwordx4 v[144:147], v240, s[40:43], s50 offen offset:0 sc0 sc1\n" "v_mfma_i32_16x16x64_i8 v[4:7], v[196:199], v[160:163], v[4:7]\n" "s_waitcnt lgkmcnt(12)\n"
    "v_mfma_i32_16x16x64_i8 v[8:11], v[200:203], v[160:163], v[8:11]\n" "ds_read_b64_tr_b8 v[196:197], v225 offset:57344\n" "v_mfma_i32_16x16x64_i8 v[40:43], v[200:203], v[164:167], v[40:43]\n"
    "ds_read_b64_tr_b8 v[198:199], v225 offset:58368\n" "v_mfma_i32_16x16x64_i8 v[72:75], v[200:203], v[168:171], v[72:75]\n" "v_mfma_i32_16x16x64_i8 v[104:107], v[200:203], v[172:175], v[104:107]\n"
    "s_waitcnt lgkmcnt(12)\n" "v_mfma_i32_16x16x64_i8 v[108:111], v[204:207], v[172:175], v[108:111]\n" "ds_read_b64_tr_b8 v[200:201], v226 offset:57344\n"
    "v_mfma_i32_16x16x64_i8 v[76:79], v[204:207], v[168:171], v[76:79]\n" "ds_read_b64_tr_b8 v[202:203], v226 offset:58368\n" "v_mfma_i32_16x16x64_i8 v[44:47], v[204:207], v[164:167], v[44:47]\n"
    "buffer_load_dwordx4 v[148:151], v240, s[40:43], s50 offen offset:1024 sc0 sc1\n" "ds_read_b32 v244, v241\n" "v_mfma_i32_16x16x64_i8 v[12:15], v[204:207], v[160:163], v[12:15]\n"
    "s_waitcnt lgkmcnt(13)\n" "v_mfma_i32_16x16x64_i8 v[16:19], v[208:211], v[160:163], v[16:19]\n" "ds_read_b64_tr_b8 v[204:205], v227 offset:57344\n"
    "v_mfma_i32_16x16x64_i8 v[48:51], v[208:211], v[164:167], v[48:51]\n" "ds_read_b64_tr_b8 v[206:207], v227 offset:58368\n" "v_mfma_i32_16x16x64_i8 v[80:83], v[208:211], v[168:171], v[80:83]\n"
    "v_mfma_i32_16x16x64_i8 v[112:115], v[208:211], v[172:175], v[112:115]\n" "s_waitcnt lgkmcnt(13)\n" "v_mfma_i32_16x16x64_i8 v[116:119], v[212:215], v[172:175], v[116:119]\n"
    "ds_read_b64_tr_b8 v[208:209], v228 offset:57344\n" "v_mfma_i32_16x16x64_i8 v[84:87], v[212:215], v[168:171], v[84:87]\n" "ds_read_b64_tr_b8 v[210:211], v228 offset:58368\n"
    "v_mfma_i32_16x16x64_i8 v[52:55], v[212:215], v[164:167], v[52:55]\n" "buffer_load_dwordx4 v[152:155], v240, s[40:43], s50 offen offset:2048 sc0 sc1\n"
    "v_mfma_i32_16x16x64_i8 v[20:23], v[212:215], v[160:163], v[20:23]\n" "s_waitcnt lgkmcnt(13)\n" "v_mfma_i32_16x16x64_i8 v[24:27], v[216:219], v[160:163], v[24:27]\n"
    "ds_read_b64_tr_b8 v[212:213], v229 offset:57344\n" "v_mfma_i32_16x16x64_i8 v[56:59], v[216:219], v[164:167], v[56:59]\n" "ds_read_b64_tr_b8 v[214:215], v229 offset:58368\n"
    "v_mfma_i32_16x16x64_i8 v[88:91], v[216:219], v[168:171], v[88:91]\n" "v_mfma_i32_16x16x64_i8 v[120:123], v[216:219], v[172:175], v[120:123]\n" "s_waitcnt lgkmcnt(13)\n"
    "v_mfma_i32_16x16x64_i8 v[124:127], v[220:223], v[172:175], v[124:127]\n" "ds_read_b64_tr_b8 v[216:217], v230 offset:57344\n" "v_mfma_i32_16x16x64_i8 v[92:95], v[220:223], v[168:171], v[92:95]\n"
    "ds_read_b64_tr_b8 v[218:219], v230 offset:58368\n" "v_mfma_i32_16x16x64_i8 v[60:63], v[220:223], v[164:167], v[60:63]\n"
    "buffer_load_dwordx4 v[156:159], v240, s[40:43], s50 offen offset:3072 sc0 sc1\n" "s_add_u32 s50, s50, 0x8000\n" "s_waitcnt lgkmcnt(8)\n" "v_readfirstlane_b32 s51, v244\n"
    "v_mfma_i32_16x16x64_i8 v[28:31], v[220:223], v[160:163], v[28:31]\n" "s_lshl_b32 s57, s52, 16\n" "s_lshl_b32 s52, s52, 1\n" "s_andn2_b32 s55, s57, s51\n" "s_cbranch_scc0 L24_%=\n" "L23_%=:\n"
    "ds_read_b32 v251, v241 offset:0\n" "s_waitcnt lgkmcnt(0)\n" "v_readfirstlane_b32 s51, v251\n" "s_nop 3\n" "s_andn2_b32 s55, s57, s51\n" "s_cbranch_scc0 L24_%=\n" "s_sleep 1\n" "s_branch L23_%=\n"
    "L24_%=:\n" "s_waitcnt vmcnt(11)\n" "v_mfma_i32_16x16x64_i8 v[0:3], v[192:195], v[176:179], v[0:3]\n" "ds_read_b64_tr_b8 v[220:221], v231 offset:57344\n" "s_waitcnt vmcnt(10)\n"
    "v_mfma_i32_16x16x64_i8 v[32:35], v[192:195], v[180:183], v[32:35]\n" "ds_read_b64_tr_b8 v[222:223], v231 offset:58368\n" "s_waitcnt vmcnt(9)\n"
    "v_mfma_i32_16x16x64_i8 v[64:67], v[192:195], v[184:187], v[64:67]\n" "s_waitcnt vmcnt(8)\n" "v_mfma_i32_16x16x64_i8 v[96:99], v[192:195], v[188:191], v[96:99]\n"
    "v_mfma_i32_16x16x64_i8 v[100:103], v[196:199], v[188:191], v[100:103]\n" "v_xor_b32 v224, 0x10000, v224\n" "ds_read_b64_tr_b8 v[192:193], v224 offset:0\n"
    "v_mfma_i32_16x16x64_i8 v[68:71], v[196:199], v[184:187], v[68:71]\n" "ds_read_b64_tr_b8 v[194:195], v224 offset:1024\n" "v_mfma_i32_16x16x64_i8 v[36:39], v[196:199], v[180:183], v[36:39]\n"
    "buffer_load_dwordx4 v[160:163], v240, s[40:43], s50 offen offset:0 sc0 sc1\n" "v_mfma_i32_16x16x64_i8 v[4:7], v[196:199], v[176:179], v[4:7]\n"
    "v_mfma_i32_16x16x64_i8 v[8:11], v[200:203], v[176:179], v[8:11]\n" "v_xor_b32 v225, 0x10000, v225\n" "ds_read_b64_tr_b8 v[196:197], v225 offset:0\n"
    "v_mfma_i32_16x16x64_i8 v[40:43], v[200:203], v[180:183], v[40:43]\n" "ds_read_b64_tr_b8 v[198:199], v225 offset:1024\n" "v_mfma_i32_16x16x64_i8 v[72:75], v[200:203], v[184:187], v[72:75]\n"
    "v_mfma_i32_16x16x64_i8 v[104:107], v[200:203], v[188:191], v[104:107]\n" "s_waitcnt lgkmcnt(12)\n" "v_mfma_i32_16x16x64_i8 v[108:111], v[204:207], v[188:191], v[108:111]\n"
    "v_xor_b32 v226, 0x10000, v226\n" "ds_read_b64_tr_b8 v[200:201], v226 offset:0\n" "v_mfma_i32_16x16x64_i8 v[76:79], v[204:207], v[184:187], v[76:79]\n"
    "ds_read_b64_tr_b8 v[202:203], v226 offset:1024\n" "v_mfma_i32_16x16x64_i8 v[44:47], v[204:207], v[180:183], v[44:47]\n"
    "buffer_load_dwordx4 v[164:167], v240, s[40:43], s50 offen offset:1024 sc0 sc1\n" "v_mfma_i32_16x16x64_i8 v[12:15], v[204:207], v[176:179], v[12:15]\n" "s_waitcnt lgkmcnt(12)\n"
    "v_mfma_i32_16x16x64_i8 v[16:19], v[208:211], v[176:179], v[16:19]\n" "v_xor_b32 v227, 0x10000, v227\n" "ds_read_b64_tr_b8 v[204:205], v227 offset:0\n"
    "v_mfma_i32_16x16x64_i8 v[48:51], v[208:211], v[180:183], v[48:51]\n" "ds_read_b64_tr_b8 v[206:207], v227 offset:1024\n" "v_mfma_i32_16x16x64_i8 v[80:83], v[208:211], v[184:187], v[80:83]\n"
    "v_mfma_i32_16x16x64_i8 v[112:115], v[208:211], v[188:191], v[112:115]\n" "s_waitcnt lgkmcnt(12)\n" "v_mfma_i32_16x16x64_i8 v[116:119], v[212:215], v[188:191], v[116:119]\n"
    "v_xor_b32 v228, 0x10000, v228\n" "ds_read_b64_tr_b8 v[208:209], v228 offset:0\n" "v_mfma_i32_16x16x64_i8 v[84:87], v[212:215], v[184:187], v[84:87]\n"
    "ds_read_b64_tr_b8 v[210:211], v228 offset:1024\n" "v_mfma_i32_16x16x64_i8 v[52:55], v[212:215], v[180:183], v[52:55]\n"
    "buffer_load_dwordx4 v[168:171], v240, s[40:43], s50 offen offset:2048 sc0 sc1\n" "v_mfma_i32_16x16x64_i8 v[20:23], v[212:215], v[176:179], v[20:23]\n" "s_waitcnt lgkmcnt(12)\n"
    "v_mfma_i32_16x16x64_i8 v[24:27], v[216:219], v[176:179], v[24:27]\n" "v_xor_b32 v229, 0x10000, v229\n" "ds_read_b64_tr_b8 v[212:213], v229 offset:0\n"
    "v_mfma_i32_16x16x64_i8 v[56:59], v[216:219], v[180:183], v[56:59]\n" "ds_read_b64_tr_b8 v[214:215], v229 offset:1024\n" "v_mfma_i32_16x16x64_i8 v[88:91], v[216:219], v[184:187], v[88:91]\n"
    "v_mfma_i32_16x16x64_i8 v[120:123], v[216:219], v[188:191], v[120:123]\n" "s_waitcnt lgkmcnt(12)\n" "v_mfma_i32_16x16x64_i8 v[124:127], v[220:223], v[188:191], v[124:127]\n"
    "v_xor_b32 v230, 0x10000, v230\n" "ds_read_b64_tr_b8 v[216:217], v230 offset:0\n" "v_mfma_i32_16x16x64_i8 v[92:95], v[220:223], v[184:187], v[92:95]\n"
    "ds_read_b64_tr_b8 v[218:219], v230 offset:1024\n" "v_mfma_i32_16x16x64_i8 v[60:63], v[220:223], v[180:183], v[60:63]\n"
    "buffer_load_dwordx4 v[172:175], v240, s[40:43], s50 offen offset:3072 sc0 sc1\n" "s_add_u32 s50, s50, 0x8000\n" "v_mfma_i32_16x16x64_i8 v[28:31], v[220:223], v[176:179], v[28:31]\n"
    "s_sub_u32 s53, s53, 1\n" "s_cmp_lg_u32 s53, 0\n" "s_cbranch_scc1 LOOP_%=\n" "s_waitcnt vmcnt(11) lgkmcnt(12)\n" "v_mfma_i32_16x16x64_i8 v[0:3], v[192:195], v[128:131], v[0:3]\n"
    "v_xor_b32 v231, 0x10000, v231\n" "ds_read_b64_tr_b8 v[220:221], v231 offset:0\n" "s_waitcnt vmcnt(10)\n" "v_mfma_i32_16x16x64_i8 v[32:35], v[192:195], v[132:135], v[32:35]\n"
    "ds_read_b64_tr_b8 v[222:223], v231 offset:1024\n" "s_waitcnt vmcnt(9)\n" "v_mfma_i32_16x16x64_i8 v[64:67], v[192:195], v[136:139], v[64:67]\n" "ds_write_b32 v242, v243\n"
    "v_add_u32 v243, 1, v243\n" "s_waitcnt vmcnt(8)\n" "v_mfma_i32_16x16x64_i8 v[96:99], v[192:195], v[140:143], v[96:99]\n" "s_waitcnt lgkmcnt(13)\n"
    "v_mfma_i32_16x16x64_i8 v[100:103], v[196:199], v[140:143], v[100:103]\n" "ds_read_b64_tr_b8 v[192:193], v224 offset:8192\n" "v_mfma_i32_16x16x64_i8 v[68:71], v[196:199], v[136:139], v[68:71]\n"
    "ds_read_b64_tr_b8 v[194:195], v224 offset:9216\n" "v_mfma_i32_16x16x64_i8 v[36:39], v[196:199], v[132:135], v[36:39]\n"
    "buffer_load_dwordx4 v[176:179], v240, s[40:43], s50 offen offset:0 sc0 sc1\n" "v_mfma_i32_16x16x64_i8 v[4:7], v[196:199], v[128:131], v[4:7]\n" "s_waitcnt lgkmcnt(13)\n"
    "v_mfma_i32_16x16x64_i8 v[8:11], v[200:203], v[128:131], v[8:11]\n" "ds_read_b64_tr_b8 v[196:197], v225 offset:8192\n" "v_mfma_i32_16x16x64_i8 v[40:43], v[200:203], v[132:135], v[40:43]\n"
    "ds_read_b64_tr_b8 v[198:199], v225 offset:9216\n" "v_mfma_i32_16x16x64_i8 v[72:75], v[200:203], v[136:139], v[72:75]\n" "v_mfma_i32_16x16x64_i8 v[104:107], v[200:203], v[140:143], v[104:107]\n"
    "s_waitcnt lgkmcnt(13)\n" "v_mfma_i32_16x16x64_i8 v[108:111], v[204:207], v[140:143], v[108:111]\n" "ds_read_b64_tr_b8 v[200:201], v226 offset:8192\n"
    "v_mfma_i32_16x16x64_i8 v[76:79], v[204:207], v[136:139], v[76:79]\n" "ds_read_b64_tr_b8 v[202:203], v226 offset:9216\n" "v_mfma_i32_16x16x64_i8 v[44:47], v[204:207], v[132:135], v[44:47]\n"
    "buffer_load_dwordx4 v[180:183], v240, s[40:43], s50 offen offset:1024 sc0 sc1\n" "ds_read_b32 v244, v241\n" "v_mfma_i32_16x16x64_i8 v[12:15], v[204:207], v[128:131], v[12:15]\n"
    "s_waitcnt lgkmcnt(14)\n" "v_mfma_i32_16x16x64_i8 v[16:19], v[208:211], v[128:131], v[16:19]\n" "ds_read_b64_tr_b8 v[204:205], v227 offset:8192\n"
    "v_mfma_i32_16x16x64_i8 v[48:51], v[208:211], v[132:135], v[48:51]\n" "ds_read_b64_tr_b8 v[206:207], v227 offset:9216\n" "v_mfma_i32_16x16x64_i8 v[80:83], v[208:211], v[136:139], v[80:83]\n"
    "v_mfma_i32_16x16x64_i8 v[112:115], v[208:211], v[140:143], v[112:115]\n" "s_waitcnt lgkmcnt(14)\n" "v_mfma_i32_16x16x64_i8 v[116:119], v[212:215], v[140:143], v[116:119]\n"
    "ds_read_b64_tr_b8 v[208:209], v228 offset:8192\n" "v_mfma_i32_16x16x64_i8 v[84:87], v[212:215], v[136:139], v[84:87]\n" "ds_read_b64_tr_b8 v[210:211], v228 offset:9216\n"
    "v_mfma_i32_16x16x64_i8 v[52:55], v[212:215], v[132:135], v[52:55]\n" "buffer_load_dwordx4 v[184:187], v240, s[40:43], s50 offen offset:2048 sc0 sc1\n"
    "v_mfma_i32_16x16x64_i8 v[20:23], v[212:215], v[128:131], v[20:23]\n" "s_waitcnt lgkmcnt(14)\n" "v_mfma_i32_16x16x64_i8 v[24:27], v[216:219], v[128:131], v[24:27]\n"
    "ds_read_b64_tr_b8 v[212:213], v229 offset:8192\n" "v_mfma_i32_16x16x64_i8 v[56:59], v[216:219], v[132:135], v[56:59]\n" "ds_read_b64_tr_b8 v[214:215], v229 offset:9216\n"
    "v_mfma_i32_16x16x64_i8 v[88:91], v[216:219], v[136:139], v[88:91]\n" "v_mfma_i32_16x16x64_i8 v[120:123], v[216:219], v[140:143], v[120:123]\n" "s_waitcnt lgkmcnt(14)\n"
    "v_mfma_i32_16x16x64_i8 v[124:127], v[220:223], v[140:143], v[124:127]\n" "ds_read_b64_tr_b8 v[216:217], v230 offset:8192\n" "v_mfma_i32_16x16x64_i8 v[92:95], v[220:223], v[136:139], v[92:95]\n"
    "ds_read_b64_tr_b8 v[218:219], v230 offset:9216\n" "v_mfma_i32_16x16x64_i8 v[60:63], v[220:223], v[132:135], v[60:63]\n"
    "buffer_load_dwordx4 v[188:191], v240, s[40:43], s50 offen offset:3072 sc0 sc1\n" "s_add_u32 s50, s50, 0x8000\n" "s_waitcnt lgkmcnt(8)\n" "v_readfirstlane_b32 s51, v244\n"
    "v_mfma_i32_16x16x64_i8 v[28:31], v[220:223], v[128:131], v[28:31]\n" "s_andn2_b32 s55, 0x800, s51\n" "s_cbranch_scc0 L26_%=\n" "L25_%=:\n" "ds_read_b32 v251, v241 offset:0\n"
    "s_waitcnt lgkmcnt(0)\n" "v_readfirstlane_b32 s51, v251\n" "s_nop 3\n" "s_andn2_b32 s55, 0x800, s51\n" "s_cbranch_scc0 L26_%=\n" "s_sleep 1\n" "s_branch L25_%=\n" "L26_%=:\n"
    "s_waitcnt vmcnt(11)\n" "v_mfma_i32_16x16x64_i8 v[0:3], v[192:195], v[144:147], v[0:3]\n" "ds_read_b64_tr_b8 v[220:221], v231 offset:8192\n" "s_waitcnt vmcnt(10)\n"
    "v_mfma_i32_16x16x64_i8 v[32:35], v[192:195], v[148:151], v[32:35]\n" "ds_read_b64_tr_b8 v[222:223], v231 offset:9216\n" "s_waitcnt vmcnt(9)\n"
    "v_mfma_i32_16x16x64_i8 v[64:67], v[192:195], v[152:155], v[64:67]\n" "s_waitcnt vmcnt(8)\n" "v_mfma_i32_16x16x64_i8 v[96:99], v[192:195], v[156:159], v[96:99]\n"
    "v_mfma_i32_16x16x64_i8 v[100:103], v[196:199], v[156:159], v[100:103]\n" "ds_read_b64_tr_b8 v[192:193], v224 offset:16384\n" "v_mfma_i32_16x16x64_i8 v[68:71], v[196:199], v[152:155], v[68:71]\n"
    "ds_read_b64_tr_b8 v[194:195], v224 offset:17408\n" "v_mfma_i32_16x16x64_i8 v[36:39], v[196:199], v[148:151], v[36:39]\n"
    "buffer_load_dwordx4 v[128:131], v240, s[40:43], s50 offen offset:0 sc0 sc1\n" "v_mfma_i32_16x16x64_i8 v[4:7], v[196:199], v[144:147], v[4:7]\n"
    "v_mfma_i32_16x16x64_i8 v[8:11], v[200:203], v[144:147], v[8:11]\n" "ds_read_b64_tr_b8 v[196:197], v225 offset:16384\n" "v_mfma_i32_16x16x64_i8 v[40:43], v[200:203], v[148:151], v[40:43]\n"
    "ds_read_b64_tr_b8 v[198:199], v225 offset:17408\n" "v_mfma_i32_16x16x64_i8 v[72:75], v[200:203], v[152:155], v[72:75]\n" "v_mfma_i32_16x16x64_i8 v[104:107], v[200:203], v[156:159], v[104:107]\n"
    "s_waitcnt lgkmcnt(12)\n" "v_mfma_i32_16x16x64_i8 v[108:111], v[204:207], v[156:159], v[108:111]\n" "ds_read_b64_tr_b8 v[200:201], v226 offset:16384\n"
    "v_mfma_i32_16x16x64_i8 v[76:79], v[204:207], v[152:155], v[76:79]\n" "ds_read_b64_tr_b8 v[202:203], v226 offset:17408\n" "v_mfma_i32_16x16x64_i8 v[44:47], v[204:207], v[148:151], v[44:47]\n"
    "buffer_load_dwordx4 v[132:135], v240, s[40:43], s50 offen offset:1024 sc0 sc1\n" "v_mfma_i32_16x16x64_i8 v[12:15], v[204:207], v[144:147], v[12:15]\n" "s_waitcnt lgkmcnt(12)\n"
    "v_mfma_i32_16x16x64_i8 v[16:19], v[208:211], v[144:147], v[16:19]\n" "ds_read_b64_tr_b8 v[204:205], v227 offset:16384\n" "v_mfma_i32_16x16x64_i8 v[48:51], v[208:211], v[148:151], v[48:51]\n"
    "ds_read_b64_tr_b8 v[206:207], v227 offset:17408\n" "v_mfma_i32_16x16x64_i8 v[80:83], v[208:211], v[152:155], v[80:83]\n" "v_mfma_i32_16x16x64_i8 v[112:115], v[208:211], v[156:159], v[112:115]\n"
    "s_waitcnt lgkmcnt(12)\n" "v_mfma_i32_16x16x64_i8 v[116:119], v[212:215], v[156:159], v[116:119]\n" "ds_read_b64_tr_b8 v[208:209], v228 offset:16384\n"
    "v_mfma_i32_16x16x64_i8 v[84:87], v[212:215], v[152:155], v[84:87]\n" "ds_read_b64_tr_b8 v[210:211], v228 offset:17408\n" "v_mfma_i32_16x16x64_i8 v[52:55], v[212:215], v[148:151], v[52:55]\n"
    "buffer_load_dwordx4 v[136:139], v240, s[40:43], s50 offen offset:2048 sc0 sc1\n" "v_mfma_i32_16x16x64_i8 v[20:23], v[212:215], v[144:147], v[20:23]\n" "s_waitcnt lgkmcnt(12)\n"
    "v_mfma_i32_16x16x64_i8 v[24:27], v[216:219], v[144:147], v[24:27]\n" "ds_read_b64_tr_b8 v[212:213], v229 offset:16384\n" "v_mfma_i32_16x16x64_i8 v[56:59], v[216:219], v[148:151], v[56:59]\n"
    "ds_read_b64_tr_b8 v[214:215], v229 offset:17408\n" "v_mfma_i32_16x16x64_i8 v[88:91], v[216:219], v[152:155], v[88:91]\n" "v_mfma_i32_16x16x64_i8 v[120:123], v[216:219], v[156:159], v[120:123]\n"
    "s_waitcnt lgkmcnt(12)\n" "v_mfma_i32_16x16x64_i8 v[124:127], v[220:223], v[156:159], v[124:127]\n" "ds_read_b64_tr_b8 v[216:217], v230 offset:16384\n"
    "v_mfma_i32_16x16x64_i8 v[92:95], v[220:223], v[152:155], v[92:95]\n" "ds_read_b64_tr_b8 v[218:219], v230 offset:17408\n" "v_mfma_i32_16x16x64_i8 v[60:63], v[220:223], v[148:151], v[60:63]\n"
    "buffer_load_dwordx4 v[140:143], v240, s[40:43], s50 offen offset:3072 sc0 sc1\n" "s_add_u32 s50, s50, 0x8000\n" "v_mfma_i32_16x16x64_i8 v[28:31], v[220:223], v[144:147], v[28:31]\n"
    "s_waitcnt vmcnt(11) lgkmcnt(12)\n" "v_mfma_i32_16x16x64_i8 v[0:3], v[192:195], v[160:163], v[0:3]\n" "ds_read_b64_tr_b8 v[220:221], v231 offset:16384\n" "s_waitcnt vmcnt(10)\n"
    "v_mfma_i32_16x16x64_i8 v[32:35], v[192:195], v[164:167], v[32:35]\n" "ds_read_b64_tr_b8 v[222:223], v231 offset:17408\n" "s_waitcnt vmcnt(9)\n"
    "v_mfma_i32_16x16x64_i8 v[64:67], v[192:195], v[168:171], v[64:67]\n" "s_waitcnt vmcnt(8)\n" "v_mfma_i32_16x16x64_i8 v[96:99], v[192:195], v[172:175], v[96:99]\n" "s_waitcnt lgkmcnt(12)\n"
    "v_mfma_i32_16x16x64_i8 v[100:103], v[196:199], v[172:175], v[100:103]\n" "ds_read_b64_tr_b8 v[192:193], v224 offset:24576\n" "v_mfma_i32_16x16x64_i8 v[68:71], v[196:199], v[168:171], v[68:71]\n"
    "ds_read_b64_tr_b8 v[194:195], v224 offset:25600\n" "v_mfma_i32_16x16x64_i8 v[36:39], v[196:199], v[164:167], v[36:39]\n"
    "buffer_load_dwordx4 v[144:147], v240, s[40:43], s50 offen offset:0 sc0 sc1\n" "v_mfma_i32_16x16x64_i8 v[4:7], v[196:199], v[160:163], v[4:7]\n" "s_waitcnt lgkmcnt(12)\n"
    "v_mfma_i32_16x16x64_i8 v[8:11], v[200:203], v[160:163], v[8:11]\n" "ds_read_b64_tr_b8 v[196:197], v225 offset:24576\n" "v_mfma_i32_16x16x64_i8 v[40:43], v[200:203], v[164:167], v[40:43]\n"
    "ds_read_b64_tr_b8 v[198:199], v225 offset:25600\n" "v_mfma_i32_16x16x64_i8 v[72:75], v[200:203], v[168:171], v[72:75]\n" "v_mfma_i32_16x16x64_i8 v[104:107], v[200:203], v[172:175], v[104:107]\n"
    "s_waitcnt lgkmcnt(12)\n" "v_mfma_i32_16x16x64_i8 v[108:111], v[204:207], v[172:175], v[108:111]\n" "ds_read_b64_tr_b8 v[200:201], v226 offset:24576\n"
    "v_mfma_i32_16x16x64_i8 v[76:79], v[204:207], v[168:171], v[76:79]\n" "ds_read_b64_tr_b8 v[202:203], v226 offset:25600\n" "v_mfma_i32_16x16x64_i8 v[44:47], v[204:207], v[164:167], v[44:47]\n"
    "buffer_load_dwordx4 v[148:151], v240, s[40:43], s50 offen offset:1024 sc0 sc1\n" "ds_read_b32 v244, v241\n" "v_mfma_i32_16x16x64_i8 v[12:15], v[204:207], v[160:163], v[12:15]\n"
    "s_waitcnt lgkmcnt(13)\n" "v_mfma_i32_16x16x64_i8 v[16:19], v[208:211], v[160:163], v[16:19]\n" "ds_read_b64_tr_b8 v[204:205], v227 offset:24576\n"
    "v_mfma_i32_16x16x64_i8 v[48:51], v[208:211], v[164:167], v[48:51]\n" "ds_read_b64_tr_b8 v[206:207], v227 offset:25600\n" "v_mfma_i32_16x16x64_i8 v[80:83], v[208:211], v[168:171], v[80:83]\n"
    "v_mfma_i32_16x16x64_i8 v[112:115], v[208:211], v[172:175], v[112:115]\n" "s_waitcnt lgkmcnt(13)\n" "v_mfma_i32_16x16x64_i8 v[116:119], v[212:215], v[172:175], v[116:119]\n"
    "ds_read_b64_tr_b8 v[208:209], v228 offset:24576\n" "v_mfma_i32_16x16x64_i8 v[84:87], v[212:215], v[168:171], v[84:87]\n" "ds_read_b64_tr_b8 v[210:211], v228 offset:25600\n"
    "v_mfma_i32_16x16x64_i8 v[52:55], v[212:215], v[164:167], v[52:55]\n" "buffer_load_dwordx4 v[152:155], v240, s[40:43], s50 offen offset:2048 sc0 sc1\n"
    "v_mfma_i32_16x16x64_i8 v[20:23], v[212:215], v[160:163], v[20:23]\n" "s_waitcnt lgkmcnt(13)\n" "v_mfma_i32_16x16x64_i8 v[24:27], v[216:219], v[160:163], v[24:27]\n"
    "ds_read_b64_tr_b8 v[212:213], v229 offset:24576\n" "v_mfma_i32_16x16x64_i8 v[56:59], v[216:219], v[164:167], v[56:59]\n" "ds_read_b64_tr_b8 v[214:215], v229 offset:25600\n"
    "v_mfma_i32_16x16x64_i8 v[88:91], v[216:219], v[168:171], v[88:91]\n" "v_mfma_i32_16x16x64_i8 v[120:123], v[216:219], v[172:175], v[120:123]\n" "s_waitcnt lgkmcnt(13)\n"
    "v_mfma_i32_16x16x64_i8 v[124:127], v[220:223], v[172:175], v[124:127]\n" "ds_read_b64_tr_b8 v[216:217], v230 offset:24576\n" "v_mfma_i32_16x16x64_i8 v[92:95], v[220:223], v[168:171], v[92:95]\n"
    "ds_read_b64_tr_b8 v[218:219], v230 offset:25600\n" "v_mfma_i32_16x16x64_i8 v[60:63], v[220:223], v[164:167], v[60:63]\n"
    "buffer_load_dwordx4 v[156:159], v240, s[40:43], s50 offen offset:3072 sc0 sc1\n" "s_add_u32 s50, s50, 0x8000\n" "s_waitcnt lgkmcnt(8)\n" "v_readfirstlane_b32 s51, v244\n"
    "v_mfma_i32_16x16x64_i8 v[28:31], v[220:223], v[160:163], v[28:31]\n" "s_andn2_b32 s55, 0x8000000, s51\n" "s_cbranch_scc0 L28_%=\n" "L27_%=:\n" "ds_read_b32 v251, v241 offset:0\n"
    "s_waitcnt lgkmcnt(0)\n" "v_readfirstlane_b32 s51, v251\n" "s_nop 3\n" "s_andn2_b32 s55, 0x8000000, s51\n" "s_cbranch_scc0 L28_%=\n" "s_sleep 1\n" "s_branch L27_%=\n" "L28_%=:\n"
    "s_waitcnt vmcnt(11)\n" "v_mfma_i32_16x16x64_i8 v[0:3], v[192:195], v[176:179], v[0:3]\n" "ds_read_b64_tr_b8 v[220:221], v231 offset:24576\n" "s_waitcnt vmcnt(10)\n"
    "v_mfma_i32_16x16x64_i8 v[32:35], v[192:195], v[180:183], v[32:35]\n" "ds_read_b64_tr_b8 v[222:223], v231 offset:25600\n" "s_waitcnt vmcnt(9)\n"
    "v_mfma_i32_16x16x64_i8 v[64:67], v[192:195], v[184:187], v[64:67]\n" "s_waitcnt vmcnt(8)\n" "v_mfma_i32_16x16x64_i8 v[96:99], v[192:195], v[188:191], v[96:99]\n"
    "v_mfma_i32_16x16x64_i8 v[100:103], v[196:199], v[188:191], v[100:103]\n" "ds_read_b64_tr_b8 v[192:193], v224 offset:32768\n" "v_mfma_i32_16x16x64_i8 v[68:71], v[196:199], v[184:187], v[68:71]\n"
    "ds_read_b64_tr_b8 v[194:195], v224 offset:33792\n" "v_mfma_i32_16x16x64_i8 v[36:39], v[196:199], v[180:183], v[36:39]\n"
    "buffer_load_dwordx4 v[160:163], v240, s[40:43], s50 offen offset:0 sc0 sc1\n" "v_mfma_i32_16x16x64_i8 v[4:7], v[196:199], v[176:179], v[4:7]\n"
    "v_mfma_i32_16x16x64_i8 v[8:11], v[200:203], v[176:179], v[8:11]\n" "ds_read_b64_tr_b8 v[196:197], v225 offset:32768\n" "v_mfma_i32_16x16x64_i8 v[40:43], v[200:203], v[180:183], v[40:43]\n"
    "ds_read_b64_tr_b8 v[198:199], v225 offset:33792\n" "v_mfma_i32_16x16x64_i8 v[72:75], v[200:203], v[184:187], v[72:75]\n" "v_mfma_i32_16x16x64_i8 v[104:107], v[200:203], v[188:191], v[104:107]\n"
    "s_waitcnt lgkmcnt(12)\n" "v_mfma_i32_16x16x64_i8 v[108:111], v[204:207], v[188:191], v[108:111]\n" "ds_read_b64_tr_b8 v[200:201], v226 offset:32768\n"
    "v_mfma_i32_16x16x64_i8 v[76:79], v[204:207], v[184:187], v[76:79]\n" "ds_read_b64_tr_b8 v[202:203], v226 offset:33792\n" "v_mfma_i32_16x16x64_i8 v[44:47], v[204:207], v[180:183], v[44:47]\n"
    "buffer_load_dwordx4 v[164:167], v240, s[40:43], s50 offen offset:1024 sc0 sc1\n" "v_mfma_i32_16x16x64_i8 v[12:15], v[204:207], v[176:179], v[12:15]\n" "s_waitcnt lgkmcnt(12)\n"
    "v_mfma_i32_16x16x64_i8 v[16:19], v[208:211], v[176:179], v[16:19]\n" "ds_read_b64_tr_b8 v[204:205], v227 offset:32768\n" "v_mfma_i32_16x16x64_i8 v[48:51], v[208:211], v[180:183], v[48:51]\n"
    "ds_read_b64_tr_b8 v[206:207], v227 offset:33792\n" "v_mfma_i32_16x16x64_i8 v[80:83], v[208:211], v[184:187], v[80:83]\n" "v_mfma_i32_16x16x64_i8 v[112:115], v[208:211], v[188:191], v[112:115]\n"
    "s_waitcnt lgkmcnt(12)\n" "v_mfma_i32_16x16x64_i8 v[116:119], v[212:215], v[188:191], v[116:119]\n" "ds_read_b64_tr_b8 v[208:209], v228 offset:32768\n"
    "v_mfma_i32_16x16x64_i8 v[84:87], v[212:215], v[184:187], v[84:87]\n" "ds_read_b64_tr_b8 v[210:211], v228 offset:33792\n" "v_mfma_i32_16x16x64_i8 v[52:55], v[212:215], v[180:183], v[52:55]\n"
    "buffer_load_dwordx4 v[168:171], v240, s[40:43], s50 offen offset:2048 sc0 sc1\n" "v_mfma_i32_16x16x64_i8 v[20:23], v[212:215], v[176:179], v[20:23]\n" "s_waitcnt lgkmcnt(12)\n"
    "v_mfma_i32_16x16x64_i8 v[24:27], v[216:219], v[176:179], v[24:27]\n" "ds_read_b64_tr_b8 v[212:213], v229 offset:32768\n" "v_mfma_i32_16x16x64_i8 v[56:59], v[216:219], v[180:183], v[56:59]\n"
    "ds_read_b64_tr_b8 v[214:215], v229 offset:33792\n" "v_mfma_i32_16x16x64_i8 v[88:91], v[216:219], v[184:187], v[88:91]\n" "v_mfma_i32_16x16x64_i8 v[120:123], v[216:219], v[188:191], v[120:123]\n"
    "s_waitcnt lgkmcnt(12)\n" "v_mfma_i32_16x16x64_i8 v[124:127], v[220:223], v[188:191], v[124:127]\n" "ds_read_b64_tr_b8 v[216:217], v230 offset:32768\n"
    "v_mfma_i32_16x16x64_i8 v[92:95], v[220:223], v[184:187], v[92:95]\n" "ds_read_b64_tr_b8 v[218:219], v230 offset:33792\n" "v_mfma_i32_16x16x64_i8 v[60:63], v[220:223], v[180:183], v[60:63]\n"
    "buffer_load_dwordx4 v[172:175], v240, s[40:43], s50 offen offset:3072 sc0 sc1\n" "s_add_u32 s50, s50, 0x8000\n" "v_mfma_i32_16x16x64_i8 v[28:31], v[220:223], v[176:179], v[28:31]\n"
    "s_waitcnt vmcnt(11) lgkmcnt(12)\n" "v_mfma_i32_16x16x64_i8 v[0:3], v[192:195], v[128:131], v[0:3]\n" "ds_read_b64_tr_b8 v[220:221], v231 offset:32768\n" "s_waitcnt vmcnt(10)\n"
    "v_mfma_i32_16x16x64_i8 v[32:35], v[192:195], v[132:135], v[32:35]\n" "ds_read_b64_tr_b8 v[222:223], v231 offset:33792\n" "s_waitcnt vmcnt(9)\n"
    "v_mfma_i32_16x16x64_i8 v[64:67], v[192:195], v[136:139], v[64:67]\n" "ds_write_b32 v242, v243\n" "v_add_u32 v243, 1, v243\n" "s_waitcnt vmcnt(8)\n"
    "v_mfma_i32_16x16x64_i8 v[96:99], v[192:195], v[140:143], v[96:99]\n" "s_waitcnt lgkmcnt(13)\n" "v_mfma_i32_16x16x64_i8 v[100:103], v[196:199], v[140:143], v[100:103]\n"
    "ds_read_b64_tr_b8 v[192:193], v224 offset:40960\n" "v_mfma_i32_16x16x64_i8 v[68:71], v[196:199], v[136:139], v[68:71]\n" "ds_read_b64_tr_b8 v[194:195], v224 offset:41984\n"
    "v_mfma_i32_16x16x64_i8 v[36:39], v[196:199], v[132:135], v[36:39]\n" "buffer_load_dwordx4 v[176:179], v240, s[40:43], s50 offen offset:0 sc0 sc1\n"
    "v_mfma_i32_16x16x64_i8 v[4:7], v[196:199], v[128:131], v[4:7]\n" "s_waitcnt lgkmcnt(13)\n" "v_mfma_i32_16x16x64_i8 v[8:11], v[200:203], v[128:131], v[8:11]\n"
    "ds_read_b64_tr_b8 v[196:197], v225 offset:40960\n" "v_mfma_i32_16x16x64_i8 v[40:43], v[200:203], v[132:135], v[40:43]\n" "ds_read_b64_tr_b8 v[198:199], v225 offset:41984\n"
    "v_mfma_i32_16x16x64_i8 v[72:75], v[200:203], v[136:139], v[72:75]\n" "v_mfma_i32_16x16x64_i8 v[104:107], v[200:203], v[140:143], v[104:107]\n" "s_waitcnt lgkmcnt(13)\n"
    "v_mfma_i32_16x16x64_i8 v[108:111], v[204:207], v[140:143], v[108:111]\n" "ds_read_b64_tr_b8 v[200:201], v226 offset:40960\n" "v_mfma_i32_16x16x64_i8 v[76:79], v[204:207], v[136:139], v[76:79]\n"
    "ds_read_b64_tr_b8 v[202:203], v226 offset:41984\n" "v_mfma_i32_16x16x64_i8 v[44:47], v[204:207], v[132:135], v[44:47]\n"
    "buffer_load_dwordx4 v[180:183], v240, s[40:43], s50 offen offset:1024 sc0 sc1\n" "ds_read_b32 v244, v241\n" "v_mfma_i32_16x16x64_i8 v[12:15], v[204:207], v[128:131], v[12:15]\n"
    "s_waitcnt lgkmcnt(14)\n" "v_mfma_i32_16x16x64_i8 v[16:19], v[208:211], v[128:131], v[16:19]\n" "ds_read_b64_tr_b8 v[204:205], v227 offset:40960\n"
    "v_mfma_i32_16x16x64_i8 v[48:51], v[208:211], v[132:135], v[48:51]\n" "ds_read_b64_tr_b8 v[206:207], v227 offset:41984\n" "v_mfma_i32_16x16x64_i8 v[80:83], v[208:211], v[136:139], v[80:83]\n"
    "v_mfma_i32_16x16x64_i8 v[112:115], v[208:211], v[140:143], v[112:115]\n" "s_waitcnt lgkmcnt(14)\n" "v_mfma_i32_16x16x64_i8 v[116:119], v[212:215], v[140:143], v[116:119]\n"
    "ds_read_b64_tr_b8 v[208:209], v228 offset:40960\n" "v_mfma_i32_16x16x64_i8 v[84:87], v[212:215], v[136:139], v[84:87]\n" "ds_read_b64_tr_b8 v[210:211], v228 offset:41984\n"
    "v_mfma_i32_16x16x64_i8 v[52:55], v[212:215], v[132:135], v[52:55]\n" "buffer_load_dwordx4 v[184:187], v240, s[40:43], s50 offen offset:2048 sc0 sc1\n"
    "v_mfma_i32_16x16x64_i8 v[20:23], v[212:215], v[128:131], v[20:23]\n" "s_waitcnt lgkmcnt(14)\n" "v_mfma_i32_16x16x64_i8 v[24:27], v[216:219], v[128:131], v[24:27]\n"
    "ds_read_b64_tr_b8 v[212:213], v229 offset:40960\n" "v_mfma_i32_16x16x64_i8 v[56:59], v[216:219], v[132:135], v[56:59]\n" "ds_read_b64_tr_b8 v[214:215], v229 offset:41984\n"
    "v_mfma_i32_16x16x64_i8 v[88:91], v[216:219], v[136:139], v[88:91]\n" "v_mfma_i32_16x16x64_i8 v[120:123], v[216:219], v[140:143], v[120:123]\n" "s_waitcnt lgkmcnt(14)\n"
    "v_mfma_i32_16x16x64_i8 v[124:127], v[220:223], v[140:143], v[124:127]\n" "ds_read_b64_tr_b8 v[216:217], v230 offset:40960\n" "v_mfma_i32_16x16x64_i8 v[92:95], v[220:223], v[136:139], v[92:95]\n"
    "ds_read_b64_tr_b8 v[218:219], v230 offset:41984\n" "v_mfma_i32_16x16x64_i8 v[60:63], v[220:223], v[132:135], v[60:63]\n"
    "buffer_load_dwordx4 v[188:191], v240, s[40:43], s50 offen offset:3072 sc0 sc1\n" "s_add_u32 s50, s50, 0x8000\n" "s_waitcnt lgkmcnt(8)\n" "v_readfirstlane_b32 s51, v244\n"
    "v_mfma_i32_16x16x64_i8 v[28:31], v[220:223], v[128:131], v[28:31]\n" "s_andn2_b32 s55, 0x1000, s51\n" "s_cbranch_scc0 L30_%=\n" "L29_%=:\n" "ds_read_b32 v251, v241 offset:0\n"
    "s_waitcnt lgkmcnt(0)\n" "v_readfirstlane_b32 s51, v251\n" "s_nop 3\n" "s_andn2_b32 s55, 0x1000, s51\n" "s_cbranch_scc0 L30_%=\n" "s_sleep 1\n" "s_branch L29_%=\n" "L30_%=:\n"
    "s_waitcnt vmcnt(11)\n" "v_mfma_i32_16x16x64_i8 v[0:3], v[192:195], v[144:147], v[0:3]\n" "ds_read_b64_tr_b8 v[220:221], v231 offset:40960\n" "s_waitcnt vmcnt(10)\n"
    "v_mfma_i32_16x16x64_i8 v[32:35], v[192:195], v[148:151], v[32:35]\n" "ds_read_b64_tr_b8 v[222:223], v231 offset:41984\n" "s_waitcnt vmcnt(9)\n"
    "v_mfma_i32_16x16x64_i8 v[64:67], v[192:195], v[152:155], v[64:67]\n" "s_waitcnt vmcnt(8)\n" "v_mfma_i32_16x16x64_i8 v[96:99], v[192:195], v[156:159], v[96:99]\n"
    "v_mfma_i32_16x16x64_i8 v[100:103], v[196:199], v[156:159], v[100:103]\n" "ds_read_b64_tr_b8 v[192:193], v224 offset:49152\n" "v_mfma_i32_16x16x64_i8 v[68:71], v[196:199], v[152:155], v[68:71]\n"
    "ds_read_b64_tr_b8 v[194:195], v224 offset:50176\n" "v_mfma_i32_16x16x64_i8 v[36:39], v[196:199], v[148:151], v[36:39]\n" "s_mov_b32 s50, 0x180000\n"
    "buffer_load_dwordx4 v[128:131], v240, s[40:43], s50 offen offset:0 sc0 sc1\n" "v_mfma_i32_16x16x64_i8 v[4:7], v[196:199], v[144:147], v[4:7]\n"
    "v_mfma_i32_16x16x64_i8 v[8:11], v[200:203], v[144:147], v[8:11]\n" "ds_read_b64_tr_b8 v[196:197], v225 offset:49152\n" "v_mfma_i32_16x16x64_i8 v[40:43], v[200:203], v[148:151], v[40:43]\n"
    "ds_read_b64_tr_b8 v[198:199], v225 offset:50176\n" "v_mfma_i32_16x16x64_i8 v[72:75], v[200:203], v[152:155], v[72:75]\n" "v_mfma_i32_16x16x64_i8 v[104:107], v[200:203], v[156:159], v[104:107]\n"
    "s_waitcnt lgkmcnt(12)\n" "v_mfma_i32_16x16x64_i8 v[108:111], v[204:207], v[156:159], v[108:111]\n" "ds_read_b64_tr_b8 v[200:201], v226 offset:49152\n"
    "v_mfma_i32_16x16x64_i8 v[76:79], v[204:207], v[152:155], v[76:79]\n" "ds_read_b64_tr_b8 v[202:203], v226 offset:50176\n" "v_mfma_i32_16x16x64_i8 v[44:47], v[204:207], v[148:151], v[44:47]\n"
    "buffer_load_dwordx4 v[132:135], v240, s[40:43], s50 offen offset:1024 sc0 sc1\n" "v_mfma_i32_16x16x64_i8 v[12:15], v[204:207], v[144:147], v[12:15]\n" "s_waitcnt lgkmcnt(12)\n"
    "v_mfma_i32_16x16x64_i8 v[16:19], v[208:211], v[144:147], v[16:19]\n" "ds_read_b64_tr_b8 v[204:205], v227 offset:49152\n" "v_mfma_i32_16x16x64_i8 v[48:51], v[208:211], v[148:151], v[48:51]\n"
    "ds_read_b64_tr_b8 v[206:207], v227 offset:50176\n" "v_mfma_i32_16x16x64_i8 v[80:83], v[208:211], v[152:155], v[80:83]\n" "v_mfma_i32_16x16x64_i8 v[112:115], v[208:211], v[156:159], v[112:115]\n"
    "s_waitcnt lgkmcnt(12)\n" "v_mfma_i32_16x16x64_i8 v[116:119], v[212:215], v[156:159], v[116:119]\n" "ds_read_b64_tr_b8 v[208:209], v228 offset:49152\n"
    "v_mfma_i32_16x16x64_i8 v[84:87], v[212:215], v[152:155], v[84:87]\n" "ds_read_b64_tr_b8 v[210:211], v228 offset:50176\n" "v_mfma_i32_16x16x64_i8 v[52:55], v[212:215], v[148:151], v[52:55]\n"
    "s_mov_b32 s50, 0x188000\n" "buffer_load_dwordx4 v[136:139], v240, s[40:43], s50 offen offset:0 sc0 sc1\n" "v_mfma_i32_16x16x64_i8 v[20:23], v[212:215], v[144:147], v[20:23]\n"
    "s_waitcnt lgkmcnt(12)\n" "v_mfma_i32_16x16x64_i8 v[24:27], v[216:219], v[144:147], v[24:27]\n" "ds_read_b64_tr_b8 v[212:213], v229 offset:49152\n"
    "v_mfma_i32_16x16x64_i8 v[56:59], v[216:219], v[148:151], v[56:59]\n" "ds_read_b64_tr_b8 v[214:215], v229 offset:50176\n" "v_mfma_i32_16x16x64_i8 v[88:91], v[216:219], v[152:155], v[88:91]\n"
    "v_mfma_i32_16x16x64_i8 v[120:123], v[216:219], v[156:159], v[120:123]\n" "s_waitcnt lgkmcnt(12)\n" "v_mfma_i32_16x16x64_i8 v[124:127], v[220:223], v[156:159], v[124:127]\n"
    "ds_read_b64_tr_b8 v[216:217], v230 offset:49152\n" "v_mfma_i32_16x16x64_i8 v[92:95], v[220:223], v[152:155], v[92:95]\n" "ds_read_b64_tr_b8 v[218:219], v230 offset:50176\n"
    "v_mfma_i32_16x16x64_i8 v[60:63], v[220:223], v[148:151], v[60:63]\n" "buffer_load_dwordx4 v[140:143], v240, s[40:43], s50 offen offset:1024 sc0 sc1\n"
    "v_mfma_i32_16x16x64_i8 v[28:31], v[220:223], v[144:147], v[28:31]\n" "s_waitcnt vmcnt(11) lgkmcnt(12)\n" "v_mfma_i32_16x16x64_i8 v[0:3], v[192:195], v[160:163], v[0:3]\n"
    "ds_read_b64_tr_b8 v[220:221], v231 offset:49152\n" "s_waitcnt vmcnt(10)\n" "v_mfma_i32_16x16x64_i8 v[32:35], v[192:195], v[164:167], v[32:35]\n" "ds_read_b64_tr_b8 v[222:223], v231 offset:50176\n"
    "s_waitcnt vmcnt(9)\n" "v_mfma_i32_16x16x64_i8 v[64:67], v[192:195], v[168:171], v[64:67]\n" "s_waitcnt vmcnt(8)\n" "v_mfma_i32_16x16x64_i8 v[96:99], v[192:195], v[172:175], v[96:99]\n"
    "s_waitcnt lgkmcnt(12)\n" "v_mfma_i32_16x16x64_i8 v[100:103], v[196:199], v[172:175], v[100:103]\n" "ds_read_b64_tr_b8 v[192:193], v224 offset:57344\n"
    "v_mfma_i32_16x16x64_i8 v[68:71], v[196:199], v[168:171], v[68:71]\n" "ds_read_b64_tr_b8 v[194:195], v224 offset:58368\n" "v_mfma_i32_16x16x64_i8 v[36:39], v[196:199], v[164:167], v[36:39]\n"
    "s_mov_b32 s50, 0x190000\n" "buffer_load_dwordx4 v[232:235], v240, s[40:43], s50 offen offset:0 sc0 sc1\n" "v_mfma_i32_16x16x64_i8 v[4:7], v[196:199], v[160:163], v[4:7]\n"
    "s_waitcnt lgkmcnt(12)\n" "v_mfma_i32_16x16x64_i8 v[8:11], v[200:203], v[160:163], v[8:11]\n" "ds_read_b64_tr_b8 v[196:197], v225 offset:57344\n"
    "v_mfma_i32_16x16x64_i8 v[40:43], v[200:203], v[164:167], v[40:43]\n" "ds_read_b64_tr_b8 v[198:199], v225 offset:58368\n" "v_mfma_i32_16x16x64_i8 v[72:75], v[200:203], v[168:171], v[72:75]\n"
    "v_mfma_i32_16x16x64_i8 v[104:107], v[200:203], v[172:175], v[104:107]\n" "s_waitcnt lgkmcnt(12)\n" "v_mfma_i32_16x16x64_i8 v[108:111], v[204:207], v[172:175], v[108:111]\n"
    "ds_read_b64_tr_b8 v[200:201], v226 offset:57344\n" "v_mfma_i32_16x16x64_i8 v[76:79], v[204:207], v[168:171], v[76:79]\n" "ds_read_b64_tr_b8 v[202:203], v226 offset:58368\n"
    "v_mfma_i32_16x16x64_i8 v[44:47], v[204:207], v[164:167], v[44:47]\n" "ds_read_b32 v244, v241\n" "buffer_load_dwordx4 v[236:239], v240, s[40:43], s50 offen offset:1024 sc0 sc1\n"
    "v_mfma_i32_16x16x64_i8 v[12:15], v[204:207], v[160:163], v[12:15]\n" "s_waitcnt lgkmcnt(13)\n" "v_mfma_i32_16x16x64_i8 v[16:19], v[208:211], v[160:163], v[16:19]\n"
    "ds_read_b64_tr_b8 v[204:205], v227 offset:57344\n" "v_mfma_i32_16x16x64_i8 v[48:51], v[208:211], v[164:167], v[48:51]\n" "ds_read_b64_tr_b8 v[206:207], v227 offset:58368\n"
    "v_mfma_i32_16x16x64_i8 v[80:83], v[208:211], v[168:171], v[80:83]\n" "v_mfma_i32_16x16x64_i8 v[112:115], v[208:211], v[172:175], v[112:115]\n" "s_waitcnt lgkmcnt(13)\n"
    "v_mfma_i32_16x16x64_i8 v[116:119], v[212:215], v[172:175], v[116:119]\n" "ds_read_b64_tr_b8 v[208:209], v228 offset:57344\n" "v_mfma_i32_16x16x64_i8 v[84:87], v[212:215], v[168:171], v[84:87]\n"
    "ds_read_b64_tr_b8 v[210:211], v228 offset:58368\n" "v_mfma_i32_16x16x64_i8 v[52:55], v[212:215], v[164:167], v[52:55]\n" "s_mov_b32 s50, 0x198000\n"
    "buffer_load_dwordx4 v[252:255], v240, s[40:43], s50 offen offset:0 sc0 sc1\n" "v_mfma_i32_16x16x64_i8 v[20:23], v[212:215], v[160:163], v[20:23]\n" "s_waitcnt lgkmcnt(13)\n"
    "v_mfma_i32_16x16x64_i8 v[24:27], v[216:219], v[160:163], v[24:27]\n" "ds_read_b64_tr_b8 v[212:213], v229 offset:57344\n" "v_mfma_i32_16x16x64_i8 v[56:59], v[216:219], v[164:167], v[56:59]\n"
    "ds_read_b64_tr_b8 v[214:215], v229 offset:58368\n" "v_mfma_i32_16x16x64_i8 v[88:91], v[216:219], v[168:171], v[88:91]\n" "v_mfma_i32_16x16x64_i8 v[120:123], v[216:219], v[172:175], v[120:123]\n"
    "s_waitcnt lgkmcnt(13)\n" "v_mfma_i32_16x16x64_i8 v[124:127], v[220:223], v[172:175], v[124:127]\n" "ds_read_b64_tr_b8 v[216:217], v230 offset:57344\n"
    "v_mfma_i32_16x16x64_i8 v[92:95], v[220:223], v[168:171], v[92:95]\n" "ds_read_b64_tr_b8 v[218:219], v230 offset:58368\n" "v_mfma_i32_16x16x64_i8 v[60:63], v[220:223], v[164:167], v[60:63]\n"
    "s_waitcnt lgkmcnt(8)\n" "v_readfirstlane_b32 s51, v244\n" "buffer_load_dwordx4 v[144:147], v240, s[40:43], s50 offen offset:1024 sc0 sc1\n"
    "v_mfma_i32_16x16x64_i8 v[28:31], v[220:223], v[160:163], v[28:31]\n" "s_andn2_b32 s55, 0x10000000, s51\n" "s_cbranch_scc0 L32_%=\n" "L31_%=:\n" "ds_read_b32 v251, v241 offset:0\n"
    "s_waitcnt lgkmcnt(0)\n" "v_readfirstlane_b32 s51, v251\n" "s_nop 3\n" "s_andn2_b32 s55, 0x10000000, s51\n" "s_cbranch_scc0 L32_%=\n" "s_sleep 1\n" "s_branch L31_%=\n" "L32_%=:\n"
    "s_waitcnt vmcnt(11)\n" "v_mfma_i32_16x16x64_i8 v[0:3], v[192:195], v[176:179], v[0:3]\n" "ds_read_b64_tr_b8 v[220:221], v231 offset:57344\n" "s_waitcnt vmcnt(10)\n"
    "v_mfma_i32_16x16x64_i8 v[32:35], v[192:195], v[180:183], v[32:35]\n" "ds_read_b64_tr_b8 v[222:223], v231 offset:58368\n" "s_waitcnt vmcnt(9)\n"
    "v_mfma_i32_16x16x64_i8 v[64:67], v[192:195], v[184:187], v[64:67]\n" "s_waitcnt vmcnt(8)\n" "v_mfma_i32_16x16x64_i8 v[96:99], v[192:195], v[188:191], v[96:99]\n"
    "v_mfma_i32_16x16x64_i8 v[100:103], v[196:199], v[188:191], v[100:103]\n" "v_xor_b32 v224, 0x10000, v224\n" "ds_read_b64_tr_b8 v[192:193], v224 offset:0\n"
    "v_mfma_i32_16x16x64_i8 v[68:71], v[196:199], v[184:187], v[68:71]\n" "ds_read_b64_tr_b8 v[194:195], v224 offset:1024\n" "v_mfma_i32_16x16x64_i8 v[36:39], v[196:199], v[180:183], v[36:39]\n"
    "s_mov_b32 s50, 0x1a0000\n" "buffer_load_dwordx4 v[148:151], v240, s[40:43], s50 offen offset:0 sc0 sc1\n" "v_mfma_i32_16x16x64_i8 v[4:7], v[196:199], v[176:179], v[4:7]\n"
    "v_mfma_i32_16x16x64_i8 v[8:11], v[200:203], v[176:179], v[8:11]\n" "v_xor_b32 v225, 0x10000, v225\n" "ds_read_b64_tr_b8 v[196:197], v225 offset:0\n"
    "v_mfma_i32_16x16x64_i8 v[40:43], v[200:203], v[180:183], v[40:43]\n" "ds_read_b64_tr_b8 v[198:199], v225 offset:1024\n" "v_mfma_i32_16x16x64_i8 v[72:75], v[200:203], v[184:187], v[72:75]\n"
    "v_mfma_i32_16x16x64_i8 v[104:107], v[200:203], v[188:191], v[104:107]\n" "s_waitcnt lgkmcnt(12)\n" "v_mfma_i32_16x16x64_i8 v[108:111], v[204:207], v[188:191], v[108:111]\n"
    "v_xor_b32 v226, 0x10000, v226\n" "ds_read_b64_tr_b8 v[200:201], v226 offset:0\n" "v_mfma_i32_16x16x64_i8 v[76:79], v[204:207], v[184:187], v[76:79]\n"
    "ds_read_b64_tr_b8 v[202:203], v226 offset:1024\n" "v_mfma_i32_16x16x64_i8 v[44:47], v[204:207], v[180:183], v[44:47]\n"
    "buffer_load_dwordx4 v[152:155], v240, s[40:43], s50 offen offset:1024 sc0 sc1\n" "v_mfma_i32_16x16x64_i8 v[12:15], v[204:207], v[176:179], v[12:15]\n" "s_waitcnt lgkmcnt(12)\n"
    "v_mfma_i32_16x16x64_i8 v[16:19], v[208:211], v[176:179], v[16:19]\n" "v_xor_b32 v227, 0x10000, v227\n" "ds_read_b64_tr_b8 v[204:205], v227 offset:0\n"
    "v_mfma_i32_16x16x64_i8 v[48:51], v[208:211], v[180:183], v[48:51]\n" "ds_read_b64_tr_b8 v[206:207], v227 offset:1024\n" "v_mfma_i32_16x16x64_i8 v[80:83], v[208:211], v[184:187], v[80:83]\n"
    "v_mfma_i32_16x16x64_i8 v[112:115], v[208:211], v[188:191], v[112:115]\n" "s_waitcnt lgkmcnt(12)\n" "v_mfma_i32_16x16x64_i8 v[116:119], v[212:215], v[188:191], v[116:119]\n"
    "v_xor_b32 v228, 0x10000, v228\n" "ds_read_b64_tr_b8 v[208:209], v228 offset:0\n" "v_mfma_i32_16x16x64_i8 v[84:87], v[212:215], v[184:187], v[84:87]\n"
    "ds_read_b64_tr_b8 v[210:211], v228 offset:1024\n" "v_mfma_i32_16x16x64_i8 v[52:55], v[212:215], v[180:183], v[52:55]\n" "s_mov_b32 s50, 0x1a8000\n"
    "buffer_load_dwordx4 v[156:159], v240, s[40:43], s50 offen offset:0 sc0 sc1\n" "v_mfma_i32_16x16x64_i8 v[20:23], v[212:215], v[176:179], v[20:23]\n" "s_waitcnt lgkmcnt(12)\n"
    "v_mfma_i32_16x16x64_i8 v[24:27], v[216:219], v[176:179], v[24:27]\n" "v_xor_b32 v229, 0x10000, v229\n" "ds_read_b64_tr_b8 v[212:213], v229 offset:0\n"
    "v_mfma_i32_16x16x64_i8 v[56:59], v[216:219], v[180:183], v[56:59]\n" "ds_read_b64_tr_b8 v[214:215], v229 offset:1024\n" "v_mfma_i32_16x16x64_i8 v[88:91], v[216:219], v[184:187], v[88:91]\n"
    "v_mfma_i32_16x16x64_i8 v[120:123], v[216:219], v[188:191], v[120:123]\n" "s_waitcnt lgkmcnt(12)\n" "v_mfma_i32_16x16x64_i8 v[124:127], v[220:223], v[188:191], v[124:127]\n"
    "v_xor_b32 v230, 0x10000, v230\n" "ds_read_b64_tr_b8 v[216:217], v230 offset:0\n" "v_mfma_i32_16x16x64_i8 v[92:95], v[220:223], v[184:187], v[92:95]\n"
    "ds_read_b64_tr_b8 v[218:219], v230 offset:1024\n" "v_mfma_i32_16x16x64_i8 v[60:63], v[220:223], v[180:183], v[60:63]\n"
    "buffer_load_dwordx4 v[160:163], v240, s[40:43], s50 offen offset:1024 sc0 sc1\n" "v_mfma_i32_16x16x64_i8 v[28:31], v[220:223], v[176:179], v[28:31]\n" "s_waitcnt vmcnt(11) lgkmcnt(12)\n"
    "v_mfma_i32_16x16x64_i8 v[0:3], v[192:195], v[128:131], v[0:3]\n" "v_xor_b32 v231, 0x10000, v231\n" "ds_read_b64_tr_b8 v[220:221], v231 offset:0\n" "ds_write_b32 v242, v243\n"
    "v_add_u32 v243, 1, v243\n" "s_waitcnt vmcnt(10)\n" "v_mfma_i32_16x16x64_i8 v[32:35], v[192:195], v[132:135], v[32:35]\n" "ds_read_b64_tr_b8 v[222:223], v231 offset:1024\n"
    "s_waitcnt lgkmcnt(13)\n" "v_mfma_i32_16x16x64_i8 v[36:39], v[196:199], v[132:135], v[36:39]\n" "ds_read_b64_tr_b8 v[192:193], v224 offset:8192\n" "s_mov_b32 s50, 0x1b0000\n"
    "buffer_load_dwordx4 v[164:167], v240, s[40:43], s50 offen offset:0 sc0 sc1\n" "v_mfma_i32_16x16x64_i8 v[4:7], v[196:199], v[128:131], v[4:7]\n" "ds_read_b64_tr_b8 v[194:195], v224 offset:9216\n"
    "s_waitcnt lgkmcnt(13)\n" "v_mfma_i32_16x16x64_i8 v[8:11], v[200:203], v[128:131], v[8:11]\n" "ds_read_b64_tr_b8 v[196:197], v225 offset:8192\n"
    "v_mfma_i32_16x16x64_i8 v[40:43], v[200:203], v[132:135], v[40:43]\n" "ds_read_b64_tr_b8 v[198:199], v225 offset:9216\n" "s_waitcnt lgkmcnt(13)\n"
    "v_mfma_i32_16x16x64_i8 v[44:47], v[204:207], v[132:135], v[44:47]\n" "ds_read_b64_tr_b8 v[200:201], v226 offset:8192\n"
    "buffer_load_dwordx4 v[168:171], v240, s[40:43], s50 offen offset:1024 sc0 sc1\n" "v_mfma_i32_16x16x64_i8 v[12:15], v[204:207], v[128:131], v[12:15]\n"
    "ds_read_b64_tr_b8 v[202:203], v226 offset:9216\n" "s_waitcnt lgkmcnt(13)\n" "v_mfma_i32_16x16x64_i8 v[16:19], v[208:211], v[128:131], v[16:19]\n" "ds_read_b64_tr_b8 v[204:205], v227 offset:8192\n"
    "v_mfma_i32_16x16x64_i8 v[48:51], v[208:211], v[132:135], v[48:51]\n" "ds_read_b64_tr_b8 v[206:207], v227 offset:9216\n" "s_waitcnt lgkmcnt(13)\n"
    "v_mfma_i32_16x16x64_i8 v[52:55], v[212:215], v[132:135], v[52:55]\n" "ds_read_b64_tr_b8 v[208:209], v228 offset:8192\n" "s_mov_b32 s50, 0x1b8000\n"
    "buffer_load_dwordx4 v[172:175], v240, s[40:43], s50 offen offset:0 sc0 sc1\n" "v_mfma_i32_16x16x64_i8 v[20:23], v[212:215], v[128:131], v[20:23]\n"
    "ds_read_b64_tr_b8 v[210:211], v228 offset:9216\n" "s_waitcnt lgkmcnt(13)\n" "v_mfma_i32_16x16x64_i8 v[24:27], v[216:219], v[128:131], v[24:27]\n" "ds_read_b64_tr_b8 v[212:213], v229 offset:8192\n"
    "v_mfma_i32_16x16x64_i8 v[56:59], v[216:219], v[132:135], v[56:59]\n" "ds_read_b64_tr_b8 v[214:215], v229 offset:9216\n" "s_waitcnt lgkmcnt(12)\n"
    "v_mfma_i32_16x16x64_i8 v[60:63], v[220:223], v[132:135], v[60:63]\n" "ds_read_b64_tr_b8 v[216:217], v230 offset:8192\n"
    "buffer_load_dwordx4 v[176:179], v240, s[40:43], s50 offen offset:1024 sc0 sc1\n" "v_mfma_i32_16x16x64_i8 v[28:31], v[220:223], v[128:131], v[28:31]\n"
    "ds_read_b64_tr_b8 v[218:219], v230 offset:9216\n" "s_waitcnt vmcnt(13) lgkmcnt(12)\n" "v_mfma_i32_16x16x64_i8 v[0:3], v[192:195], v[136:139], v[0:3]\n"
    "ds_read_b64_tr_b8 v[220:221], v231 offset:8192\n" "s_waitcnt vmcnt(12)\n" "v_mfma_i32_16x16x64_i8 v[32:35], v[192:195], v[140:143], v[32:35]\n" "ds_read_b64_tr_b8 v[222:223], v231 offset:9216\n"
    "s_waitcnt lgkmcnt(12)\n" "v_mfma_i32_16x16x64_i8 v[36:39], v[196:199], v[140:143], v[36:39]\n" "ds_read_b64_tr_b8 v[192:193], v224 offset:16384\n" "s_mov_b32 s50, 0x1c0000\n"
    "buffer_load_dwordx4 v[180:183], v240, s[40:43], s50 offen offset:0 sc0 sc1\n" "v_mfma_i32_16x16x64_i8 v[4:7], v[196:199], v[136:139], v[4:7]\n" "ds_read_b64_tr_b8 v[194:195], v224 offset:17408\n"
    "s_waitcnt lgkmcnt(12)\n" "v_mfma_i32_16x16x64_i8 v[8:11], v[200:203], v[136:139], v[8:11]\n" "ds_read_b64_tr_b8 v[196:197], v225 offset:16384\n"
    "v_mfma_i32_16x16x64_i8 v[40:43], v[200:203], v[140:143], v[40:43]\n" "ds_read_b64_tr_b8 v[198:199], v225 offset:17408\n" "s_waitcnt lgkmcnt(12)\n"
    "v_mfma_i32_16x16x64_i8 v[44:47], v[204:207], v[140:143], v[44:47]\n" "ds_read_b64_tr_b8 v[200:201], v226 offset:16384\n"
    "buffer_load_dwordx4 v[184:187], v240, s[40:43], s50 offen offset:1024 sc0 sc1\n" "v_mfma_i32_16x16x64_i8 v[12:15], v[204:207], v[136:139], v[12:15]\n"
    "ds_read_b64_tr_b8 v[202:203], v226 offset:17408\n" "s_waitcnt lgkmcnt(12)\n" "v_mfma_i32_16x16x64_i8 v[16:19], v[208:211], v[136:139], v[16:19]\n"
    "ds_read_b64_tr_b8 v[204:205], v227 offset:16384\n" "v_mfma_i32_16x16x64_i8 v[48:51], v[208:211], v[140:143], v[48:51]\n" "ds_read_b64_tr_b8 v[206:207], v227 offset:17408\n"
    "s_waitcnt lgkmcnt(12)\n" "v_mfma_i32_16x16x64_i8 v[52:55], v[212:215], v[140:143], v[52:55]\n" "ds_read_b64_tr_b8 v[208:209], v228 offset:16384\n" "s_mov_b32 s50, 0x1c8000\n"
    "buffer_load_dwordx4 v[188:191], v240, s[40:43], s50 offen offset:0 sc0 sc1\n" "v_mfma_i32_16x16x64_i8 v[20:23], v[212:215], v[136:139], v[20:23]\n"
    "ds_read_b64_tr_b8 v[210:211], v228 offset:17408\n" "s_waitcnt lgkmcnt(12)\n" "v_mfma_i32_16x16x64_i8 v[24:27], v[216:219], v[136:139], v[24:27]\n"
    "ds_read_b64_tr_b8 v[212:213], v229 offset:16384\n" "v_mfma_i32_16x16x64_i8 v[56:59], v[216:219], v[140:143], v[56:59]\n" "ds_read_b64_tr_b8 v[214:215], v229 offset:17408\n"
    "s_waitcnt lgkmcnt(12)\n" "v_mfma_i32_16x16x64_i8 v[60:63], v[220:223], v[140:143], v[60:63]\n" "ds_read_b64_tr_b8 v[216:217], v230 offset:16384\n"
    "buffer_load_dwordx4 v[128:131], v240, s[40:43], s50 offen offset:1024 sc0 sc1\n" "v_mfma_i32_16x16x64_i8 v[28:31], v[220:223], v[136:139], v[28:31]\n"
    "ds_read_b64_tr_b8 v[218:219], v230 offset:17408\n" "s_waitcnt vmcnt(15) lgkmcnt(12)\n" "v_mfma_i32_16x16x64_i8 v[0:3], v[192:195], v[232:235], v[0:3]\n"
    "ds_read_b64_tr_b8 v[220:221], v231 offset:16384\n" "s_waitcnt vmcnt(14)\n" "v_mfma_i32_16x16x64_i8 v[32:35], v[192:195], v[236:239], v[32:35]\n" "ds_read_b64_tr_b8 v[222:223], v231 offset:17408\n"
    "s_waitcnt lgkmcnt(12)\n" "v_mfma_i32_16x16x64_i8 v[36:39], v[196:199], v[236:239], v[36:39]\n" "ds_read_b64_tr_b8 v[192:193], v224 offset:24576\n" "s_mov_b32 s50, 0x1d0000\n"
    "buffer_load_dwordx4 v[132:135], v240, s[40:43], s50 offen offset:0 sc0 sc1\n" "v_mfma_i32_16x16x64_i8 v[4:7], v[196:199], v[232:235], v[4:7]\n" "ds_read_b64_tr_b8 v[194:195], v224 offset:25600\n"
    "s_waitcnt lgkmcnt(12)\n" "v_mfma_i32_16x16x64_i8 v[8:11], v[200:203], v[232:235], v[8:11]\n" "ds_read_b64_tr_b8 v[196:197], v225 offset:24576\n"
    "v_mfma_i32_16x16x64_i8 v[40:43], v[200:203], v[236:239], v[40:43]\n" "ds_read_b64_tr_b8 v[198:199], v225 offset:25600\n" "s_waitcnt lgkmcnt(12)\n"
    "v_mfma_i32_16x16x64_i8 v[44:47], v[204:207], v[236:239], v[44:47]\n" "ds_read_b64_tr_b8 v[200:201], v226 offset:24576\n" "ds_read_b32 v244, v241\n"
    "buffer_load_dwordx4 v[136:139], v240, s[40:43], s50 offen offset:1024 sc0 sc1\n" "v_mfma_i32_16x16x64_i8 v[12:15], v[204:207], v[232:235], v[12:15]\n"
    "ds_read_b64_tr_b8 v[202:203], v226 offset:25600\n" "s_waitcnt lgkmcnt(13)\n" "v_mfma_i32_16x16x64_i8 v[16:19], v[208:211], v[232:235], v[16:19]\n"
    "ds_read_b64_tr_b8 v[204:205], v227 offset:24576\n" "v_mfma_i32_16x16x64_i8 v[48:51], v[208:211], v[236:239], v[48:51]\n" "ds_read_b64_tr_b8 v[206:207], v227 offset:25600\n"
    "s_waitcnt lgkmcnt(13)\n" "v_mfma_i32_16x16x64_i8 v[52:55], v[212:215], v[236:239], v[52:55]\n" "ds_read_b64_tr_b8 v[208:209], v228 offset:24576\n" "s_mov_b32 s50, 0x1d8000\n"
    "buffer_load_dwordx4 v[140:143], v240, s[40:43], s50 offen offset:0 sc0 sc1\n" "v_mfma_i32_16x16x64_i8 v[20:23], v[212:215], v[232:235], v[20:23]\n"
    "ds_read_b64_tr_b8 v[210:211], v228 offset:25600\n" "s_waitcnt lgkmcnt(13)\n" "v_mfma_i32_16x16x64_i8 v[24:27], v[216:219], v[232:235], v[24:27]\n"
    "ds_read_b64_tr_b8 v[212:213], v229 offset:24576\n" "v_mfma_i32_16x16x64_i8 v[56:59], v[216:219], v[236:239], v[56:59]\n" "ds_read_b64_tr_b8 v[214:215], v229 offset:25600\n"
    "s_waitcnt lgkmcnt(13)\n" "v_mfma_i32_16x16x64_i8 v[60:63], v[220:223], v[236:239], v[60:63]\n" "ds_read_b64_tr_b8 v[216:217], v230 offset:24576\n" "s_waitcnt lgkmcnt(8)\n"
    "v_readfirstlane_b32 s51, v244\n" "v_mfma_i32_16x16x64_i8 v[28:31], v[220:223], v[232:235], v[28:31]\n" "ds_read_b64_tr_b8 v[218:219], v230 offset:25600\n" "s_waitcnt vmcnt(16)\n"
    "v_mfma_i32_16x16x64_i8 v[0:3], v[192:195], v[252:255], v[0:3]\n" "ds_read_b64_tr_b8 v[220:221], v231 offset:24576\n"
    "buffer_load_dwordx4 v[232:235], v240, s[40:43], s50 offen offset:1024 sc0 sc1\n" "s_waitcnt vmcnt(16)\n" "v_mfma_i32_16x16x64_i8 v[32:35], v[192:195], v[144:147], v[32:35]\n"
    "ds_read_b64_tr_b8 v[222:223], v231 offset:25600\n" "s_andn2_b32 s55, 0x20000000, s51\n" "s_cbranch_scc0 L34_%=\n" "L33_%=:\n" "ds_read_b32 v251, v241 offset:0\n" "s_waitcnt lgkmcnt(0)\n"
    "v_readfirstlane_b32 s51, v251\n" "s_nop 3\n" "s_andn2_b32 s55, 0x20000000, s51\n" "s_cbranch_scc0 L34_%=\n" "s_sleep 1\n" "s_branch L33_%=\n" "L34_%=:\n"
    "v_mfma_i32_16x16x64_i8 v[36:39], v[196:199], v[144:147], v[36:39]\n" "ds_read_b64_tr_b8 v[192:193], v224 offset:32768\n" "v_mfma_i32_16x16x64_i8 v[4:7], v[196:199], v[252:255], v[4:7]\n"
    "ds_read_b64_tr_b8 v[194:195], v224 offset:33792\n" "s_waitcnt lgkmcnt(12)\n" "v_mfma_i32_16x16x64_i8 v[8:11], v[200:203], v[252:255], v[8:11]\n" "ds_read_b64_tr_b8 v[196:197], v225 offset:32768\n"
    "s_mov_b32 s50, 0x1e0000\n" "buffer_load_dwordx4 v[236:239], v240, s[40:43], s50 offen offset:0 sc0 sc1\n" "v_mfma_i32_16x16x64_i8 v[40:43], v[200:203], v[144:147], v[40:43]\n"
    "ds_read_b64_tr_b8 v[198:199], v225 offset:33792\n" "s_waitcnt lgkmcnt(12)\n" "v_mfma_i32_16x16x64_i8 v[44:47], v[204:207], v[144:147], v[44:47]\n"
    "ds_read_b64_tr_b8 v[200:201], v226 offset:32768\n" "v_mfma_i32_16x16x64_i8 v[12:15], v[204:207], v[252:255], v[12:15]\n" "ds_read_b64_tr_b8 v[202:203], v226 offset:33792\n"
    "s_waitcnt lgkmcnt(12)\n" "v_mfma_i32_16x16x64_i8 v[16:19], v[208:211], v[252:255], v[16:19]\n" "ds_read_b64_tr_b8 v[204:205], v227 offset:32768\n"
    "v_mfma_i32_16x16x64_i8 v[48:51], v[208:211], v[144:147], v[48:51]\n" "ds_read_b64_tr_b8 v[206:207], v227 offset:33792\n" "s_waitcnt lgkmcnt(12)\n"
    "v_mfma_i32_16x16x64_i8 v[52:55], v[212:215], v[144:147], v[52:55]\n" "ds_read_b64_tr_b8 v[208:209], v228 offset:32768\n" "v_mfma_i32_16x16x64_i8 v[20:23], v[212:215], v[252:255], v[20:23]\n"
    "ds_read_b64_tr_b8 v[210:211], v228 offset:33792\n" "s_waitcnt lgkmcnt(12)\n" "v_mfma_i32_16x16x64_i8 v[24:27], v[216:219], v[252:255], v[24:27]\n"
    "ds_read_b64_tr_b8 v[212:213], v229 offset:32768\n" "v_mfma_i32_16x16x64_i8 v[56:59], v[216:219], v[144:147], v[56:59]\n" "ds_read_b64_tr_b8 v[214:215], v229 offset:33792\n"
    "s_waitcnt lgkmcnt(12)\n" "v_mfma_i32_16x16x64_i8 v[60:63], v[220:223], v[144:147], v[60:63]\n" "ds_read_b64_tr_b8 v[216:217], v230 offset:32768\n"
    "v_mfma_i32_16x16x64_i8 v[28:31], v[220:223], v[252:255], v[28:31]\n" "ds_read_b64_tr_b8 v[218:219], v230 offset:33792\n" "s_waitcnt vmcnt(16) lgkmcnt(12)\n"
    "v_mfma_i32_16x16x64_i8 v[0:3], v[192:195], v[148:151], v[0:3]\n" "ds_read_b64_tr_b8 v[220:221], v231 offset:32768\n"
    "buffer_load_dwordx4 v[144:147], v240, s[40:43], s50 offen offset:1024 sc0 sc1\n" "s_waitcnt vmcnt(16)\n" "v_mfma_i32_16x16x64_i8 v[32:35], v[192:195], v[152:155], v[32:35]\n"
    "ds_read_b64_tr_b8 v[222:223], v231 offset:33792\n" "s_waitcnt lgkmcnt(12)\n" "v_mfma_i32_16x16x64_i8 v[36:39], v[196:199], v[152:155], v[36:39]\n"
    "ds_read_b64_tr_b8 v[192:193], v224 offset:40960\n" "v_mfma_i32_16x16x64_i8 v[4:7], v[196:199], v[148:151], v[4:7]\n" "ds_read_b64_tr_b8 v[194:195], v224 offset:41984\n" "s_waitcnt lgkmcnt(12)\n"
    "v_mfma_i32_16x16x64_i8 v[8:11], v[200:203], v[148:151], v[8:11]\n" "ds_read_b64_tr_b8 v[196:197], v225 offset:40960\n" "s_mov_b32 s50, 0x1e8000\n"
    "buffer_load_dwordx4 v[252:255], v240, s[40:43], s50 offen offset:0 sc0 sc1\n" "v_mfma_i32_16x16x64_i8 v[40:43], v[200:203], v[152:155], v[40:43]\n"
    "ds_read_b64_tr_b8 v[198:199], v225 offset:41984\n" "s_waitcnt lgkmcnt(12)\n" "v_mfma_i32_16x16x64_i8 v[44:47], v[204:207], v[152:155], v[44:47]\n"
    "ds_read_b64_tr_b8 v[200:201], v226 offset:40960\n" "v_mfma_i32_16x16x64_i8 v[12:15], v[204:207], v[148:151], v[12:15]\n" "ds_read_b64_tr_b8 v[202:203], v226 offset:41984\n"
    "s_waitcnt lgkmcnt(12)\n" "v_mfma_i32_16x16x64_i8 v[16:19], v[208:211], v[148:151], v[16:19]\n" "ds_read_b64_tr_b8 v[204:205], v227 offset:40960\n"
    "v_mfma_i32_16x16x64_i8 v[48:51], v[208:211], v[152:155], v[48:51]\n" "ds_read_b64_tr_b8 v[206:207], v227 offset:41984\n" "s_waitcnt lgkmcnt(12)\n"
    "v_mfma_i32_16x16x64_i8 v[52:55], v[212:215], v[152:155], v[52:55]\n" "ds_read_b64_tr_b8 v[208:209], v228 offset:40960\n" "v_mfma_i32_16x16x64_i8 v[20:23], v[212:215], v[148:151], v[20:23]\n"
    "ds_read_b64_tr_b8 v[210:211], v228 offset:41984\n" "s_waitcnt lgkmcnt(12)\n" "v_mfma_i32_16x16x64_i8 v[24:27], v[216:219], v[148:151], v[24:27]\n"
    "ds_read_b64_tr_b8 v[212:213], v229 offset:40960\n" "v_mfma_i32_16x16x64_i8 v[56:59], v[216:219], v[152:155], v[56:59]\n" "ds_read_b64_tr_b8 v[214:215], v229 offset:41984\n"
    "s_waitcnt lgkmcnt(12)\n" "v_mfma_i32_16x16x64_i8 v[60:63], v[220:223], v[152:155], v[60:63]\n" "ds_read_b64_tr_b8 v[216:217], v230 offset:40960\n"
    "v_mfma_i32_16x16x64_i8 v[28:31], v[220:223], v[148:151], v[28:31]\n" "ds_read_b64_tr_b8 v[218:219], v230 offset:41984\n" "s_waitcnt vmcnt(16) lgkmcnt(12)\n"
    "v_mfma_i32_16x16x64_i8 v[0:3], v[192:195], v[156:159], v[0:3]\n" "ds_read_b64_tr_b8 v[220:221], v231 offset:40960\n"
    "buffer_load_dwordx4 v[148:151], v240, s[40:43], s50 offen offset:1024 sc0 sc1\n" "s_waitcnt vmcnt(16)\n" "v_mfma_i32_16x16x64_i8 v[32:35], v[192:195], v[160:163], v[32:35]\n"
    "ds_read_b64_tr_b8 v[222:223], v231 offset:41984\n" "s_waitcnt lgkmcnt(12)\n" "v_mfma_i32_16x16x64_i8 v[36:39], v[196:199], v[160:163], v[36:39]\n"
    "ds_read_b64_tr_b8 v[192:193], v224 offset:49152\n" "v_mfma_i32_16x16x64_i8 v[4:7], v[196:199], v[156:159], v[4:7]\n" "ds_read_b64_tr_b8 v[194:195], v224 offset:50176\n" "s_waitcnt lgkmcnt(12)\n"
    "v_mfma_i32_16x16x64_i8 v[8:11], v[200:203], v[156:159], v[8:11]\n" "ds_read_b64_tr_b8 v[196:197], v225 offset:49152\n" "s_mov_b32 s50, 0x1f0000\n"
    "buffer_load_dwordx4 v[152:155], v240, s[40:43], s50 offen offset:0 sc0 sc1\n" "v_mfma_i32_16x16x64_i8 v[40:43], v[200:203], v[160:163], v[40:43]\n"
    "ds_read_b64_tr_b8 v[198:199], v225 offset:50176\n" "s_waitcnt lgkmcnt(12)\n" "v_mfma_i32_16x16x64_i8 v[44:47], v[204:207], v[160:163], v[44:47]\n"
    "ds_read_b64_tr_b8 v[200:201], v226 offset:49152\n" "v_mfma_i32_16x16x64_i8 v[12:15], v[204:207], v[156:159], v[12:15]\n" "ds_read_b64_tr_b8 v[202:203], v226 offset:50176\n"
    "s_waitcnt lgkmcnt(12)\n" "v_mfma_i32_16x16x64_i8 v[16:19], v[208:211], v[156:159], v[16:19]\n" "ds_read_b64_tr_b8 v[204:205], v227 offset:49152\n"
    "v_mfma_i32_16x16x64_i8 v[48:51], v[208:211], v[160:163], v[48:51]\n" "ds_read_b64_tr_b8 v[206:207], v227 offset:50176\n" "s_waitcnt lgkmcnt(12)\n"
    "v_mfma_i32_16x16x64_i8 v[52:55], v[212:215], v[160:163], v[52:55]\n" "ds_read_b64_tr_b8 v[208:209], v228 offset:49152\n" "v_mfma_i32_16x16x64_i8 v[20:23], v[212:215], v[156:159], v[20:23]\n"
    "ds_read_b64_tr_b8 v[210:211], v228 offset:50176\n" "s_waitcnt lgkmcnt(12)\n" "v_mfma_i32_16x16x64_i8 v[24:27], v[216:219], v[156:159], v[24:27]\n"
    "ds_read_b64_tr_b8 v[212:213], v229 offset:49152\n" "v_mfma_i32_16x16x64_i8 v[56:59], v[216:219], v[160:163], v[56:59]\n" "ds_read_b64_tr_b8 v[214:215], v229 offset:50176\n"
    "s_waitcnt lgkmcnt(12)\n" "v_mfma_i32_16x16x64_i8 v[60:63], v[220:223], v[160:163], v[60:63]\n" "ds_read_b64_tr_b8 v[216:217], v230 offset:49152\n"
    "v_mfma_i32_16x16x64_i8 v[28:31], v[220:223], v[156:159], v[28:31]\n" "ds_read_b64_tr_b8 v[218:219], v230 offset:50176\n" "s_waitcnt vmcnt(16) lgkmcnt(12)\n"
    "v_mfma_i32_16x16x64_i8 v[0:3], v[192:195], v[164:167], v[0:3]\n" "ds_read_b64_tr_b8 v[220:221], v231 offset:49152\n"
    "buffer_load_dwordx4 v[156:159], v240, s[40:43], s50 offen offset:1024 sc0 sc1\n" "s_waitcnt vmcnt(16)\n" "v_mfma_i32_16x16x64_i8 v[32:35], v[192:195], v[168:171], v[32:35]\n"
    "ds_read_b64_tr_b8 v[222:223], v231 offset:50176\n" "s_waitcnt lgkmcnt(12)\n" "v_mfma_i32_16x16x64_i8 v[36:39], v[196:199], v[168:171], v[36:39]\n"
    "ds_read_b64_tr_b8 v[192:193], v224 offset:57344\n" "v_mfma_i32_16x16x64_i8 v[4:7], v[196:199], v[164:167], v[4:7]\n" "ds_read_b64_tr_b8 v[194:195], v224 offset:58368\n" "s_waitcnt lgkmcnt(12)\n"
    "v_mfma_i32_16x16x64_i8 v[8:11], v[200:203], v[164:167], v[8:11]\n" "ds_read_b64_tr_b8 v[196:197], v225 offset:57344\n" "s_mov_b32 s50, 0x1f8000\n"
    "buffer_load_dwordx4 v[160:163], v240, s[40:43], s50 offen offset:0 sc0 sc1\n" "v_mfma_i32_16x16x64_i8 v[40:43], v[200:203], v[168:171], v[40:43]\n"
    "ds_read_b64_tr_b8 v[198:199], v225 offset:58368\n" "s_waitcnt lgkmcnt(12)\n" "v_mfma_i32_16x16x64_i8 v[44:47], v[204:207], v[168:171], v[44:47]\n"
    "ds_read_b64_tr_b8 v[200:201], v226 offset:57344\n" "ds_read_b32 v244, v241\n" "v_mfma_i32_16x16x64_i8 v[12:15], v[204:207], v[164:167], v[12:15]\n"
    "ds_read_b64_tr_b8 v[202:203], v226 offset:58368\n" "s_waitcnt lgkmcnt(13)\n" "v_mfma_i32_16x16x64_i8 v[16:19], v[208:211], v[164:167], v[16:19]\n"
    "ds_read_b64_tr_b8 v[204:205], v227 offset:57344\n" "v_mfma_i32_16x16x64_i8 v[48:51], v[208:211], v[168:171], v[48:51]\n" "ds_read_b64_tr_b8 v[206:207], v227 offset:58368\n"
    "s_waitcnt lgkmcnt(13)\n" "v_mfma_i32_16x16x64_i8 v[52:55], v[212:215], v[168:171], v[52:55]\n" "ds_read_b64_tr_b8 v[208:209], v228 offset:57344\n"
    "v_mfma_i32_16x16x64_i8 v[20:23], v[212:215], v[164:167], v[20:23]\n" "ds_read_b64_tr_b8 v[210:211], v228 offset:58368\n" "s_waitcnt lgkmcnt(13)\n"
    "v_mfma_i32_16x16x64_i8 v[24:27], v[216:219], v[164:167], v[24:27]\n" "ds_read_b64_tr_b8 v[212:213], v229 offset:57344\n" "v_mfma_i32_16x16x64_i8 v[56:59], v[216:219], v[168:171], v[56:59]\n"
    "ds_read_b64_tr_b8 v[214:215], v229 offset:58368\n" "s_waitcnt lgkmcnt(13)\n" "v_mfma_i32_16x16x64_i8 v[60:63], v[220:223], v[168:171], v[60:63]\n"
    "ds_read_b64_tr_b8 v[216:217], v230 offset:57344\n" "s_waitcnt lgkmcnt(8)\n" "v_readfirstlane_b32 s51, v244\n" "v_mfma_i32_16x16x64_i8 v[28:31], v[220:223], v[164:167], v[28:31]\n"
    "ds_read_b64_tr_b8 v[218:219], v230 offset:58368\n" "s_waitcnt vmcnt(16)\n" "v_mfma_i32_16x16x64_i8 v[0:3], v[192:195], v[172:175], v[0:3]\n" "ds_read_b64_tr_b8 v[220:221], v231 offset:57344\n"
    "buffer_load_dwordx4 v[164:167], v240, s[40:43], s50 offen offset:1024 sc0 sc1\n" "s_waitcnt vmcnt(16)\n" "v_mfma_i32_16x16x64_i8 v[32:35], v[192:195], v[176:179], v[32:35]\n"
    "ds_read_b64_tr_b8 v[222:223], v231 offset:58368\n" "s_andn2_b32 s55, 0x40000000, s51\n" "s_cbranch_scc0 L36_%=\n" "L35_%=:\n" "ds_read_b32 v251, v241 offset:0\n" "s_waitcnt lgkmcnt(0)\n"
    "v_readfirstlane_b32 s51, v251\n" "s_nop 3\n" "s_andn2_b32 s55, 0x40000000, s51\n" "s_cbranch_scc0 L36_%=\n" "s_sleep 1\n" "s_branch L35_%=\n" "L36_%=:\n"
    "v_mfma_i32_16x16x64_i8 v[36:39], v[196:199], v[176:179], v[36:39]\n" "v_xor_b32 v224, 0x10000, v224\n" "ds_read_b64_tr_b8 v[192:193], v224 offset:0\n"
    "v_mfma_i32_16x16x64_i8 v[4:7], v[196:199], v[172:175], v[4:7]\n" "ds_read_b64_tr_b8 v[194:195], v224 offset:1024\n" "s_waitcnt lgkmcnt(12)\n"
    "v_mfma_i32_16x16x64_i8 v[8:11], v[200:203], v[172:175], v[8:11]\n" "v_xor_b32 v225, 0x10000, v225\n" "ds_read_b64_tr_b8 v[196:197], v225 offset:0\n" "s_mov_b32 s50, 0x180000\n"
    "buffer_load_dwordx4 v[168:171], v240, s[40:43], s50 offen offset:2048 sc0 sc1\n" "v_mfma_i32_16x16x64_i8 v[40:43], v[200:203], v[176:179], v[40:43]\n"
    "ds_read_b64_tr_b8 v[198:199], v225 offset:1024\n" "s_waitcnt lgkmcnt(12)\n" "v_mfma_i32_16x16x64_i8 v[44:47], v[204:207], v[176:179], v[44:47]\n" "ds_read_b64_tr_b8 v[200:201], v224 offset:8192\n"
    "ds_read_b32 v244, v241\n" "v_mfma_i32_16x16x64_i8 v[12:15], v[204:207], v[172:175], v[12:15]\n" "ds_read_b64_tr_b8 v[202:203], v224 offset:9216\n" "s_waitcnt lgkmcnt(13)\n"
    "v_mfma_i32_16x16x64_i8 v[16:19], v[208:211], v[172:175], v[16:19]\n" "ds_read_b64_tr_b8 v[204:205], v225 offset:8192\n" "v_mfma_i32_16x16x64_i8 v[48:51], v[208:211], v[176:179], v[48:51]\n"
    "ds_read_b64_tr_b8 v[206:207], v225 offset:9216\n" "s_waitcnt lgkmcnt(13)\n" "v_mfma_i32_16x16x64_i8 v[52:55], v[212:215], v[176:179], v[52:55]\n"
    "ds_read_b64_tr_b8 v[208:209], v224 offset:16384\n" "v_mfma_i32_16x16x64_i8 v[20:23], v[212:215], v[172:175], v[20:23]\n" "ds_read_b64_tr_b8 v[210:211], v224 offset:17408\n"
    "s_waitcnt lgkmcnt(13)\n" "v_mfma_i32_16x16x64_i8 v[24:27], v[216:219], v[172:175], v[24:27]\n" "ds_read_b64_tr_b8 v[212:213], v225 offset:16384\n"
    "v_mfma_i32_16x16x64_i8 v[56:59], v[216:219], v[176:179], v[56:59]\n" "ds_read_b64_tr_b8 v[214:215], v225 offset:17408\n" "s_waitcnt lgkmcnt(13)\n"
    "v_mfma_i32_16x16x64_i8 v[60:63], v[220:223], v[176:179], v[60:63]\n" "ds_read_b64_tr_b8 v[216:217], v224 offset:24576\n" "s_waitcnt lgkmcnt(8)\n" "v_readfirstlane_b32 s51, v244\n"
    "v_mfma_i32_16x16x64_i8 v[28:31], v[220:223], v[172:175], v[28:31]\n" "ds_read_b64_tr_b8 v[218:219], v224 offset:25600\n" "s_waitcnt vmcnt(16)\n"
    "v_mfma_i32_16x16x64_i8 v[0:3], v[192:195], v[180:183], v[0:3]\n" "ds_read_b64_tr_b8 v[220:221], v225 offset:24576\n"
    "buffer_load_dwordx4 v[172:175], v240, s[40:43], s50 offen offset:3072 sc0 sc1\n" "s_waitcnt vmcnt(16)\n" "v_mfma_i32_16x16x64_i8 v[32:35], v[192:195], v[184:187], v[32:35]\n"
    "ds_read_b64_tr_b8 v[222:223], v225 offset:25600\n" "s_andn2_b32 s55, 0x80000000, s51\n" "s_cbranch_scc0 L38_%=\n" "L37_%=:\n" "ds_read_b32 v251, v241 offset:0\n" "s_waitcnt lgkmcnt(0)\n"
    "v_readfirstlane_b32 s51, v251\n" "s_nop 3\n" "s_andn2_b32 s55, 0x80000000, s51\n" "s_cbranch_scc0 L38_%=\n" "s_sleep 1\n" "s_branch L37_%=\n" "L38_%=:\n"
    "v_mfma_i32_16x16x64_i8 v[36:39], v[196:199], v[184:187], v[36:39]\n" "ds_read_b64_tr_b8 v[192:193], v224 offset:32768\n" "v_mfma_i32_16x16x64_i8 v[4:7], v[196:199], v[180:183], v[4:7]\n"
    "ds_read_b64_tr_b8 v[194:195], v224 offset:33792\n" "s_waitcnt vmcnt(15) lgkmcnt(12)\n" "v_mfma_i32_16x16x64_i8 v[0:3], v[200:203], v[188:191], v[0:3]\n"
    "ds_read_b64_tr_b8 v[196:197], v225 offset:32768\n" "s_mov_b32 s50, 0x188000\n" "buffer_load_dwordx4 v[176:179], v240, s[40:43], s50 offen offset:2048 sc0 sc1\n" "s_waitcnt vmcnt(15)\n"
    "v_mfma_i32_16x16x64_i8 v[32:35], v[200:203], v[128:131], v[32:35]\n" "ds_read_b64_tr_b8 v[198:199], v225 offset:33792\n" "s_waitcnt lgkmcnt(12)\n"
    "v_mfma_i32_16x16x64_i8 v[36:39], v[204:207], v[128:131], v[36:39]\n" "ds_read_b64_tr_b8 v[200:201], v224 offset:40960\n" "v_mfma_i32_16x16x64_i8 v[4:7], v[204:207], v[188:191], v[4:7]\n"
    "ds_read_b64_tr_b8 v[202:203], v224 offset:41984\n" "s_waitcnt vmcnt(14) lgkmcnt(12)\n" "v_mfma_i32_16x16x64_i8 v[0:3], v[208:211], v[132:135], v[0:3]\n"
    "ds_read_b64_tr_b8 v[204:205], v225 offset:40960\n" "s_waitcnt vmcnt(13)\n" "v_mfma_i32_16x16x64_i8 v[32:35], v[208:211], v[136:139], v[32:35]\n" "ds_read_b64_tr_b8 v[206:207], v225 offset:41984\n"
    "s_waitcnt lgkmcnt(12)\n" "v_mfma_i32_16x16x64_i8 v[36:39], v[212:215], v[136:139], v[36:39]\n" "ds_read_b64_tr_b8 v[208:209], v224 offset:49152\n"
    "v_mfma_i32_16x16x64_i8 v[4:7], v[212:215], v[132:135], v[4:7]\n" "ds_read_b64_tr_b8 v[210:211], v224 offset:50176\n" "s_waitcnt vmcnt(12) lgkmcnt(12)\n"
    "v_mfma_i32_16x16x64_i8 v[0:3], v[216:219], v[140:143], v[0:3]\n" "ds_read_b64_tr_b8 v[212:213], v225 offset:49152\n" "s_waitcnt vmcnt(11)\n"
    "v_mfma_i32_16x16x64_i8 v[32:35], v[216:219], v[232:235], v[32:35]\n" "ds_read_b64_tr_b8 v[214:215], v225 offset:50176\n" "s_waitcnt lgkmcnt(12)\n"
    "v_mfma_i32_16x16x64_i8 v[36:39], v[220:223], v[232:235], v[36:39]\n" "ds_read_b64_tr_b8 v[216:217], v224 offset:57344\n" "v_mfma_i32_16x16x64_i8 v[4:7], v[220:223], v[140:143], v[4:7]\n"
    "ds_read_b64_tr_b8 v[218:219], v224 offset:58368\n" "s_waitcnt vmcnt(10) lgkmcnt(12)\n" "v_mfma_i32_16x16x64_i8 v[0:3], v[192:195], v[236:239], v[0:3]\n"
    "ds_read_b64_tr_b8 v[220:221], v225 offset:57344\n" "s_waitcnt vmcnt(9)\n" "v_mfma_i32_16x16x64_i8 v[32:35], v[192:195], v[144:147], v[32:35]\n" "ds_read_b64_tr_b8 v[222:223], v225 offset:58368\n"
    "s_waitcnt lgkmcnt(12)\n" "v_mfma_i32_16x16x64_i8 v[36:39], v[196:199], v[144:147], v[36:39]\n" "v_xor_b32 v226, 0x10000, v226\n" "ds_read_b64_tr_b8 v[192:193], v226 offset:0\n"
    "v_mfma_i32_16x16x64_i8 v[4:7], v[196:199], v[236:239], v[4:7]\n" "ds_read_b64_tr_b8 v[194:195], v226 offset:1024\n" "s_waitcnt vmcnt(8) lgkmcnt(12)\n"
    "v_mfma_i32_16x16x64_i8 v[0:3], v[200:203], v[252:255], v[0:3]\n" "v_xor_b32 v227, 0x10000, v227\n" "ds_read_b64_tr_b8 v[196:197], v227 offset:0\n" "s_waitcnt vmcnt(7)\n"
    "v_mfma_i32_16x16x64_i8 v[32:35], v[200:203], v[148:151], v[32:35]\n" "ds_read_b64_tr_b8 v[198:199], v227 offset:1024\n" "s_waitcnt lgkmcnt(12)\n"
    "v_mfma_i32_16x16x64_i8 v[36:39], v[204:207], v[148:151], v[36:39]\n" "ds_read_b64_tr_b8 v[200:201], v226 offset:8192\n" "v_mfma_i32_16x16x64_i8 v[4:7], v[204:207], v[252:255], v[4:7]\n"
    "ds_read_b64_tr_b8 v[202:203], v226 offset:9216\n" "s_waitcnt vmcnt(6) lgkmcnt(12)\n" "v_mfma_i32_16x16x64_i8 v[0:3], v[208:211], v[152:155], v[0:3]\n"
    "ds_read_b64_tr_b8 v[204:205], v227 offset:8192\n" "s_waitcnt vmcnt(5)\n" "v_mfma_i32_16x16x64_i8 v[32:35], v[208:211], v[156:159], v[32:35]\n" "ds_read_b64_tr_b8 v[206:207], v227 offset:9216\n"
    "s_waitcnt lgkmcnt(12)\n" "v_mfma_i32_16x16x64_i8 v[36:39], v[212:215], v[156:159], v[36:39]\n" "ds_read_b64_tr_b8 v[208:209], v226 offset:16384\n"
    "v_mfma_i32_16x16x64_i8 v[4:7], v[212:215], v[152:155], v[4:7]\n" "ds_read_b64_tr_b8 v[210:211], v226 offset:17408\n" "s_waitcnt vmcnt(4) lgkmcnt(12)\n"
    "v_mfma_i32_16x16x64_i8 v[0:3], v[216:219], v[160:163], v[0:3]\n" "ds_read_b64_tr_b8 v[212:213], v227 offset:16384\n" "s_waitcnt vmcnt(3)\n"
    "v_mfma_i32_16x16x64_i8 v[32:35], v[216:219], v[164:167], v[32:35]\n" "ds_read_b64_tr_b8 v[214:215], v227 offset:17408\n" "s_waitcnt lgkmcnt(12)\n"
    "v_mfma_i32_16x16x64_i8 v[36:39], v[220:223], v[164:167], v[36:39]\n" "ds_read_b64_tr_b8 v[216:217], v226 offset:24576\n" "v_mfma_i32_16x16x64_i8 v[4:7], v[220:223], v[160:163], v[4:7]\n"
    "ds_read_b64_tr_b8 v[218:219], v226 offset:25600\n" "s_waitcnt lgkmcnt(12)\n" "v_mfma_i32_16x16x64_i8 v[8:11], v[192:195], v[180:183], v[8:11]\n" "ds_read_b64_tr_b8 v[220:221], v227 offset:24576\n"
    "v_mfma_i32_16x16x64_i8 v[40:43], v[192:195], v[184:187], v[40:43]\n" "ds_read_b64_tr_b8 v[222:223], v227 offset:25600\n" "s_waitcnt lgkmcnt(12)\n"
    "v_mfma_i32_16x16x64_i8 v[44:47], v[196:199], v[184:187], v[44:47]\n" "ds_read_b64_tr_b8 v[192:193], v226 offset:32768\n" "v_mfma_i32_16x16x64_i8 v[12:15], v[196:199], v[180:183], v[12:15]\n"
    "ds_read_b64_tr_b8 v[194:195], v226 offset:33792\n" "ds_write_b128 v245, v[0:3] offset:0\n" "s_waitcnt lgkmcnt(13)\n" "v_mfma_i32_16x16x64_i8 v[8:11], v[200:203], v[188:191], v[8:11]\n"
    "ds_read_b64_tr_b8 v[196:197], v227 offset:32768\n" "buffer_load_dwordx4 v[0:3], v240, s[40:43], s50 offen offset:3072 sc0 sc1\n"
    "v_mfma_i32_16x16x64_i8 v[40:43], v[200:203], v[128:131], v[40:43]\n" "ds_read_b64_tr_b8 v[198:199], v227 offset:33792\n" "ds_write_b128 v245, v[32:35] offset:2048\n" "s_waitcnt lgkmcnt(14)\n"
    "v_mfma_i32_16x16x64_i8 v[44:47], v[204:207], v[128:131], v[44:47]\n" "ds_read_b64_tr_b8 v[200:201], v226 offset:40960\n" "v_mfma_i32_16x16x64_i8 v[12:15], v[204:207], v[188:191], v[12:15]\n"
    "ds_read_b64_tr_b8 v[202:203], v226 offset:41984\n" "ds_write_b128 v246, v[4:7] offset:0\n" "s_waitcnt lgkmcnt(15)\n" "v_mfma_i32_16x16x64_i8 v[8:11], v[208:211], v[132:135], v[8:11]\n"
    "ds_read_b64_tr_b8 v[204:205], v227 offset:40960\n" "s_mov_b32 s50, 0x190000\n" "buffer_load_dwordx4 v[32:35], v240, s[40:43], s50 offen offset:2048 sc0 sc1\n"
    "v_mfma_i32_16x16x64_i8 v[40:43], v[208:211], v[136:139], v[40:43]\n" "ds_read_b64_tr_b8 v[206:207], v227 offset:41984\n" "ds_write_b128 v246, v[36:39] offset:2048\n" "v_mov_b32 v251, 1\n"
    "ds_write_b32 v242, v251 offset:16\n" "s_waitcnt lgkmcnt(15)\n" "v_mfma_i32_16x16x64_i8 v[44:47], v[212:215], v[136:139], v[44:47]\n" "ds_read_b64_tr_b8 v[208:209], v226 offset:49152\n"
    "v_mfma_i32_16x16x64_i8 v[12:15], v[212:215], v[132:135], v[12:15]\n" "ds_read_b64_tr_b8 v[210:211], v226 offset:50176\n" "v_mfma_i32_16x16x64_i8 v[8:11], v[216:219], v[140:143], v[8:11]\n"
    "ds_read_b64_tr_b8 v[212:213], v227 offset:49152\n" "buffer_load_dwordx4 v[4:7], v240, s[40:43], s50 offen offset:3072 sc0 sc1\n"
    "v_mfma_i32_16x16x64_i8 v[40:43], v[216:219], v[232:235], v[40:43]\n" "ds_read_b64_tr_b8 v[214:215], v227 offset:50176\n" "s_waitcnt lgkmcnt(15)\n"
    "v_mfma_i32_16x16x64_i8 v[44:47], v[220:223], v[232:235], v[44:47]\n" "ds_read_b64_tr_b8 v[216:217], v226 offset:57344\n" "v_mfma_i32_16x16x64_i8 v[12:15], v[220:223], v[140:143], v[12:15]\n"
    "ds_read_b64_tr_b8 v[218:219], v226 offset:58368\n" "v_mfma_i32_16x16x64_i8 v[8:11], v[192:195], v[236:239], v[8:11]\n" "ds_read_b64_tr_b8 v[220:221], v227 offset:57344\n"
    "s_mov_b32 s50, 0x198000\n" "buffer_load_dwordx4 v[36:39], v240, s[40:43], s50 offen offset:2048 sc0 sc1\n" "v_mfma_i32_16x16x64_i8 v[40:43], v[192:195], v[144:147], v[40:43]\n"
    "ds_read_b64_tr_b8 v[222:223], v227 offset:58368\n" "s_waitcnt lgkmcnt(15)\n" "v_mfma_i32_16x16x64_i8 v[44:47], v[196:199], v[144:147], v[44:47]\n" "v_xor_b32 v228, 0x10000, v228\n"
    "ds_read_b64_tr_b8 v[192:193], v228 offset:0\n" "v_mfma_i32_16x16x64_i8 v[12:15], v[196:199], v[236:239], v[12:15]\n" "ds_read_b64_tr_b8 v[194:195], v228 offset:1024\n" "s_waitcnt lgkmcnt(15)\n"
    "v_mfma_i32_16x16x64_i8 v[8:11], v[200:203], v[252:255], v[8:11]\n" "v_xor_b32 v229, 0x10000, v229\n" "ds_read_b64_tr_b8 v[196:197], v229 offset:0\n"
    "v_mfma_i32_16x16x64_i8 v[40:43], v[200:203], v[148:151], v[40:43]\n" "ds_read_b64_tr_b8 v[198:199], v229 offset:1024\n" "s_waitcnt lgkmcnt(14)\n"
    "v_mfma_i32_16x16x64_i8 v[44:47], v[204:207], v[148:151], v[44:47]\n" "ds_read_b64_tr_b8 v[200:201], v228 offset:8192\n" "ds_read_b32 v247, v242 offset:32\n"
    "v_mfma_i32_16x16x64_i8 v[12:15], v[204:207], v[252:255], v[12:15]\n" "ds_read_b64_tr_b8 v[202:203], v228 offset:9216\n" "s_waitcnt lgkmcnt(13)\n"
    "v_mfma_i32_16x16x64_i8 v[8:11], v[208:211], v[152:155], v[8:11]\n" "ds_read_b64_tr_b8 v[204:205], v229 offset:8192\n" "v_mfma_i32_16x16x64_i8 v[40:43], v[208:211], v[156:159], v[40:43]\n"
    "ds_read_b64_tr_b8 v[206:207], v229 offset:9216\n" "s_waitcnt lgkmcnt(13)\n" "v_mfma_i32_16x16x64_i8 v[44:47], v[212:215], v[156:159], v[44:47]\n"
    "ds_read_b64_tr_b8 v[208:209], v228 offset:16384\n" "v_mfma_i32_16x16x64_i8 v[12:15], v[212:215], v[152:155], v[12:15]\n" "ds_read_b64_tr_b8 v[210:211], v228 offset:17408\n"
    "s_waitcnt lgkmcnt(13)\n" "v_mfma_i32_16x16x64_i8 v[8:11], v[216:219], v[160:163], v[8:11]\n" "ds_read_b64_tr_b8 v[212:213], v229 offset:16384\n"
    "v_mfma_i32_16x16x64_i8 v[40:43], v[216:219], v[164:167], v[40:43]\n" "ds_read_b64_tr_b8 v[214:215], v229 offset:17408\n" "s_waitcnt lgkmcnt(13)\n"
    "v_mfma_i32_16x16x64_i8 v[44:47], v[220:223], v[164:167], v[44:47]\n" "ds_read_b64_tr_b8 v[216:217], v228 offset:24576\n" "s_waitcnt lgkmcnt(8)\n" "v_readfirstlane_b32 s54, v247\n"
    "v_mfma_i32_16x16x64_i8 v[12:15], v[220:223], v[160:163], v[12:15]\n" "ds_read_b64_tr_b8 v[218:219], v228 offset:25600\n" "v_mfma_i32_16x16x64_i8 v[16:19], v[192:195], v[180:183], v[16:19]\n"
    "ds_read_b64_tr_b8 v[220:221], v229 offset:24576\n" "v_mfma_i32_16x16x64_i8 v[48:51], v[192:195], v[184:187], v[48:51]\n" "ds_read_b64_tr_b8 v[222:223], v229 offset:25600\n"
    "s_andn2_b32 s55, 1, s54\n" "s_cbranch_scc0 L40_%=\n" "L39_%=:\n" "ds_read_b32 v251, v242 offset:32\n" "s_waitcnt lgkmcnt(0)\n" "v_readfirstlane_b32 s54, v251\n" "s_nop 3\n"
    "s_andn2_b32 s55, 1, s54\n" "s_cbranch_scc0 L40_%=\n" "s_sleep 1\n" "s_branch L39_%=\n" "L40_%=:\n" "v_mfma_i32_16x16x64_i8 v[52:55], v[196:199], v[184:187], v[52:55]\n"
    "ds_read_b64_tr_b8 v[192:193], v228 offset:32768\n" "v_mfma_i32_16x16x64_i8 v[20:23], v[196:199], v[180:183], v[20:23]\n" "ds_read_b64_tr_b8 v[194:195], v228 offset:33792\n"
    "ds_write_b128 v245, v[8:11] offset:0\n" "s_waitcnt lgkmcnt(13)\n" "v_mfma_i32_16x16x64_i8 v[16:19], v[200:203], v[188:191], v[16:19]\n" "ds_read_b64_tr_b8 v[196:197], v229 offset:32768\n"
    "buffer_load_dwordx4 v[8:11], v240, s[40:43], s50 offen offset:3072 sc0 sc1\n" "v_mfma_i32_16x16x64_i8 v[48:51], v[200:203], v[128:131], v[48:51]\n"
    "ds_read_b64_tr_b8 v[198:199], v229 offset:33792\n" "ds_write_b128 v245, v[40:43] offset:2048\n" "s_waitcnt lgkmcnt(14)\n" "v_mfma_i32_16x16x64_i8 v[52:55], v[204:207], v[128:131], v[52:55]\n"
    "ds_read_b64_tr_b8 v[200:201], v228 offset:40960\n" "v_mfma_i32_16x16x64_i8 v[20:23], v[204:207], v[188:191], v[20:23]\n" "ds_read_b64_tr_b8 v[202:203], v228 offset:41984\n"
    "ds_write_b128 v246, v[12:15] offset:0\n" "s_waitcnt lgkmcnt(15)\n" "v_mfma_i32_16x16x64_i8 v[16:19], v[208:211], v[132:135], v[16:19]\n" "ds_read_b64_tr_b8 v[204:205], v229 offset:40960\n"
    "s_mov_b32 s50, 0x1a0000\n" "buffer_load_dwordx4 v[40:43], v240, s[40:43], s50 offen offset:2048 sc0 sc1\n" "v_mfma_i32_16x16x64_i8 v[48:51], v[208:211], v[136:139], v[48:51]\n"
    "ds_read_b64_tr_b8 v[206:207], v229 offset:41984\n" "ds_write_b128 v246, v[44:47] offset:2048\n" "v_mov_b32 v251, 2\n" "ds_write_b32 v242, v251 offset:16\n" "s_waitcnt lgkmcnt(15)\n"
    "v_mfma_i32_16x16x64_i8 v[52:55], v[212:215], v[136:139], v[52:55]\n" "ds_read_b64_tr_b8 v[208:209], v228 offset:49152\n" "v_mfma_i32_16x16x64_i8 v[20:23], v[212:215], v[132:135], v[20:23]\n"
    "ds_read_b64_tr_b8 v[210:211], v228 offset:50176\n" "v_mfma_i32_16x16x64_i8 v[16:19], v[216:219], v[140:143], v[16:19]\n" "ds_read_b64_tr_b8 v[212:213], v229 offset:49152\n"
    "buffer_load_dwordx4 v[12:15], v240, s[40:43], s50 offen offset:3072 sc0 sc1\n" "v_mfma_i32_16x16x64_i8 v[48:51], v[216:219], v[232:235], v[48:51]\n"
    "ds_read_b64_tr_b8 v[214:215], v229 offset:50176\n" "s_waitcnt lgkmcnt(15)\n" "v_mfma_i32_16x16x64_i8 v[52:55], v[220:223], v[232:235], v[52:55]\n"
    "ds_read_b64_tr_b8 v[216:217], v228 offset:57344\n" "v_mfma_i32_16x16x64_i8 v[20:23], v[220:223], v[140:143], v[20:23]\n" "ds_read_b64_tr_b8 v[218:219], v228 offset:58368\n"
    "v_mfma_i32_16x16x64_i8 v[16:19], v[192:195], v[236:239], v[16:19]\n" "ds_read_b64_tr_b8 v[220:221], v229 offset:57344\n" "s_mov_b32 s50, 0x1a8000\n"
    "buffer_load_dwordx4 v[44:47], v240, s[40:43], s50 offen offset:2048 sc0 sc1\n" "v_mfma_i32_16x16x64_i8 v[48:51], v[192:195], v[144:147], v[48:51]\n"
    "ds_read_b64_tr_b8 v[222:223], v229 offset:58368\n" "s_waitcnt lgkmcnt(15)\n" "v_mfma_i32_16x16x64_i8 v[52:55], v[196:199], v[144:147], v[52:55]\n" "v_xor_b32 v230, 0x10000, v230\n"
    "ds_read_b64_tr_b8 v[192:193], v230 offset:0\n" "v_mfma_i32_16x16x64_i8 v[20:23], v[196:199], v[236:239], v[20:23]\n" "ds_read_b64_tr_b8 v[194:195], v230 offset:1024\n" "s_waitcnt lgkmcnt(15)\n"
    "v_mfma_i32_16x16x64_i8 v[16:19], v[200:203], v[252:255], v[16:19]\n" "v_xor_b32 v231, 0x10000, v231\n" "ds_read_b64_tr_b8 v[196:197], v231 offset:0\n"
    "v_mfma_i32_16x16x64_i8 v[48:51], v[200:203], v[148:151], v[48:51]\n" "ds_read_b64_tr_b8 v[198:199], v231 offset:1024\n" "s_waitcnt lgkmcnt(14)\n"
    "v_mfma_i32_16x16x64_i8 v[52:55], v[204:207], v[148:151], v[52:55]\n" "ds_read_b64_tr_b8 v[200:201], v230 offset:8192\n" "ds_read_b32 v247, v242 offset:32\n"
    "v_mfma_i32_16x16x64_i8 v[20:23], v[204:207], v[252:255], v[20:23]\n" "ds_read_b64_tr_b8 v[202:203], v230 offset:9216\n" "s_waitcnt lgkmcnt(13)\n"
    "v_mfma_i32_16x16x64_i8 v[16:19], v[208:211], v[152:155], v[16:19]\n" "ds_read_b64_tr_b8 v[204:205], v231 offset:8192\n" "v_mfma_i32_16x16x64_i8 v[48:51], v[208:211], v[156:159], v[48:51]\n"
    "ds_read_b64_tr_b8 v[206:207], v231 offset:9216\n" "s_waitcnt lgkmcnt(13)\n" "v_mfma_i32_16x16x64_i8 v[52:55], v[212:215], v[156:159], v[52:55]\n"
    "ds_read_b64_tr_b8 v[208:209], v230 offset:16384\n" "v_mfma_i32_16x16x64_i8 v[20:23], v[212:215], v[152:155], v[20:23]\n" "ds_read_b64_tr_b8 v[210:211], v230 offset:17408\n"
    "s_waitcnt lgkmcnt(13)\n" "v_mfma_i32_16x16x64_i8 v[16:19], v[216:219], v[160:163], v[16:19]\n" "ds_read_b64_tr_b8 v[212:213], v231 offset:16384\n"
    "v_mfma_i32_16x16x64_i8 v[48:51], v[216:219], v[164:167], v[48:51]\n" "ds_read_b64_tr_b8 v[214:215], v231 offset:17408\n" "s_waitcnt lgkmcnt(13)\n"
    "v_mfma_i32_16x16x64_i8 v[52:55], v[220:223], v[164:167], v[52:55]\n" "ds_read_b64_tr_b8 v[216:217], v230 offset:24576\n" "s_waitcnt lgkmcnt(8)\n" "v_readfirstlane_b32 s54, v247\n"
    "v_mfma_i32_16x16x64_i8 v[20:23], v[220:223], v[160:163], v[20:23]\n" "ds_read_b64_tr_b8 v[218:219], v230 offset:25600\n" "v_mfma_i32_16x16x64_i8 v[24:27], v[192:195], v[180:183], v[24:27]\n"
    "ds_read_b64_tr_b8 v[220:221], v231 offset:24576\n" "v_mfma_i32_16x16x64_i8 v[56:59], v[192:195], v[184:187], v[56:59]\n" "ds_read_b64_tr_b8 v[222:223], v231 offset:25600\n"
    "s_andn2_b32 s55, 2, s54\n" "s_cbranch_scc0 L42_%=\n" "L41_%=:\n" "ds_read_b32 v251, v242 offset:32\n" "s_waitcnt lgkmcnt(0)\n" "v_readfirstlane_b32 s54, v251\n" "s_nop 3\n"
    "s_andn2_b32 s55, 2, s54\n" "s_cbranch_scc0 L42_%=\n" "s_sleep 1\n" "s_branch L41_%=\n" "L42_%=:\n" "v_mfma_i32_16x16x64_i8 v[60:63], v[196:199], v[184:187], v[60:63]\n"
    "ds_read_b64_tr_b8 v[192:193], v230 offset:32768\n" "v_mfma_i32_16x16x64_i8 v[28:31], v[196:199], v[180:183], v[28:31]\n" "ds_read_b64_tr_b8 v[194:195], v230 offset:33792\n"
    "ds_write_b128 v245, v[16:19] offset:0\n" "s_waitcnt lgkmcnt(13)\n" "v_mfma_i32_16x16x64_i8 v[24:27], v[200:203], v[188:191], v[24:27]\n" "ds_read_b64_tr_b8 v[196:197], v231 offset:32768\n"
    "buffer_load_dwordx4 v[16:19], v240, s[40:43], s50 offen offset:3072 sc0 sc1\n" "v_mfma_i32_16x16x64_i8 v[56:59], v[200:203], v[128:131], v[56:59]\n"
    "ds_read_b64_tr_b8 v[198:199], v231 offset:33792\n" "ds_write_b128 v245, v[48:51] offset:2048\n" "s_waitcnt lgkmcnt(14)\n" "v_mfma_i32_16x16x64_i8 v[60:63], v[204:207], v[128:131], v[60:63]\n"
    "ds_read_b64_tr_b8 v[200:201], v230 offset:40960\n" "v_mfma_i32_16x16x64_i8 v[28:31], v[204:207], v[188:191], v[28:31]\n" "ds_read_b64_tr_b8 v[202:203], v230 offset:41984\n"
    "ds_write_b128 v246, v[20:23] offset:0\n" "s_waitcnt lgkmcnt(15)\n" "v_mfma_i32_16x16x64_i8 v[24:27], v[208:211], v[132:135], v[24:27]\n" "ds_read_b64_tr_b8 v[204:205], v231 offset:40960\n"
    "s_mov_b32 s50, 0x1b0000\n" "buffer_load_dwordx4 v[180:183], v240, s[40:43], s50 offen offset:2048 sc0 sc1\n" "v_mfma_i32_16x16x64_i8 v[56:59], v[208:211], v[136:139], v[56:59]\n"
    "ds_read_b64_tr_b8 v[206:207], v231 offset:41984\n" "ds_write_b128 v246, v[52:55] offset:2048\n" "v_mov_b32 v251, 3\n" "ds_write_b32 v242, v251 offset:16\n" "s_waitcnt lgkmcnt(15)\n"
    "v_mfma_i32_16x16x64_i8 v[60:63], v[212:215], v[136:139], v[60:63]\n" "ds_read_b64_tr_b8 v[208:209], v230 offset:49152\n" "v_mfma_i32_16x16x64_i8 v[28:31], v[212:215], v[132:135], v[28:31]\n"
    "ds_read_b64_tr_b8 v[210:211], v230 offset:50176\n" "v_mfma_i32_16x16x64_i8 v[24:27], v[216:219], v[140:143], v[24:27]\n" "ds_read_b64_tr_b8 v[212:213], v231 offset:49152\n"
    "buffer_load_dwordx4 v[184:187], v240, s[40:43], s50 offen offset:3072 sc0 sc1\n" "v_mfma_i32_16x16x64_i8 v[56:59], v[216:219], v[232:235], v[56:59]\n"
    "ds_read_b64_tr_b8 v[214:215], v231 offset:50176\n" "s_waitcnt lgkmcnt(15)\n" "v_mfma_i32_16x16x64_i8 v[60:63], v[220:223], v[232:235], v[60:63]\n"
    "ds_read_b64_tr_b8 v[216:217], v230 offset:57344\n" "v_mfma_i32_16x16x64_i8 v[28:31], v[220:223], v[140:143], v[28:31]\n" "ds_read_b64_tr_b8 v[218:219], v230 offset:58368\n"
    "v_mfma_i32_16x16x64_i8 v[24:27], v[192:195], v[236:239], v[24:27]\n" "ds_read_b64_tr_b8 v[220:221], v231 offset:57344\n" "s_mov_b32 s50, 0x1b8000\n"
    "buffer_load_dwordx4 v[48:51], v240, s[40:43], s50 offen offset:2048 sc0 sc1\n" "v_mfma_i32_16x16x64_i8 v[56:59], v[192:195], v[144:147], v[56:59]\n"
    "ds_read_b64_tr_b8 v[222:223], v231 offset:58368\n" "s_waitcnt lgkmcnt(15)\n" "v_mfma_i32_16x16x64_i8 v[60:63], v[196:199], v[144:147], v[60:63]\n" "v_xor_b32 v224, 0x10000, v224\n"
    "ds_read_b64_tr_b8 v[192:193], v224 offset:0\n" "v_mfma_i32_16x16x64_i8 v[28:31], v[196:199], v[236:239], v[28:31]\n" "ds_read_b64_tr_b8 v[194:195], v224 offset:1024\n" "s_waitcnt lgkmcnt(15)\n"
    "v_mfma_i32_16x16x64_i8 v[24:27], v[200:203], v[252:255], v[24:27]\n" "v_xor_b32 v225, 0x10000, v225\n" "ds_read_b64_tr_b8 v[196:197], v225 offset:0\n"
    "buffer_load_dwordx4 v[20:23], v240, s[40:43], s50 offen offset:3072 sc0 sc1\n" "v_mfma_i32_16x16x64_i8 v[56:59], v[200:203], v[148:151], v[56:59]\n"
    "ds_read_b64_tr_b8 v[198:199], v225 offset:1024\n" "s_waitcnt lgkmcnt(14)\n" "v_mfma_i32_16x16x64_i8 v[60:63], v[204:207], v[148:151], v[60:63]\n" "ds_read_b64_tr_b8 v[200:201], v224 offset:8192\n"
    "ds_read_b32 v247, v242 offset:32\n" "v_mfma_i32_16x16x64_i8 v[28:31], v[204:207], v[252:255], v[28:31]\n" "ds_read_b64_tr_b8 v[202:203], v224 offset:9216\n" "s_waitcnt lgkmcnt(13)\n"
    "v_mfma_i32_16x16x64_i8 v[24:27], v[208:211], v[152:155], v[24:27]\n" "ds_read_b64_tr_b8 v[204:205], v225 offset:8192\n" "s_mov_b32 s50, 0x1c0000\n"
    "buffer_load_dwordx4 v[128:131], v240, s[40:43], s50 offen offset:2048 sc0 sc1\n" "v_mfma_i32_16x16x64_i8 v[56:59], v[208:211], v[156:159], v[56:59]\n"
    "ds_read_b64_tr_b8 v[206:207], v225 offset:9216\n" "s_waitcnt lgkmcnt(13)\n" "v_mfma_i32_16x16x64_i8 v[60:63], v[212:215], v[156:159], v[60:63]\n"
    "ds_read_b64_tr_b8 v[208:209], v224 offset:16384\n" "v_mfma_i32_16x16x64_i8 v[28:31], v[212:215], v[152:155], v[28:31]\n" "ds_read_b64_tr_b8 v[210:211], v224 offset:17408\n"
    "s_waitcnt lgkmcnt(13)\n" "v_mfma_i32_16x16x64_i8 v[24:27], v[216:219], v[160:163], v[24:27]\n" "ds_read_b64_tr_b8 v[212:213], v225 offset:16384\n"
    "buffer_load_dwordx4 v[188:191], v240, s[40:43], s50 offen offset:3072 sc0 sc1\n" "v_mfma_i32_16x16x64_i8 v[56:59], v[216:219], v[164:167], v[56:59]\n"
    "ds_read_b64_tr_b8 v[214:215], v225 offset:17408\n" "s_waitcnt lgkmcnt(13)\n" "v_mfma_i32_16x16x64_i8 v[60:63], v[220:223], v[164:167], v[60:63]\n"
    "ds_read_b64_tr_b8 v[216:217], v224 offset:24576\n" "s_waitcnt lgkmcnt(8)\n" "v_readfirstlane_b32 s54, v247\n" "v_mfma_i32_16x16x64_i8 v[28:31], v[220:223], v[160:163], v[28:31]\n"
    "ds_read_b64_tr_b8 v[218:219], v224 offset:25600\n" "s_waitcnt vmcnt(17)\n" "v_mfma_i32_16x16x64_i8 v[64:67], v[192:195], v[168:171], v[64:67]\n" "ds_read_b64_tr_b8 v[220:221], v225 offset:24576\n"
    "s_mov_b32 s50, 0x1c8000\n" "buffer_load_dwordx4 v[52:55], v240, s[40:43], s50 offen offset:2048 sc0 sc1\n" "s_waitcnt vmcnt(17)\n"
    "v_mfma_i32_16x16x64_i8 v[96:99], v[192:195], v[172:175], v[96:99]\n" "ds_read_b64_tr_b8 v[222:223], v225 offset:25600\n" "s_andn2_b32 s55, 4, s54\n" "s_cbranch_scc0 L44_%=\n" "L43_%=:\n"
    "ds_read_b32 v251, v242 offset:32\n" "s_waitcnt lgkmcnt(0)\n" "v_readfirstlane_b32 s54, v251\n" "s_nop 3\n" "s_andn2_b32 s55, 4, s54\n" "s_cbranch_scc0 L44_%=\n" "s_sleep 1\n" "s_branch L43_%=\n"
    "L44_%=:\n" "v_mfma_i32_16x16x64_i8 v[100:103], v[196:199], v[172:175], v[100:103]\n" "ds_read_b64_tr_b8 v[192:193], v224 offset:32768\n"
    "v_mfma_i32_16x16x64_i8 v[68:71], v[196:199], v[168:171], v[68:71]\n" "ds_read_b64_tr_b8 v[194:195], v224 offset:33792\n" "ds_write_b128 v245, v[24:27] offset:0\n"
    "s_waitcnt vmcnt(16) lgkmcnt(13)\n" "v_mfma_i32_16x16x64_i8 v[64:67], v[200:203], v[176:179], v[64:67]\n" "ds_read_b64_tr_b8 v[196:197], v225 offset:32768\n"
    "buffer_load_dwordx4 v[132:135], v240, s[40:43], s50 offen offset:3072 sc0 sc1\n" "s_waitcnt vmcnt(16)\n" "v_mfma_i32_16x16x64_i8 v[96:99], v[200:203], v[0:3], v[96:99]\n"
    "ds_read_b64_tr_b8 v[198:199], v225 offset:33792\n" "ds_write_b128 v245, v[56:59] offset:2048\n" "s_waitcnt lgkmcnt(14)\n" "v_mfma_i32_16x16x64_i8 v[100:103], v[204:207], v[0:3], v[100:103]\n"
    "ds_read_b64_tr_b8 v[200:201], v224 offset:40960\n" "v_mfma_i32_16x16x64_i8 v[68:71], v[204:207], v[176:179], v[68:71]\n" "ds_read_b64_tr_b8 v[202:203], v224 offset:41984\n"
    "ds_write_b128 v246, v[28:31] offset:0\n" "s_waitcnt vmcnt(15) lgkmcnt(15)\n" "v_mfma_i32_16x16x64_i8 v[64:67], v[208:211], v[32:35], v[64:67]\n" "ds_read_b64_tr_b8 v[204:205], v225 offset:40960\n"
    "s_mov_b32 s50, 0x1d0000\n" "buffer_load_dwordx4 v[136:139], v240, s[40:43], s50 offen offset:2048 sc0 sc1\n" "s_waitcnt vmcnt(15)\n"
    "v_mfma_i32_16x16x64_i8 v[96:99], v[208:211], v[4:7], v[96:99]\n" "ds_read_b64_tr_b8 v[206:207], v225 offset:41984\n" "ds_write_b128 v246, v[60:63] offset:2048\n" "v_mov_b32 v251, 4\n"
    "ds_write_b32 v242, v251 offset:16\n" "s_waitcnt lgkmcnt(15)\n" "v_mfma_i32_16x16x64_i8 v[100:103], v[212:215], v[4:7], v[100:103]\n" "v_xor_b32 v226, 0x10000, v226\n"
    "ds_read_b64_tr_b8 v[208:209], v226 offset:0\n" "v_mfma_i32_16x16x64_i8 v[68:71], v[212:215], v[32:35], v[68:71]\n" "ds_read_b64_tr_b8 v[210:211], v226 offset:1024\n" "s_waitcnt vmcnt(14)\n"
    "v_mfma_i32_16x16x64_i8 v[64:67], v[216:219], v[36:39], v[64:67]\n" "v_xor_b32 v227, 0x10000, v227\n" "ds_read_b64_tr_b8 v[212:213], v227 offset:0\n"
    "buffer_load_dwordx4 v[140:143], v240, s[40:43], s50 offen offset:3072 sc0 sc1\n" "s_waitcnt vmcnt(14)\n" "v_mfma_i32_16x16x64_i8 v[96:99], v[216:219], v[8:11], v[96:99]\n"
    "ds_read_b64_tr_b8 v[214:215], v227 offset:1024\n" "s_waitcnt lgkmcnt(15)\n" "v_mfma_i32_16x16x64_i8 v[100:103], v[220:223], v[8:11], v[100:103]\n"
    "ds_read_b64_tr_b8 v[216:217], v226 offset:8192\n" "v_mfma_i32_16x16x64_i8 v[68:71], v[220:223], v[36:39], v[68:71]\n" "ds_read_b64_tr_b8 v[218:219], v226 offset:9216\n" "s_waitcnt vmcnt(13)\n"
    "v_mfma_i32_16x16x64_i8 v[64:67], v[192:195], v[40:43], v[64:67]\n" "ds_read_b64_tr_b8 v[220:221], v227 offset:8192\n" "s_mov_b32 s50, 0x1d8000\n"
    "buffer_load_dwordx4 v[232:235], v240, s[40:43], s50 offen offset:2048 sc0 sc1\n" "s_waitcnt vmcnt(13)\n" "v_mfma_i32_16x16x64_i8 v[96:99], v[192:195], v[12:15], v[96:99]\n"
    "ds_read_b64_tr_b8 v[222:223], v227 offset:9216\n" "s_waitcnt lgkmcnt(15)\n" "v_mfma_i32_16x16x64_i8 v[100:103], v[196:199], v[12:15], v[100:103]\n"
    "ds_read_b64_tr_b8 v[192:193], v226 offset:16384\n" "v_mfma_i32_16x16x64_i8 v[68:71], v[196:199], v[40:43], v[68:71]\n" "ds_read_b64_tr_b8 v[194:195], v226 offset:17408\n"
    "s_waitcnt vmcnt(12) lgkmcnt(15)\n" "v_mfma_i32_16x16x64_i8 v[64:67], v[200:203], v[44:47], v[64:67]\n" "ds_read_b64_tr_b8 v[196:197], v227 offset:16384\n"
    "buffer_load_dwordx4 v[144:147], v240, s[40:43], s50 offen offset:3072 sc0 sc1\n" "s_waitcnt vmcnt(12)\n" "v_mfma_i32_16x16x64_i8 v[96:99], v[200:203], v[16:19], v[96:99]\n"
    "ds_read_b64_tr_b8 v[198:199], v227 offset:17408\n" "s_waitcnt lgkmcnt(14)\n" "v_mfma_i32_16x16x64_i8 v[100:103], v[204:207], v[16:19], v[100:103]\n"
    "ds_read_b64_tr_b8 v[200:201], v226 offset:24576\n" "v_mfma_i32_16x16x64_i8 v[68:71], v[204:207], v[44:47], v[68:71]\n" "ds_read_b64_tr_b8 v[202:203], v226 offset:25600\n" "s_waitcnt lgkmcnt(12)\n"
    "v_mfma_i32_16x16x64_i8 v[72:75], v[208:211], v[168:171], v[72:75]\n" "ds_read_b64_tr_b8 v[204:205], v227 offset:24576\n" "s_mov_b32 s50, 0x1e0000\n"
    "buffer_load_dwordx4 v[236:239], v240, s[40:43], s50 offen offset:2048 sc0 sc1\n" "v_mfma_i32_16x16x64_i8 v[104:107], v[208:211], v[172:175], v[104:107]\n"
    "ds_read_b64_tr_b8 v[206:207], v227 offset:25600\n" "s_waitcnt lgkmcnt(12)\n" "v_mfma_i32_16x16x64_i8 v[108:111], v[212:215], v[172:175], v[108:111]\n"
    "ds_read_b64_tr_b8 v[208:209], v226 offset:32768\n" "v_mfma_i32_16x16x64_i8 v[76:79], v[212:215], v[168:171], v[76:79]\n" "ds_read_b64_tr_b8 v[210:211], v226 offset:33792\n"
    "s_waitcnt lgkmcnt(12)\n" "v_mfma_i32_16x16x64_i8 v[72:75], v[216:219], v[176:179], v[72:75]\n" "ds_read_b64_tr_b8 v[212:213], v227 offset:32768\n"
    "buffer_load_dwordx4 v[148:151], v240, s[40:43], s50 offen offset:3072 sc0 sc1\n" "v_mfma_i32_16x16x64_i8 v[104:107], v[216:219], v[0:3], v[104:107]\n"
    "ds_read_b64_tr_b8 v[214:215], v227 offset:33792\n" "s_waitcnt lgkmcnt(12)\n" "v_mfma_i32_16x16x64_i8 v[108:111], v[220:223], v[0:3], v[108:111]\n"
    "ds_read_b64_tr_b8 v[216:217], v226 offset:40960\n" "v_mfma_i32_16x16x64_i8 v[76:79], v[220:223], v[176:179], v[76:79]\n" "ds_read_b64_tr_b8 v[218:219], v226 offset:41984\n"
    "s_waitcnt lgkmcnt(12)\n" "v_mfma_i32_16x16x64_i8 v[72:75], v[192:195], v[32:35], v[72:75]\n" "ds_read_b64_tr_b8 v[220:221], v227 offset:40960\n" "s_mov_b32 s50, 0x1e8000\n"
    "buffer_load_dwordx4 v[252:255], v240, s[40:43], s50 offen offset:2048 sc0 sc1\n" "v_mfma_i32_16x16x64_i8 v[104:107], v[192:195], v[4:7], v[104:107]\n"
    "ds_read_b64_tr_b8 v[222:223], v227 offset:41984\n" "s_waitcnt lgkmcnt(12)\n" "v_mfma_i32_16x16x64_i8 v[108:111], v[196:199], v[4:7], v[108:111]\n"
    "ds_read_b64_tr_b8 v[192:193], v224 offset:49152\n" "v_mfma_i32_16x16x64_i8 v[76:79], v[196:199], v[32:35], v[76:79]\n" "ds_read_b64_tr_b8 v[194:195], v224 offset:50176\n" "s_waitcnt lgkmcnt(12)\n"
    "v_mfma_i32_16x16x64_i8 v[72:75], v[200:203], v[36:39], v[72:75]\n" "ds_read_b64_tr_b8 v[196:197], v225 offset:49152\n"
    "buffer_load_dwordx4 v[152:155], v240, s[40:43], s50 offen offset:3072 sc0 sc1\n" "v_mfma_i32_16x16x64_i8 v[104:107], v[200:203], v[8:11], v[104:107]\n"
    "ds_read_b64_tr_b8 v[198:199], v225 offset:50176\n" "s_waitcnt lgkmcnt(12)\n" "v_mfma_i32_16x16x64_i8 v[108:111], v[204:207], v[8:11], v[108:111]\n"
    "ds_read_b64_tr_b8 v[200:201], v224 offset:57344\n" "v_mfma_i32_16x16x64_i8 v[76:79], v[204:207], v[36:39], v[76:79]\n" "ds_read_b64_tr_b8 v[202:203], v224 offset:58368\n" "s_waitcnt lgkmcnt(12)\n"
    "v_mfma_i32_16x16x64_i8 v[72:75], v[208:211], v[40:43], v[72:75]\n" "ds_read_b64_tr_b8 v[204:205], v225 offset:57344\n" "s_mov_b32 s50, 0x1f0000\n"
    "buffer_load_dwordx4 v[156:159], v240, s[40:43], s50 offen offset:2048 sc0 sc1\n" "v_mfma_i32_16x16x64_i8 v[104:107], v[208:211], v[12:15], v[104:107]\n"
    "ds_read_b64_tr_b8 v[206:207], v225 offset:58368\n" "s_waitcnt lgkmcnt(12)\n" "v_mfma_i32_16x16x64_i8 v[108:111], v[212:215], v[12:15], v[108:111]\n" "v_xor_b32 v224, 0x10000, v224\n"
    "ds_read_b64_tr_b8 v[208:209], v224 offset:0\n" "v_mfma_i32_16x16x64_i8 v[76:79], v[212:215], v[40:43], v[76:79]\n" "ds_read_b64_tr_b8 v[210:211], v224 offset:1024\n" "s_waitcnt lgkmcnt(12)\n"
    "v_mfma_i32_16x16x64_i8 v[72:75], v[216:219], v[44:47], v[72:75]\n" "v_xor_b32 v225, 0x10000, v225\n" "ds_read_b64_tr_b8 v[212:213], v225 offset:0\n"
    "buffer_load_dwordx4 v[160:163], v240, s[40:43], s50 offen offset:3072 sc0 sc1\n" "v_mfma_i32_16x16x64_i8 v[104:107], v[216:219], v[16:19], v[104:107]\n"
    "ds_read_b64_tr_b8 v[214:215], v225 offset:1024\n" "s_waitcnt lgkmcnt(12)\n" "v_mfma_i32_16x16x64_i8 v[108:111], v[220:223], v[16:19], v[108:111]\n"
    "ds_read_b64_tr_b8 v[216:217], v224 offset:8192\n" "v_mfma_i32_16x16x64_i8 v[76:79], v[220:223], v[44:47], v[76:79]\n" "ds_read_b64_tr_b8 v[218:219], v224 offset:9216\n"
    "s_waitcnt vmcnt(17) lgkmcnt(12)\n" "v_mfma_i32_16x16x64_i8 v[64:67], v[192:195], v[180:183], v[64:67]\n" "ds_read_b64_tr_b8 v[220:221], v225 offset:8192\n" "s_mov_b32 s50, 0x1f8000\n"
    "buffer_load_dwordx4 v[164:167], v240, s[40:43], s50 offen offset:2048 sc0 sc1\n" "s_waitcnt vmcnt(17)\n" "v_mfma_i32_16x16x64_i8 v[96:99], v[192:195], v[184:187], v[96:99]\n"
    "ds_read_b64_tr_b8 v[222:223], v225 offset:9216\n" "s_waitcnt lgkmcnt(12)\n" "v_mfma_i32_16x16x64_i8 v[100:103], v[196:199], v[184:187], v[100:103]\n"
    "ds_read_b64_tr_b8 v[192:193], v224 offset:16384\n" "v_mfma_i32_16x16x64_i8 v[68:71], v[196:199], v[180:183], v[68:71]\n" "ds_read_b64_tr_b8 v[194:195], v224 offset:17408\n"
    "s_waitcnt vmcnt(16) lgkmcnt(12)\n" "v_mfma_i32_16x16x64_i8 v[64:67], v[200:203], v[48:51], v[64:67]\n" "ds_read_b64_tr_b8 v[196:197], v225 offset:16384\n"
    "buffer_load_dwordx4 v[24:27], v240, s[40:43], s50 offen offset:3072 sc0 sc1\n" "s_waitcnt vmcnt(16)\n" "v_mfma_i32_16x16x64_i8 v[96:99], v[200:203], v[20:23], v[96:99]\n"
    "ds_read_b64_tr_b8 v[198:199], v225 offset:17408\n" "s_waitcnt lgkmcnt(12)\n" "v_mfma_i32_16x16x64_i8 v[100:103], v[204:207], v[20:23], v[100:103]\n"
    "ds_read_b64_tr_b8 v[200:201], v224 offset:24576\n" "v_mfma_i32_16x16x64_i8 v[68:71], v[204:207], v[48:51], v[68:71]\n" "ds_read_b64_tr_b8 v[202:203], v224 offset:25600\n"
    "s_waitcnt vmcnt(15) lgkmcnt(12)\n" "v_mfma_i32_16x16x64_i8 v[64:67], v[208:211], v[128:131], v[64:67]\n" "ds_read_b64_tr_b8 v[204:205], v225 offset:24576\n" "s_waitcnt vmcnt(14)\n"
    "v_mfma_i32_16x16x64_i8 v[96:99], v[208:211], v[188:191], v[96:99]\n" "ds_read_b64_tr_b8 v[206:207], v225 offset:25600\n" "s_waitcnt lgkmcnt(12)\n"
    "v_mfma_i32_16x16x64_i8 v[100:103], v[212:215], v[188:191], v[100:103]\n" "ds_read_b64_tr_b8 v[208:209], v224 offset:32768\n" "v_mfma_i32_16x16x64_i8 v[68:71], v[212:215], v[128:131], v[68:71]\n"
    "ds_read_b64_tr_b8 v[210:211], v224 offset:33792\n" "s_waitcnt vmcnt(13) lgkmcnt(12)\n" "v_mfma_i32_16x16x64_i8 v[64:67], v[216:219], v[52:55], v[64:67]\n"
    "ds_read_b64_tr_b8 v[212:213], v225 offset:32768\n" "s_waitcnt vmcnt(12)\n" "v_mfma_i32_16x16x64_i8 v[96:99], v[216:219], v[132:135], v[96:99]\n" "ds_read_b64_tr_b8 v[214:215], v225 offset:33792\n"
    "s_waitcnt lgkmcnt(12)\n" "v_mfma_i32_16x16x64_i8 v[100:103], v[220:223], v[132:135], v[100:103]\n" "ds_read_b64_tr_b8 v[216:217], v224 offset:40960\n"
    "v_mfma_i32_16x16x64_i8 v[68:71], v[220:223], v[52:55], v[68:71]\n" "ds_read_b64_tr_b8 v[218:219], v224 offset:41984\n" "s_waitcnt vmcnt(11) lgkmcnt(12)\n"
    "v_mfma_i32_16x16x64_i8 v[64:67], v[192:195], v[136:139], v[64:67]\n" "ds_read_b64_tr_b8 v[220:221], v225 offset:40960\n" "s_waitcnt vmcnt(10)\n"
    "v_mfma_i32_16x16x64_i8 v[96:99], v[192:195], v[140:143], v[96:99]\n" "ds_read_b64_tr_b8 v[222:223], v225 offset:41984\n" "s_waitcnt lgkmcnt(12)\n"
    "v_mfma_i32_16x16x64_i8 v[100:103], v[196:199], v[140:143], v[100:103]\n" "ds_read_b64_tr_b8 v[192:193], v224 offset:49152\n" "v_mfma_i32_16x16x64_i8 v[68:71], v[196:199], v[136:139], v[68:71]\n"
    "ds_read_b64_tr_b8 v[194:195], v224 offset:50176\n" "s_waitcnt vmcnt(9) lgkmcnt(12)\n" "v_mfma_i32_16x16x64_i8 v[64:67], v[200:203], v[232:235], v[64:67]\n"
    "ds_read_b64_tr_b8 v[196:197], v225 offset:49152\n" "s_waitcnt vmcnt(8)\n" "v_mfma_i32_16x16x64_i8 v[96:99], v[200:203], v[144:147], v[96:99]\n" "ds_read_b64_tr_b8 v[198:199], v225 offset:50176\n"
    "s_waitcnt lgkmcnt(12)\n" "v_mfma_i32_16x16x64_i8 v[100:103], v[204:207], v[144:147], v[100:103]\n" "ds_read_b64_tr_b8 v[200:201], v224 offset:57344\n"
    "v_mfma_i32_16x16x64_i8 v[68:71], v[204:207], v[232:235], v[68:71]\n" "ds_read_b64_tr_b8 v[202:203], v224 offset:58368\n" "s_waitcnt vmcnt(7) lgkmcnt(12)\n"
    "v_mfma_i32_16x16x64_i8 v[64:67], v[208:211], v[236:239], v[64:67]\n" "ds_read_b64_tr_b8 v[204:205], v225 offset:57344\n" "s_waitcnt vmcnt(6)\n"
    "v_mfma_i32_16x16x64_i8 v[96:99], v[208:211], v[148:151], v[96:99]\n" "ds_read_b64_tr_b8 v[206:207], v225 offset:58368\n" "s_waitcnt lgkmcnt(12)\n"
    "v_mfma_i32_16x16x64_i8 v[100:103], v[212:215], v[148:151], v[100:103]\n" "ds_read_b64_tr_b8 v[208:209], v226 offset:49152\n" "v_mfma_i32_16x16x64_i8 v[68:71], v[212:215], v[236:239], v[68:71]\n"
    "ds_read_b64_tr_b8 v[210:211], v226 offset:50176\n" "s_waitcnt vmcnt(5) lgkmcnt(12)\n" "v_mfma_i32_16x16x64_i8 v[64:67], v[216:219], v[252:255], v[64:67]\n"
    "ds_read_b64_tr_b8 v[212:213], v227 offset:49152\n" "s_waitcnt vmcnt(4)\n" "v_mfma_i32_16x16x64_i8 v[96:99], v[216:219], v[152:155], v[96:99]\n" "ds_read_b64_tr_b8 v[214:215], v227 offset:50176\n"
    "s_waitcnt lgkmcnt(12)\n" "v_mfma_i32_16x16x64_i8 v[100:103], v[220:223], v[152:155], v[100:103]\n" "ds_read_b64_tr_b8 v[216:217], v226 offset:57344\n" "ds_read_b32 v247, v242 offset:32\n"
    "v_mfma_i32_16x16x64_i8 v[68:71], v[220:223], v[252:255], v[68:71]\n" "ds_read_b64_tr_b8 v[218:219], v226 offset:58368\n" "s_waitcnt vmcnt(3) lgkmcnt(13)\n"
    "v_mfma_i32_16x16x64_i8 v[64:67], v[192:195], v[156:159], v[64:67]\n" "ds_read_b64_tr_b8 v[220:221], v227 offset:57344\n" "s_waitcnt vmcnt(2)\n"
    "v_mfma_i32_16x16x64_i8 v[96:99], v[192:195], v[160:163], v[96:99]\n" "ds_read_b64_tr_b8 v[222:223], v227 offset:58368\n" "s_waitcnt lgkmcnt(13)\n"
    "v_mfma_i32_16x16x64_i8 v[100:103], v[196:199], v[160:163], v[100:103]\n" "v_xor_b32 v226, 0x10000, v226\n" "ds_read_b64_tr_b8 v[192:193], v226 offset:0\n"
    "v_mfma_i32_16x16x64_i8 v[68:71], v[196:199], v[156:159], v[68:71]\n" "ds_read_b64_tr_b8 v[194:195], v226 offset:1024\n" "s_waitcnt vmcnt(1) lgkmcnt(13)\n"
    "v_mfma_i32_16x16x64_i8 v[64:67], v[200:203], v[164:167], v[64:67]\n" "v_xor_b32 v227, 0x10000, v227\n" "ds_read_b64_tr_b8 v[196:197], v227 offset:0\n" "s_waitcnt vmcnt(0)\n"
    "v_mfma_i32_16x16x64_i8 v[96:99], v[200:203], v[24:27], v[96:99]\n" "ds_read_b64_tr_b8 v[198:199], v227 offset:1024\n" "s_waitcnt lgkmcnt(13)\n"
    "v_mfma_i32_16x16x64_i8 v[100:103], v[204:207], v[24:27], v[100:103]\n" "ds_read_b64_tr_b8 v[200:201], v226 offset:8192\n" "s_waitcnt lgkmcnt(8)\n" "v_readfirstlane_b32 s54, v247\n"
    "v_mfma_i32_16x16x64_i8 v[68:71], v[204:207], v[164:167], v[68:71]\n" "ds_read_b64_tr_b8 v[202:203], v226 offset:9216\n" "v_mfma_i32_16x16x64_i8 v[72:75], v[208:211], v[180:183], v[72:75]\n"
    "ds_read_b64_tr_b8 v[204:205], v227 offset:8192\n" "v_mfma_i32_16x16x64_i8 v[104:107], v[208:211], v[184:187], v[104:107]\n" "ds_read_b64_tr_b8 v[206:207], v227 offset:9216\n"
    "s_andn2_b32 s55, 8, s54\n" "s_cbranch_scc0 L46_%=\n" "L45_%=:\n" "ds_read_b32 v251, v242 offset:32\n" "s_waitcnt lgkmcnt(0)\n" "v_readfirstlane_b32 s54, v251\n" "s_nop 3\n"
    "s_andn2_b32 s55, 8, s54\n" "s_cbranch_scc0 L46_%=\n" "s_sleep 1\n" "s_branch L45_%=\n" "L46_%=:\n" "v_mfma_i32_16x16x64_i8 v[108:111], v[212:215], v[184:187], v[108:111]\n"
    "ds_read_b64_tr_b8 v[208:209], v226 offset:16384\n" "v_mfma_i32_16x16x64_i8 v[76:79], v[212:215], v[180:183], v[76:79]\n" "ds_read_b64_tr_b8 v[210:211], v226 offset:17408\n"
    "ds_write_b128 v245, v[64:67] offset:0\n" "s_waitcnt lgkmcnt(13)\n" "v_mfma_i32_16x16x64_i8 v[72:75], v[216:219], v[48:51], v[72:75]\n" "ds_read_b64_tr_b8 v[212:213], v227 offset:16384\n"
    "v_mfma_i32_16x16x64_i8 v[104:107], v[216:219], v[20:23], v[104:107]\n" "ds_read_b64_tr_b8 v[214:215], v227 offset:17408\n" "ds_write_b128 v245, v[96:99] offset:2048\n" "s_waitcnt lgkmcnt(14)\n"
    "v_mfma_i32_16x16x64_i8 v[108:111], v[220:223], v[20:23], v[108:111]\n" "ds_read_b64_tr_b8 v[216:217], v226 offset:24576\n" "v_mfma_i32_16x16x64_i8 v[76:79], v[220:223], v[48:51], v[76:79]\n"
    "ds_read_b64_tr_b8 v[218:219], v226 offset:25600\n" "ds_write_b128 v246, v[68:71] offset:0\n" "s_waitcnt lgkmcnt(15)\n" "v_mfma_i32_16x16x64_i8 v[72:75], v[192:195], v[128:131], v[72:75]\n"
    "ds_read_b64_tr_b8 v[220:221], v227 offset:24576\n" "v_mfma_i32_16x16x64_i8 v[104:107], v[192:195], v[188:191], v[104:107]\n" "ds_read_b64_tr_b8 v[222:223], v227 offset:25600\n"
    "ds_write_b128 v246, v[100:103] offset:2048\n" "v_mov_b32 v251, 5\n" "ds_write_b32 v242, v251 offset:16\n" "s_waitcnt lgkmcnt(15)\n"
    "v_mfma_i32_16x16x64_i8 v[108:111], v[196:199], v[188:191], v[108:111]\n" "ds_read_b64_tr_b8 v[192:193], v226 offset:32768\n" "v_mfma_i32_16x16x64_i8 v[76:79], v[196:199], v[128:131], v[76:79]\n"
    "ds_read_b64_tr_b8 v[194:195], v226 offset:33792\n" "v_mfma_i32_16x16x64_i8 v[72:75], v[200:203], v[52:55], v[72:75]\n" "ds_read_b64_tr_b8 v[196:197], v227 offset:32768\n"
    "v_mfma_i32_16x16x64_i8 v[104:107], v[200:203], v[132:135], v[104:107]\n" "ds_read_b64_tr_b8 v[198:199], v227 offset:33792\n" "s_waitcnt lgkmcnt(15)\n"
    "v_mfma_i32_16x16x64_i8 v[108:111], v[204:207], v[132:135], v[108:111]\n" "ds_read_b64_tr_b8 v[200:201], v226 offset:40960\n" "v_mfma_i32_16x16x64_i8 v[76:79], v[204:207], v[52:55], v[76:79]\n"
    "ds_read_b64_tr_b8 v[202:203], v226 offset:41984\n" "v_mfma_i32_16x16x64_i8 v[72:75], v[208:211], v[136:139], v[72:75]\n" "ds_read_b64_tr_b8 v[204:205], v227 offset:40960\n"
    "v_mfma_i32_16x16x64_i8 v[104:107], v[208:211], v[140:143], v[104:107]\n" "ds_read_b64_tr_b8 v[206:207], v227 offset:41984\n" "s_waitcnt lgkmcnt(15)\n"
    "v_mfma_i32_16x16x64_i8 v[108:111], v[212:215], v[140:143], v[108:111]\n" "ds_read_b64_tr_b8 v[208:209], v226 offset:49152\n" "v_mfma_i32_16x16x64_i8 v[76:79], v[212:215], v[136:139], v[76:79]\n"
    "ds_read_b64_tr_b8 v[210:211], v226 offset:50176\n" "s_waitcnt lgkmcnt(15)\n" "v_mfma_i32_16x16x64_i8 v[72:75], v[216:219], v[232:235], v[72:75]\n"
    "ds_read_b64_tr_b8 v[212:213], v227 offset:49152\n" "v_mfma_i32_16x16x64_i8 v[104:107], v[216:219], v[144:147], v[104:107]\n" "ds_read_b64_tr_b8 v[214:215], v227 offset:50176\n"
    "s_waitcnt lgkmcnt(14)\n" "v_mfma_i32_16x16x64_i8 v[108:111], v[220:223], v[144:147], v[108:111]\n" "ds_read_b64_tr_b8 v[216:217], v226 offset:57344\n"
    "v_mfma_i32_16x16x64_i8 v[76:79], v[220:223], v[232:235], v[76:79]\n" "ds_read_b64_tr_b8 v[218:219], v226 offset:58368\n" "s_waitcnt lgkmcnt(12)\n"
    "v_mfma_i32_16x16x64_i8 v[72:75], v[192:195], v[236:239], v[72:75]\n" "ds_read_b64_tr_b8 v[220:221], v227 offset:57344\n" "v_mfma_i32_16x16x64_i8 v[104:107], v[192:195], v[148:151], v[104:107]\n"
    "ds_read_b64_tr_b8 v[222:223], v227 offset:58368\n" "s_waitcnt lgkmcnt(12)\n" "v_mfma_i32_16x16x64_i8 v[108:111], v[196:199], v[148:151], v[108:111]\n" "v_xor_b32 v228, 0x10000, v228\n"
    "ds_read_b64_tr_b8 v[192:193], v228 offset:0\n" "v_mfma_i32_16x16x64_i8 v[76:79], v[196:199], v[236:239], v[76:79]\n" "ds_read_b64_tr_b8 v[194:195], v228 offset:1024\n" "s_waitcnt lgkmcnt(12)\n"
    "v_mfma_i32_16x16x64_i8 v[72:75], v[200:203], v[252:255], v[72:75]\n" "v_xor_b32 v229, 0x10000, v229\n" "ds_read_b64_tr_b8 v[196:197], v229 offset:0\n"
    "v_mfma_i32_16x16x64_i8 v[104:107], v[200:203], v[152:155], v[104:107]\n" "ds_read_b64_tr_b8 v[198:199], v229 offset:1024\n" "s_waitcnt lgkmcnt(12)\n"
    "v_mfma_i32_16x16x64_i8 v[108:111], v[204:207], v[152:155], v[108:111]\n" "ds_read_b64_tr_b8 v[200:201], v228 offset:8192\n" "ds_read_b32 v247, v242 offset:32\n"
    "v_mfma_i32_16x16x64_i8 v[76:79], v[204:207], v[252:255], v[76:79]\n" "ds_read_b64_tr_b8 v[202:203], v228 offset:9216\n" "s_waitcnt lgkmcnt(13)\n"
    "v_mfma_i32_16x16x64_i8 v[72:75], v[208:211], v[156:159], v[72:75]\n" "ds_read_b64_tr_b8 v[204:205], v229 offset:8192\n" "v_mfma_i32_16x16x64_i8 v[104:107], v[208:211], v[160:163], v[104:107]\n"
    "ds_read_b64_tr_b8 v[206:207], v229 offset:9216\n" "s_waitcnt lgkmcnt(13)\n" "v_mfma_i32_16x16x64_i8 v[108:111], v[212:215], v[160:163], v[108:111]\n"
    "ds_read_b64_tr_b8 v[208:209], v228 offset:16384\n" "v_mfma_i32_16x16x64_i8 v[76:79], v[212:215], v[156:159], v[76:79]\n" "ds_read_b64_tr_b8 v[210:211], v228 offset:17408\n"
    "s_waitcnt lgkmcnt(13)\n" "v_mfma_i32_16x16x64_i8 v[72:75], v[216:219], v[164:167], v[72:75]\n" "ds_read_b64_tr_b8 v[212:213], v229 offset:16384\n"
    "v_mfma_i32_16x16x64_i8 v[104:107], v[216:219], v[24:27], v[104:107]\n" "ds_read_b64_tr_b8 v[214:215], v229 offset:17408\n" "s_waitcnt lgkmcnt(13)\n"
    "v_mfma_i32_16x16x64_i8 v[108:111], v[220:223], v[24:27], v[108:111]\n" "ds_read_b64_tr_b8 v[216:217], v228 offset:24576\n" "s_waitcnt lgkmcnt(8)\n" "v_readfirstlane_b32 s54, v247\n"
    "v_mfma_i32_16x16x64_i8 v[76:79], v[220:223], v[164:167], v[76:79]\n" "ds_read_b64_tr_b8 v[218:219], v228 offset:25600\n" "v_mfma_i32_16x16x64_i8 v[80:83], v[192:195], v[168:171], v[80:83]\n"
    "ds_read_b64_tr_b8 v[220:221], v229 offset:24576\n" "v_mfma_i32_16x16x64_i8 v[112:115], v[192:195], v[172:175], v[112:115]\n" "ds_read_b64_tr_b8 v[222:223], v229 offset:25600\n"
    "s_andn2_b32 s55, 16, s54\n" "s_cbranch_scc0 L48_%=\n" "L47_%=:\n" "ds_read_b32 v251, v242 offset:32\n" "s_waitcnt lgkmcnt(0)\n" "v_readfirstlane_b32 s54, v251\n" "s_nop 3\n"
    "s_andn2_b32 s55, 16, s54\n" "s_cbranch_scc0 L48_%=\n" "s_sleep 1\n" "s_branch L47_%=\n" "L48_%=:\n" "v_mfma_i32_16x16x64_i8 v[116:119], v[196:199], v[172:175], v[116:119]\n"
    "ds_read_b64_tr_b8 v[192:193], v228 offset:32768\n" "v_mfma_i32_16x16x64_i8 v[84:87], v[196:199], v[168:171], v[84:87]\n" "ds_read_b64_tr_b8 v[194:195], v228 offset:33792\n"
    "ds_write_b128 v245, v[72:75] offset:0\n" "s_waitcnt lgkmcnt(13)\n" "v_mfma_i32_16x16x64_i8 v[80:83], v[200:203], v[176:179], v[80:83]\n" "ds_read_b64_tr_b8 v[196:197], v229 offset:32768\n"
    "v_mfma_i32_16x16x64_i8 v[112:115], v[200:203], v[0:3], v[112:115]\n" "ds_read_b64_tr_b8 v[198:199], v229 offset:33792\n" "ds_write_b128 v245, v[104:107] offset:2048\n" "s_waitcnt lgkmcnt(14)\n"
    "v_mfma_i32_16x16x64_i8 v[116:119], v[204:207], v[0:3], v[116:119]\n" "ds_read_b64_tr_b8 v[200:201], v228 offset:40960\n" "v_mfma_i32_16x16x64_i8 v[84:87], v[204:207], v[176:179], v[84:87]\n"
    "ds_read_b64_tr_b8 v[202:203], v228 offset:41984\n" "ds_write_b128 v246, v[76:79] offset:0\n" "s_waitcnt lgkmcnt(15)\n" "v_mfma_i32_16x16x64_i8 v[80:83], v[208:211], v[32:35], v[80:83]\n"
    "ds_read_b64_tr_b8 v[204:205], v229 offset:40960\n" "v_mfma_i32_16x16x64_i8 v[112:115], v[208:211], v[4:7], v[112:115]\n" "ds_read_b64_tr_b8 v[206:207], v229 offset:41984\n"
    "ds_write_b128 v246, v[108:111] offset:2048\n" "v_mov_b32 v251, 6\n" "ds_write_b32 v242, v251 offset:16\n" "s_waitcnt lgkmcnt(15)\n"
    "v_mfma_i32_16x16x64_i8 v[116:119], v[212:215], v[4:7], v[116:119]\n" "ds_read_b64_tr_b8 v[208:209], v228 offset:49152\n" "v_mfma_i32_16x16x64_i8 v[84:87], v[212:215], v[32:35], v[84:87]\n"
    "ds_read_b64_tr_b8 v[210:211], v228 offset:50176\n" "v_mfma_i32_16x16x64_i8 v[80:83], v[216:219], v[36:39], v[80:83]\n" "ds_read_b64_tr_b8 v[212:213], v229 offset:49152\n"
    "v_mfma_i32_16x16x64_i8 v[112:115], v[216:219], v[8:11], v[112:115]\n" "ds_read_b64_tr_b8 v[214:215], v229 offset:50176\n" "s_waitcnt lgkmcnt(15)\n"
    "v_mfma_i32_16x16x64_i8 v[116:119], v[220:223], v[8:11], v[116:119]\n" "ds_read_b64_tr_b8 v[216:217], v228 offset:57344\n" "v_mfma_i32_16x16x64_i8 v[84:87], v[220:223], v[36:39], v[84:87]\n"
    "ds_read_b64_tr_b8 v[218:219], v228 offset:58368\n" "v_mfma_i32_16x16x64_i8 v[80:83], v[192:195], v[40:43], v[80:83]\n" "ds_read_b64_tr_b8 v[220:221], v229 offset:57344\n"
    "v_mfma_i32_16x16x64_i8 v[112:115], v[192:195], v[12:15], v[112:115]\n" "ds_read_b64_tr_b8 v[222:223], v229 offset:58368\n" "s_waitcnt lgkmcnt(15)\n"
    "v_mfma_i32_16x16x64_i8 v[116:119], v[196:199], v[12:15], v[116:119]\n" "v_xor_b32 v228, 0x10000, v228\n" "ds_read_b64_tr_b8 v[192:193], v228 offset:0\n"
    "v_mfma_i32_16x16x64_i8 v[84:87], v[196:199], v[40:43], v[84:87]\n" "ds_read_b64_tr_b8 v[194:195], v228 offset:1024\n" "s_waitcnt lgkmcnt(15)\n"
    "v_mfma_i32_16x16x64_i8 v[80:83], v[200:203], v[44:47], v[80:83]\n" "v_xor_b32 v229, 0x10000, v229\n" "ds_read_b64_tr_b8 v[196:197], v229 offset:0\n"
    "v_mfma_i32_16x16x64_i8 v[112:115], v[200:203], v[16:19], v[112:115]\n" "ds_read_b64_tr_b8 v[198:199], v229 offset:1024\n" "s_waitcnt lgkmcnt(14)\n"
    "v_mfma_i32_16x16x64_i8 v[116:119], v[204:207], v[16:19], v[116:119]\n" "ds_read_b64_tr_b8 v[200:201], v228 offset:8192\n" "v_mfma_i32_16x16x64_i8 v[84:87], v[204:207], v[44:47], v[84:87]\n"
    "ds_read_b64_tr_b8 v[202:203], v228 offset:9216\n" "s_waitcnt lgkmcnt(12)\n" "v_mfma_i32_16x16x64_i8 v[80:83], v[208:211], v[180:183], v[80:83]\n" "ds_read_b64_tr_b8 v[204:205], v229 offset:8192\n"
    "v_mfma_i32_16x16x64_i8 v[112:115], v[208:211], v[184:187], v[112:115]\n" "ds_read_b64_tr_b8 v[206:207], v229 offset:9216\n" "s_waitcnt lgkmcnt(12)\n"
    "v_mfma_i32_16x16x64_i8 v[116:119], v[212:215], v[184:187], v[116:119]\n" "ds_read_b64_tr_b8 v[208:209], v228 offset:16384\n" "v_mfma_i32_16x16x64_i8 v[84:87], v[212:215], v[180:183], v[84:87]\n"
    "ds_read_b64_tr_b8 v[210:211], v228 offset:17408\n" "s_waitcnt lgkmcnt(12)\n" "v_mfma_i32_16x16x64_i8 v[80:83], v[216:219], v[48:51], v[80:83]\n" "ds_read_b64_tr_b8 v[212:213], v229 offset:16384\n"
    "v_mfma_i32_16x16x64_i8 v[112:115], v[216:219], v[20:23], v[112:115]\n" "ds_read_b64_tr_b8 v[214:215], v229 offset:17408\n" "s_waitcnt lgkmcnt(12)\n"
    "v_mfma_i32_16x16x64_i8 v[116:119], v[220:223], v[20:23], v[116:119]\n" "ds_read_b64_tr_b8 v[216:217], v228 offset:24576\n" "v_mfma_i32_16x16x64_i8 v[84:87], v[220:223], v[48:51], v[84:87]\n"
    "ds_read_b64_tr_b8 v[218:219], v228 offset:25600\n" "s_waitcnt lgkmcnt(12)\n" "v_mfma_i32_16x16x64_i8 v[80:83], v[192:195], v[128:131], v[80:83]\n"
    "ds_read_b64_tr_b8 v[220:221], v229 offset:24576\n" "v_mfma_i32_16x16x64_i8 v[112:115], v[192:195], v[188:191], v[112:115]\n" "ds_read_b64_tr_b8 v[222:223], v229 offset:25600\n"
    "s_waitcnt lgkmcnt(12)\n" "v_mfma_i32_16x16x64_i8 v[116:119], v[196:199], v[188:191], v[116:119]\n" "ds_read_b64_tr_b8 v[192:193], v228 offset:32768\n"
    "v_mfma_i32_16x16x64_i8 v[84:87], v[196:199], v[128:131], v[84:87]\n" "ds_read_b64_tr_b8 v[194:195], v228 offset:33792\n" "s_waitcnt lgkmcnt(12)\n"
    "v_mfma_i32_16x16x64_i8 v[80:83], v[200:203], v[52:55], v[80:83]\n" "ds_read_b64_tr_b8 v[196:197], v229 offset:32768\n" "v_mfma_i32_16x16x64_i8 v[112:115], v[200:203], v[132:135], v[112:115]\n"
    "ds_read_b64_tr_b8 v[198:199], v229 offset:33792\n" "s_waitcnt lgkmcnt(12)\n" "v_mfma_i32_16x16x64_i8 v[116:119], v[204:207], v[132:135], v[116:119]\n"
    "ds_read_b64_tr_b8 v[200:201], v228 offset:40960\n" "v_mfma_i32_16x16x64_i8 v[84:87], v[204:207], v[52:55], v[84:87]\n" "ds_read_b64_tr_b8 v[202:203], v228 offset:41984\n" "s_waitcnt lgkmcnt(12)\n"
    "v_mfma_i32_16x16x64_i8 v[80:83], v[208:211], v[136:139], v[80:83]\n" "ds_read_b64_tr_b8 v[204:205], v229 offset:40960\n" "v_mfma_i32_16x16x64_i8 v[112:115], v[208:211], v[140:143], v[112:115]\n"
    "ds_read_b64_tr_b8 v[206:207], v229 offset:41984\n" "s_waitcnt lgkmcnt(12)\n" "v_mfma_i32_16x16x64_i8 v[116:119], v[212:215], v[140:143], v[116:119]\n"
    "ds_read_b64_tr_b8 v[208:209], v228 offset:49152\n" "v_mfma_i32_16x16x64_i8 v[84:87], v[212:215], v[136:139], v[84:87]\n" "ds_read_b64_tr_b8 v[210:211], v228 offset:50176\n"
    "s_waitcnt lgkmcnt(12)\n" "v_mfma_i32_16x16x64_i8 v[80:83], v[216:219], v[232:235], v[80:83]\n" "ds_read_b64_tr_b8 v[212:213], v229 offset:49152\n"
    "v_mfma_i32_16x16x64_i8 v[112:115], v[216:219], v[144:147], v[112:115]\n" "ds_read_b64_tr_b8 v[214:215], v229 offset:50176\n" "s_waitcnt lgkmcnt(12)\n"
    "v_mfma_i32_16x16x64_i8 v[116:119], v[220:223], v[144:147], v[116:119]\n" "ds_read_b64_tr_b8 v[216:217], v228 offset:57344\n" "v_mfma_i32_16x16x64_i8 v[84:87], v[220:223], v[232:235], v[84:87]\n"
    "ds_read_b64_tr_b8 v[218:219], v228 offset:58368\n" "s_waitcnt lgkmcnt(12)\n" "v_mfma_i32_16x16x64_i8 v[80:83], v[192:195], v[236:239], v[80:83]\n"
    "ds_read_b64_tr_b8 v[220:221], v229 offset:57344\n" "v_mfma_i32_16x16x64_i8 v[112:115], v[192:195], v[148:151], v[112:115]\n" "ds_read_b64_tr_b8 v[222:223], v229 offset:58368\n"
    "s_waitcnt lgkmcnt(12)\n" "v_mfma_i32_16x16x64_i8 v[116:119], v[196:199], v[148:151], v[116:119]\n" "v_xor_b32 v230, 0x10000, v230\n" "ds_read_b64_tr_b8 v[192:193], v230 offset:0\n"
    "v_mfma_i32_16x16x64_i8 v[84:87], v[196:199], v[236:239], v[84:87]\n" "ds_read_b64_tr_b8 v[194:195], v230 offset:1024\n" "s_waitcnt lgkmcnt(12)\n"
    "v_mfma_i32_16x16x64_i8 v[80:83], v[200:203], v[252:255], v[80:83]\n" "v_xor_b32 v231, 0x10000, v231\n" "ds_read_b64_tr_b8 v[196:197], v231 offset:0\n"
    "v_mfma_i32_16x16x64_i8 v[112:115], v[200:203], v[152:155], v[112:115]\n" "ds_read_b64_tr_b8 v[198:199], v231 offset:1024\n" "s_waitcnt lgkmcnt(12)\n"
    "v_mfma_i32_16x16x64_i8 v[116:119], v[204:207], v[152:155], v[116:119]\n" "ds_read_b64_tr_b8 v[200:201], v230 offset:8192\n" "ds_read_b32 v247, v242 offset:32\n"
    "v_mfma_i32_16x16x64_i8 v[84:87], v[204:207], v[252:255], v[84:87]\n" "ds_read_b64_tr_b8 v[202:203], v230 offset:9216\n" "s_waitcnt lgkmcnt(13)\n"
    "v_mfma_i32_16x16x64_i8 v[80:83], v[208:211], v[156:159], v[80:83]\n" "ds_read_b64_tr_b8 v[204:205], v231 offset:8192\n" "v_mfma_i32_16x16x64_i8 v[112:115], v[208:211], v[160:163], v[112:115]\n"
    "ds_read_b64_tr_b8 v[206:207], v231 offset:9216\n" "s_waitcnt lgkmcnt(13)\n" "v_mfma_i32_16x16x64_i8 v[116:119], v[212:215], v[160:163], v[116:119]\n"
    "ds_read_b64_tr_b8 v[208:209], v230 offset:16384\n" "v_mfma_i32_16x16x64_i8 v[84:87], v[212:215], v[156:159], v[84:87]\n" "ds_read_b64_tr_b8 v[210:211], v230 offset:17408\n"
    "s_waitcnt lgkmcnt(13)\n" "v_mfma_i32_16x16x64_i8 v[80:83], v[216:219], v[164:167], v[80:83]\n" "ds_read_b64_tr_b8 v[212:213], v231 offset:16384\n"
    "v_mfma_i32_16x16x64_i8 v[112:115], v[216:219], v[24:27], v[112:115]\n" "ds_read_b64_tr_b8 v[214:215], v231 offset:17408\n" "s_waitcnt lgkmcnt(13)\n"
    "v_mfma_i32_16x16x64_i8 v[116:119], v[220:223], v[24:27], v[116:119]\n" "ds_read_b64_tr_b8 v[216:217], v230 offset:24576\n" "s_waitcnt lgkmcnt(8)\n" "v_readfirstlane_b32 s54, v247\n"
    "v_mfma_i32_16x16x64_i8 v[84:87], v[220:223], v[164:167], v[84:87]\n" "ds_read_b64_tr_b8 v[218:219], v230 offset:25600\n" "v_mfma_i32_16x16x64_i8 v[88:91], v[192:195], v[168:171], v[88:91]\n"
    "ds_read_b64_tr_b8 v[220:221], v231 offset:24576\n" "v_mfma_i32_16x16x64_i8 v[120:123], v[192:195], v[172:175], v[120:123]\n" "ds_read_b64_tr_b8 v[222:223], v231 offset:25600\n"
    "s_andn2_b32 s55, 32, s54\n" "s_cbranch_scc0 L50_%=\n" "L49_%=:\n" "ds_read_b32 v251, v242 offset:32\n" "s_waitcnt lgkmcnt(0)\n" "v_readfirstlane_b32 s54, v251\n" "s_nop 3\n"
    "s_andn2_b32 s55, 32, s54\n" "s_cbranch_scc0 L50_%=\n" "s_sleep 1\n" "s_branch L49_%=\n" "L50_%=:\n" "v_mfma_i32_16x16x64_i8 v[124:127], v[196:199], v[172:175], v[124:127]\n"
    "ds_read_b64_tr_b8 v[192:193], v230 offset:32768\n" "v_mfma_i32_16x16x64_i8 v[92:95], v[196:199], v[168:171], v[92:95]\n" "ds_read_b64_tr_b8 v[194:195], v230 offset:33792\n"
    "ds_write_b128 v245, v[80:83] offset:0\n" "s_waitcnt lgkmcnt(13)\n" "v_mfma_i32_16x16x64_i8 v[88:91], v[200:203], v[176:179], v[88:91]\n" "ds_read_b64_tr_b8 v[196:197], v231 offset:32768\n"
    "v_mfma_i32_16x16x64_i8 v[120:123], v[200:203], v[0:3], v[120:123]\n" "ds_read_b64_tr_b8 v[198:199], v231 offset:33792\n" "ds_write_b128 v245, v[112:115] offset:2048\n" "s_waitcnt lgkmcnt(14)\n"
    "v_mfma_i32_16x16x64_i8 v[124:127], v[204:207], v[0:3], v[124:127]\n" "ds_read_b64_tr_b8 v[200:201], v230 offset:40960\n" "v_mfma_i32_16x16x64_i8 v[92:95], v[204:207], v[176:179], v[92:95]\n"
    "ds_read_b64_tr_b8 v[202:203], v230 offset:41984\n" "ds_write_b128 v246, v[84:87] offset:0\n" "s_waitcnt lgkmcnt(15)\n" "v_mfma_i32_16x16x64_i8 v[88:91], v[208:211], v[32:35], v[88:91]\n"
    "ds_read_b64_tr_b8 v[204:205], v231 offset:40960\n" "v_mfma_i32_16x16x64_i8 v[120:123], v[208:211], v[4:7], v[120:123]\n" "ds_read_b64_tr_b8 v[206:207], v231 offset:41984\n"
    "ds_write_b128 v246, v[116:119] offset:2048\n" "v_mov_b32 v251, 7\n" "ds_write_b32 v242, v251 offset:16\n" "s_waitcnt lgkmcnt(15)\n"
    "v_mfma_i32_16x16x64_i8 v[124:127], v[212:215], v[4:7], v[124:127]\n" "ds_read_b64_tr_b8 v[208:209], v230 offset:49152\n" "v_mfma_i32_16x16x64_i8 v[92:95], v[212:215], v[32:35], v[92:95]\n"
    "ds_read_b64_tr_b8 v[210:211], v230 offset:50176\n" "v_mfma_i32_16x16x64_i8 v[88:91], v[216:219], v[36:39], v[88:91]\n" "ds_read_b64_tr_b8 v[212:213], v231 offset:49152\n"
    "v_mfma_i32_16x16x64_i8 v[120:123], v[216:219], v[8:11], v[120:123]\n" "ds_read_b64_tr_b8 v[214:215], v231 offset:50176\n" "s_waitcnt lgkmcnt(15)\n"
    "v_mfma_i32_16x16x64_i8 v[124:127], v[220:223], v[8:11], v[124:127]\n" "ds_read_b64_tr_b8 v[216:217], v230 offset:57344\n" "v_mfma_i32_16x16x64_i8 v[92:95], v[220:223], v[36:39], v[92:95]\n"
    "ds_read_b64_tr_b8 v[218:219], v230 offset:58368\n" "v_mfma_i32_16x16x64_i8 v[88:91], v[192:195], v[40:43], v[88:91]\n" "ds_read_b64_tr_b8 v[220:221], v231 offset:57344\n"
    "v_mfma_i32_16x16x64_i8 v[120:123], v[192:195], v[12:15], v[120:123]\n" "ds_read_b64_tr_b8 v[222:223], v231 offset:58368\n" "s_waitcnt lgkmcnt(15)\n"
    "v_mfma_i32_16x16x64_i8 v[124:127], v[196:199], v[12:15], v[124:127]\n" "v_xor_b32 v230, 0x10000, v230\n" "ds_read_b64_tr_b8 v[192:193], v230 offset:0\n"
    "v_mfma_i32_16x16x64_i8 v[92:95], v[196:199], v[40:43], v[92:95]\n" "ds_read_b64_tr_b8 v[194:195], v230 offset:1024\n" "s_waitcnt lgkmcnt(15)\n"
    "v_mfma_i32_16x16x64_i8 v[88:91], v[200:203], v[44:47], v[88:91]\n" "v_xor_b32 v231, 0x10000, v231\n" "ds_read_b64_tr_b8 v[196:197], v231 offset:0\n"
    "v_mfma_i32_16x16x64_i8 v[120:123], v[200:203], v[16:19], v[120:123]\n" "ds_read_b64_tr_b8 v[198:199], v231 offset:1024\n" "s_waitcnt lgkmcnt(14)\n"
    "v_mfma_i32_16x16x64_i8 v[124:127], v[204:207], v[16:19], v[124:127]\n" "ds_read_b64_tr_b8 v[200:201], v230 offset:8192\n" "v_mfma_i32_16x16x64_i8 v[92:95], v[204:207], v[44:47], v[92:95]\n"
    "ds_read_b64_tr_b8 v[202:203], v230 offset:9216\n" "s_waitcnt lgkmcnt(12)\n" "v_mfma_i32_16x16x64_i8 v[88:91], v[208:211], v[180:183], v[88:91]\n" "ds_read_b64_tr_b8 v[204:205], v231 offset:8192\n"
    "v_mfma_i32_16x16x64_i8 v[120:123], v[208:211], v[184:187], v[120:123]\n" "ds_read_b64_tr_b8 v[206:207], v231 offset:9216\n" "s_waitcnt lgkmcnt(12)\n"
    "v_mfma_i32_16x16x64_i8 v[124:127], v[212:215], v[184:187], v[124:127]\n" "ds_read_b64_tr_b8 v[208:209], v230 offset:16384\n" "v_mfma_i32_16x16x64_i8 v[92:95], v[212:215], v[180:183], v[92:95]\n"
    "ds_read_b64_tr_b8 v[210:211], v230 offset:17408\n" "s_waitcnt lgkmcnt(12)\n" "v_mfma_i32_16x16x64_i8 v[88:91], v[216:219], v[48:51], v[88:91]\n" "ds_read_b64_tr_b8 v[212:213], v231 offset:16384\n"
    "v_mfma_i32_16x16x64_i8 v[120:123], v[216:219], v[20:23], v[120:123]\n" "ds_read_b64_tr_b8 v[214:215], v231 offset:17408\n" "s_waitcnt lgkmcnt(12)\n"
    "v_mfma_i32_16x16x64_i8 v[124:127], v[220:223], v[20:23], v[124:127]\n" "ds_read_b64_tr_b8 v[216:217], v230 offset:24576\n" "v_mfma_i32_16x16x64_i8 v[92:95], v[220:223], v[48:51], v[92:95]\n"
    "ds_read_b64_tr_b8 v[218:219], v230 offset:25600\n" "s_waitcnt lgkmcnt(12)\n" "v_mfma_i32_16x16x64_i8 v[88:91], v[192:195], v[128:131], v[88:91]\n"
    "ds_read_b64_tr_b8 v[220:221], v231 offset:24576\n" "v_mfma_i32_16x16x64_i8 v[120:123], v[192:195], v[188:191], v[120:123]\n" "ds_read_b64_tr_b8 v[222:223], v231 offset:25600\n"
    "s_waitcnt lgkmcnt(12)\n" "v_mfma_i32_16x16x64_i8 v[124:127], v[196:199], v[188:191], v[124:127]\n" "ds_read_b64_tr_b8 v[192:193], v230 offset:32768\n"
    "v_mfma_i32_16x16x64_i8 v[92:95], v[196:199], v[128:131], v[92:95]\n" "ds_read_b64_tr_b8 v[194:195], v230 offset:33792\n" "s_waitcnt lgkmcnt(12)\n"
    "v_mfma_i32_16x16x64_i8 v[88:91], v[200:203], v[52:55], v[88:91]\n" "ds_read_b64_tr_b8 v[196:197], v231 offset:32768\n" "v_mfma_i32_16x16x64_i8 v[120:123], v[200:203], v[132:135], v[120:123]\n"
    "ds_read_b64_tr_b8 v[198:199], v231 offset:33792\n" "s_waitcnt lgkmcnt(12)\n" "v_mfma_i32_16x16x64_i8 v[124:127], v[204:207], v[132:135], v[124:127]\n"
    "ds_read_b64_tr_b8 v[200:201], v230 offset:40960\n" "v_mfma_i32_16x16x64_i8 v[92:95], v[204:207], v[52:55], v[92:95]\n" "ds_read_b64_tr_b8 v[202:203], v230 offset:41984\n" "s_waitcnt lgkmcnt(12)\n"
    "v_mfma_i32_16x16x64_i8 v[88:91], v[208:211], v[136:139], v[88:91]\n" "ds_read_b64_tr_b8 v[204:205], v231 offset:40960\n" "v_mfma_i32_16x16x64_i8 v[120:123], v[208:211], v[140:143], v[120:123]\n"
    "ds_read_b64_tr_b8 v[206:207], v231 offset:41984\n" "s_waitcnt lgkmcnt(12)\n" "v_mfma_i32_16x16x64_i8 v[124:127], v[212:215], v[140:143], v[124:127]\n"
    "ds_read_b64_tr_b8 v[208:209], v230 offset:49152\n" "v_mfma_i32_16x16x64_i8 v[92:95], v[212:215], v[136:139], v[92:95]\n" "ds_read_b64_tr_b8 v[210:211], v230 offset:50176\n"
    "s_waitcnt lgkmcnt(12)\n" "v_mfma_i32_16x16x64_i8 v[88:91], v[216:219], v[232:235], v[88:91]\n" "ds_read_b64_tr_b8 v[212:213], v231 offset:49152\n"
    "v_mfma_i32_16x16x64_i8 v[120:123], v[216:219], v[144:147], v[120:123]\n" "ds_read_b64_tr_b8 v[214:215], v231 offset:50176\n" "s_waitcnt lgkmcnt(12)\n"
    "v_mfma_i32_16x16x64_i8 v[124:127], v[220:223], v[144:147], v[124:127]\n" "ds_read_b64_tr_b8 v[216:217], v230 offset:57344\n" "v_mfma_i32_16x16x64_i8 v[92:95], v[220:223], v[232:235], v[92:95]\n"
    "ds_read_b64_tr_b8 v[218:219], v230 offset:58368\n" "s_waitcnt lgkmcnt(12)\n" "v_mfma_i32_16x16x64_i8 v[88:91], v[192:195], v[236:239], v[88:91]\n"
    "ds_read_b64_tr_b8 v[220:221], v231 offset:57344\n" "v_mfma_i32_16x16x64_i8 v[120:123], v[192:195], v[148:151], v[120:123]\n" "ds_read_b64_tr_b8 v[222:223], v231 offset:58368\n"
    "s_waitcnt lgkmcnt(12)\n" "v_mfma_i32_16x16x64_i8 v[124:127], v[196:199], v[148:151], v[124:127]\n" "v_mfma_i32_16x16x64_i8 v[92:95], v[196:199], v[236:239], v[92:95]\n" "s_waitcnt lgkmcnt(10)\n"
    "v_mfma_i32_16x16x64_i8 v[88:91], v[200:203], v[252:255], v[88:91]\n" "ds_read_b32 v247, v242 offset:32\n" "v_mfma_i32_16x16x64_i8 v[120:123], v[200:203], v[152:155], v[120:123]\n"
    "s_waitcnt lgkmcnt(9)\n" "v_mfma_i32_16x16x64_i8 v[124:127], v[204:207], v[152:155], v[124:127]\n" "v_mfma_i32_16x16x64_i8 v[92:95], v[204:207], v[252:255], v[92:95]\n" "s_waitcnt lgkmcnt(7)\n"
    "v_mfma_i32_16x16x64_i8 v[88:91], v[208:211], v[156:159], v[88:91]\n" "v_mfma_i32_16x16x64_i8 v[120:123], v[208:211], v[160:163], v[120:123]\n" "s_waitcnt lgkmcnt(5)\n"
    "v_mfma_i32_16x16x64_i8 v[124:127], v[212:215], v[160:163], v[124:127]\n" "v_mfma_i32_16x16x64_i8 v[92:95], v[212:215], v[156:159], v[92:95]\n" "s_waitcnt lgkmcnt(3)\n"
    "v_mfma_i32_16x16x64_i8 v[88:91], v[216:219], v[164:167], v[88:91]\n" "s_waitcnt lgkmcnt(0)\n" "v_readfirstlane_b32 s54, v247\n"
    "v_mfma_i32_16x16x64_i8 v[120:123], v[216:219], v[24:27], v[120:123]\n" "v_mfma_i32_16x16x64_i8 v[124:127], v[220:223], v[24:27], v[124:127]\n"
    "v_mfma_i32_16x16x64_i8 v[92:95], v[220:223], v[164:167], v[92:95]\n" "s_andn2_b32 s55, 64, s54\n" "s_cbranch_scc0 L52_%=\n" "L51_%=:\n" "ds_read_b32 v251, v242 offset:32\n"
    "s_waitcnt lgkmcnt(0)\n" "v_readfirstlane_b32 s54, v251\n" "s_nop 3\n" "s_andn2_b32 s55, 64, s54\n" "s_cbranch_scc0 L52_%=\n" "s_sleep 1\n" "s_branch L51_%=\n" "L52_%=:\n" "s_nop 0\n"
    "ds_write_b128 v245, v[88:91] offset:0\n" "s_nop 1\n" "ds_write_b128 v245, v[120:123] offset:2048\n" "s_nop 0\n" "ds_write_b128 v246, v[92:95] offset:0\n"
    "ds_write_b128 v246, v[124:127] offset:2048\n" "v_mov_b32 v251, 8\n" "ds_write_b32 v242, v251 offset:16\n" ASTAMP(s_memrealtime, 16) ASTAMP(s_memtime, 40) "s_waitcnt lgkmcnt(0)\n" "s_endpgm\n"
    :: "{s[40:43]}"(srdX), "{s[44:45]}"(wsb), "{s46}"(xa), "{s[48:49]}"(dbg), "{v224}"(a0), "{v240}"(voffX), "{v241}"(lb + L_RDY),
       "{v242}"(lb + L_PROG + wid * 4), "{v245}"(aw0), "{v248}"(dvo), "{v249}"(xa0)
    : "memory", "s59", "v255");
}

extern "C" void kernel_launch(int rank, int nranks, ncclComm_t comm, hipStream_t stream,
                              void* const* d_in, const int* in_sizes, int n_in,
                              void* d_out, int out_size, const int* in_dtypes, int out_dtype,
                              void* const* d_out_all, void* const* d_ws_all, size_t ws_bytes) {
  if (nranks != NR || (unsigned)rank >= (unsigned)NR || n_in < 4 || ws_bytes < WS_NEED
      || in_sizes[0] != KK * KL || in_sizes[1] != KK * NN || out_size != ML * NN) {
    fprintf(stderr, "kernel_launch: unexpected shapes\n");
    return;
  }
  Args a;
  for (int p = 0; p < NR; ++p) a.ws[p] = (char*)d_ws_all[(rank + p) % NR];
  a2a_gemm<<<256, 512, 0, stream>>>((const int8_t*)d_in[0], (const int8_t*)d_in[1], (const float*)d_in[2], (const float*)d_in[3],
                                    (float*)d_out, a, (unsigned)rank);
  HIP_CHECK(hipGetLastError());
}
```
